```python
import numpy as np

kernel_name = "gpu_gemm_rs_m8192_k8192_n8192_bf16_relu"


ABSMAX_RTOL = 0.008

COMPUTE_DTYPE = "bfloat16"
EPILOGUE = "relu"
IN_QUANT = None
OUT_QUANT = None
IN_DTYPES = {"x": "bfloat16", "w_mat": "bfloat16"}
WORLD_SIZE = 8

def _to_bf16(x):
    u = np.ascontiguousarray(x, dtype=np.float32).view(np.uint32)
    return ((u + 0x7FFF + ((u >> 16) & 1)) & 0xFFFF0000).view(np.float32)

def setup_inputs():

    x = _to_bf16(np.random.randn(8192, 8192).astype(np.float32))
    w_mat = _to_bf16((np.random.randn(8192, 8192) / np.sqrt(8192)).astype(np.float32))
    return {"x": x, "w_mat": w_mat}

def _epilogue(y):
    return np.maximum(y, 0.0).astype(np.float32)

def reference(x, w_mat):
    y = (x @ w_mat).astype(np.float32)
    return _epilogue(y)
```

```cpp
#include <hip/hip_runtime.h>
#include <rccl/rccl.h>
#include <cstdio>

#define HIP_CHECK(expr) do { hipError_t _e = (expr); if (_e != hipSuccess) { \
    fprintf(stderr, "HIP error %s at %s:%d\n", hipGetErrorString(_e), __FILE__, __LINE__); \
    return; } } while (0)

#define LAS __attribute__((address_space(3)))
#define GAS __attribute__((address_space(1)))
typedef short bf16x8 __attribute__((ext_vector_type(8)));
typedef short s16x4 __attribute__((ext_vector_type(4)));
typedef short s16x2 __attribute__((ext_vector_type(2)));
typedef unsigned short u16x2 __attribute__((ext_vector_type(2)));
typedef __bf16 bf16x2 __attribute__((ext_vector_type(2)));
typedef float f32x2 __attribute__((ext_vector_type(2)));
typedef float f32x4 __attribute__((ext_vector_type(4)));
typedef unsigned u32x4 __attribute__((ext_vector_type(4)));
typedef unsigned long long u64;
typedef __amdgpu_buffer_rsrc_t rsrc_t;

#define NC1 2
#define NC2 2
#define NRED 4
#define J1 4
#define PAUSE2 15
static_assert(2 * (PAUSE2 - 7) <= 6 * J1, "shift 2 may only depend on queue 1 before it starts pulling");
static_assert(NC1 + NC2 + NRED <= 32 && NC1 >= 1 && NC1 <= 8, "roles");

constexpr int NWG = 256, HTB = 16384;
constexpr unsigned SA_ = 2048, SB_ = 16384;
constexpr size_t MiB = 1ull << 20;
constexpr size_t OFF_CALL = 0;
constexpr size_t OFF_TMO = 32768, OFF_Q1 = OFF_TMO + 128, OFF_Q2 = OFF_TMO + 256, OFF_QO = OFF_TMO + 384;
constexpr size_t OFF_RDY = 40960;
constexpr size_t OFF_HIN = 65536;
constexpr size_t OFF_HOUT = OFF_HIN + 524288;
constexpr size_t OFF_STGX = 1 * MiB, OFF_STGW = 25 * MiB;
constexpr size_t OFF_PIN = 49 * MiB;
constexpr size_t OFF_PINC = 97 * MiB;
constexpr size_t OFF_POUT = 129 * MiB;
constexpr size_t OFF_POUTC = 177 * MiB;
constexpr size_t OFF_DBG = 209 * MiB;
constexpr size_t WS_NEED = 210 * MiB;
constexpr unsigned NIN = 1536, NOUT = 1024, TH1 = 192 * J1;
constexpr unsigned W1 = NC1 * 64, W2 = NC2 * 64, WR = NRED * 64;
constexpr int LDS_BYTES = 8 * HTB + 1024;
constexpr u64 DEADLINE = 100000000ull;

struct Params { const char* xin; const char* win; char* out; char* ws[8]; int rank; int pad; };

#define RFL(x) __builtin_amdgcn_readfirstlane(x)
#define WAIT_V(n) asm volatile("s_waitcnt vmcnt(" #n ")" ::: "memory")
#define WAIT_L(n) asm volatile("s_waitcnt lgkmcnt(" #n ")" ::: "memory")
#define BAR __builtin_amdgcn_s_barrier()
#define SCHED __builtin_amdgcn_sched_barrier(0)
#define FENCE asm volatile("" ::: "memory")
#define DI __device__ __forceinline__
constexpr int SYS = 17;

DI rsrc_t mk(const void* p, unsigned bytes) { return __builtin_amdgcn_make_buffer_rsrc((void*)p, (short)0, (int)bytes, 0x00020000); }
DI u32x4 ld16(rsrc_t r, unsigned v, unsigned s) { return __builtin_amdgcn_raw_buffer_load_b128(r, v, s, SYS); }
DI u32x4 ld16p(rsrc_t r, unsigned v, unsigned s) { return __builtin_amdgcn_raw_buffer_load_b128(r, v, s, 0); }
DI void st16(u32x4 x, rsrc_t r, unsigned v, unsigned s) { __builtin_amdgcn_raw_buffer_store_b128(x, r, v + s, 0, SYS); }
DI unsigned aload(const char* p) { return __hip_atomic_load((const GAS unsigned*)p, __ATOMIC_RELAXED, __HIP_MEMORY_SCOPE_SYSTEM); }
DI void astore(char* p, unsigned v) { __hip_atomic_store((GAS unsigned*)p, v, __ATOMIC_RELAXED, __HIP_MEMORY_SCOPE_SYSTEM); }
DI unsigned perm(unsigned hi, unsigned lo, unsigned sel) { return __builtin_amdgcn_perm(hi, lo, sel); }
DI unsigned cvt2(float lo, float hi) { return __builtin_bit_cast(unsigned, __builtin_convertvector((f32x2){lo, hi}, bf16x2)); }

#define STAMP(i) do { } while (0)

DI bool give_up(char* ws, unsigned call, u64 t0) {
  if (RFL(aload(ws + OFF_TMO)) == call) return true;
  if (__builtin_amdgcn_s_memrealtime() - t0 > DEADLINE) { astore(ws + OFF_TMO, call); return true; }
  __builtin_amdgcn_s_sleep(2);
  return false;
}

DI u32x4 pack_unit(const u32x4 (&r)[4], unsigned tag, int lane, rsrc_t dst, unsigned doff, rsrc_t dstc, unsigned coff) {
  u16x2 mx = {0, 0};
#pragma unroll
  for (int c = 0; c < 4; ++c)
#pragma unroll
    for (int i = 0; i < 4; ++i) mx = __builtin_elementwise_max(mx, __builtin_bit_cast(u16x2, r[c][i] & 0x7FFF7FFFu));
  const unsigned f = (mx.x > mx.y ? mx.x : mx.y) >> 8;
  unsigned F = 0;
#pragma unroll
  for (int bit = 64; bit; bit >>= 1) F = __builtin_amdgcn_ballot_w64(f >= (F | bit)) ? (F | bit) : F;
  const unsigned base = F > 7 ? F - 7 : 0, B4 = base * 0x01010101u;
  unsigned A[8], H[8], T[8], P[4], e = 0;
#pragma unroll
  for (int c = 0; c < 4; ++c) {
    A[2 * c] = perm(r[c][1], r[c][0], 0x06040200u); A[2 * c + 1] = perm(r[c][3], r[c][2], 0x06040200u);
    H[2 * c] = perm(r[c][1], r[c][0], 0x07050301u); H[2 * c + 1] = perm(r[c][3], r[c][2], 0x07050301u);
    T[2 * c] = (H[2 * c] | 0x80808080u) - B4; T[2 * c + 1] = (H[2 * c + 1] | 0x80808080u) - B4;
    e |= T[2 * c] | T[2 * c + 1];
    P[c] = (H[2 * c] & 0x80808080u) | (T[2 * c] & 0x07070707u) | ((T[2 * c + 1] & 0x07070707u) << 3) | ((H[2 * c + 1] & 0x80808080u) >> 1);
  }
  st16((u32x4){A[0], A[1], A[2], A[3]}, dst, lane * 16, doff);
  st16((u32x4){A[4], A[5], A[6], A[7]}, dst, lane * 16 + 1024, doff);
  st16((u32x4){P[0], P[1], P[2], P[3]}, dst, lane * 16 + 2048, doff);
  unsigned w0 = (tag << 8) | base; u64 pt = 0;
  if (__builtin_amdgcn_ballot_w64((e & 0x78787878u) != 0)) {
    unsigned X[8]; int cnt = 0; unsigned code = 0;
#pragma unroll
    for (int k = 7; k >= 0; --k) {
      X[k] = (H[k] & 0x7F7F7F7Fu) ^ ((T[k] & 0x07070707u) + B4);
      const unsigned nz = ((X[k] >> 3) + 0x0F0F0F0Fu) & 0x10101010u;
      cnt += __builtin_popcount(nz);
      if (X[k]) { const unsigned by = (unsigned)__builtin_ctz(X[k]) >> 3; code = ((unsigned)lane << 10) | (k << 7) | (by << 5) | ((X[k] >> (8 * by + 3)) & 15u); }
    }
    u64 m = __builtin_amdgcn_ballot_w64(cnt != 0);
    if (__builtin_amdgcn_ballot_w64(cnt > 1) || __builtin_popcountll(m) > 4) {
      w0 |= 0x80u;
      st16((u32x4){X[0], X[1], X[2], X[3]}, dstc, lane * 16, coff);
      st16((u32x4){X[4], X[5], X[6], X[7]}, dstc, lane * 16 + 1024, coff);
    } else {
      while (m) { const int l = __builtin_ctzll(m); m &= m - 1; pt = (pt << 16) | (unsigned)__builtin_amdgcn_readlane((int)code, l); }
    }
  }
  return (u32x4){w0, (unsigned)pt, w0, (unsigned)(pt >> 32)};
}
DI void unpack_hi(unsigned (&H)[8], u32x4 P, unsigned hw, unsigned p0, unsigned p1, int lane, rsrc_t srcc, unsigned coff) {
  const unsigned B4 = (hw & 0x7Fu) * 0x01010101u;
#pragma unroll
  for (int c = 0; c < 4; ++c) {
    H[2 * c] = (P[c] & 0x87878787u) + B4;
    H[2 * c + 1] = (((P[c] >> 3) & 0x07070707u) + B4) | ((P[c] << 1) & 0x80808080u);
  }
  if (p0 | (hw & 0x80u)) {
    if (hw & 0x80u) {
      const u32x4 x0 = ld16(srcc, lane * 16, coff), x1 = ld16(srcc, lane * 16 + 1024, coff);
#pragma unroll
      for (int k = 0; k < 4; ++k) { H[k] ^= x0[k]; H[4 + k] ^= x1[k]; }
    } else {
      u64 pt = ((u64)p1 << 32) | p0;
      while (pt) {
        const unsigned code = (unsigned)pt & 0xFFFFu; pt >>= 16;
        const unsigned dv = (unsigned)lane == (code >> 10) ? (code & 15u) << (8 * ((code >> 5) & 3u) + 3) : 0u, kk = (code >> 7) & 7u;
#pragma unroll
        for (int k = 0; k < 8; ++k) H[k] ^= dv & (kk == (unsigned)k ? ~0u : 0u);
      }
    }
  }
}
DI u32x4 wait_hdr(const char* hp, unsigned tag, char* ws, unsigned call) {
  const u64 t0 = __builtin_amdgcn_s_memrealtime();
  u32x4 h;
  for (;;) {
    asm volatile("global_load_dwordx4 %0, %1, off sc0 sc1\n\ts_waitcnt vmcnt(0)" : "=&v"(h) : "v"(hp) : "memory");
    const bool ok = (h[0] >> 8) == tag && (h[2] >> 8) == tag;
    if ((__builtin_amdgcn_ballot_w64(ok) & 0xFFull) == 0xFFull) break;
    if (give_up(ws, call, t0)) { h = (u32x4){0u, 0u, 0u, 0u}; break; }
  }
  return h;
}
DI unsigned claim(char* ctr, int lane) { unsigned v = 0; if (lane == 0) v = __hip_atomic_fetch_add((GAS unsigned*)ctr, 1u, __ATOMIC_RELAXED, __HIP_MEMORY_SCOPE_AGENT); return v; }
#define RL(x, l) ((unsigned)__builtin_amdgcn_readlane((int)(x), (l)))

__global__ __launch_bounds__(512, 2) void gemm_rs(Params p) {
  extern __shared__ __attribute__((aligned(16))) unsigned char smem[];
  LAS unsigned char* lds = (LAS unsigned char*)smem;
  const int tid = threadIdx.x, wid = RFL(tid >> 6), lane = tid & 63, wr = wid >> 2, wc = wid & 3, fr = lane & 15, fq = lane >> 4;
  const int rank = p.rank & 7, ra = rank >> 2, rb = (rank >> 1) & 1, rc = rank & 1, rcol = rank >> 1;
  char* const ws = p.ws[rank];
  const int bid = blockIdx.x, xcd = bid & 7, idx = bid >> 3;
  const int tm = (xcd >> 1) * 4 + (idx & 3), tn = (xcd & 1) * 8 + (idx >> 2);
  const int rho = (idx & ~3) | ((idx & 1) << 1) | ((idx >> 1) & 1);
  STAMP(0);
  const unsigned callv = aload(ws + OFF_CALL + bid * 128);
  const unsigned loX = (lane >> 3) * SA_ + (lane & 7) * 16, loW = (lane >> 5) * SB_ + (lane & 31) * 16;

  unsigned call, par, tag;
  {
    const rsrc_t rx = mk(p.xin, 16u << 20), rw = mk(p.win, 16u << 20);
    const int np = NWG - 8 * NC1, pid = (rho - NC1) * 8 + xcd;
    int blk[5]; bool val[5];
#pragma unroll
    for (int k = 0; k < 5; ++k) { const int g = k == 0 ? bid : 256 + pid + (k - 1) * np; val[k] = k == 0 || (rho >= NC1 && g < 1024);
      const int gg = val[k] ? g : bid, op = (gg >> 7) & 1, jj = ((gg >> 6) & 1) * 8 + (gg >> 8) * 2 + ((gg >> 5) & 1); blk[k] = op * 512 + jj * 32 + (gg & 31); }
    u32x4 r[5][4];
#define PK_LOAD(k) do { const int _op = blk[k] >> 9, _jj = (blk[k] >> 5) & 15, _ix = blk[k] & 31; \
      const unsigned _so = _op ? (_jj * 64 + wid * 8) * SB_ + _ix * 512 : (_ix * 256 + wid * 32) * SA_ + _jj * 128, _cs = _op ? 2 * SB_ : 8 * SA_; \
      _Pragma("unroll") for (int c = 0; c < 4; ++c) r[k][c] = ld16p(_op ? rw : rx, _op ? loW : loX, _so + c * _cs); } while (0)
#define PK_PACK(k) do { const u32x4 _h = pack_unit(r[k], tag, lane, rpi, blk[k] * 24576 + wid * 3072, rpc, blk[k] * 16384 + wid * 2048); \
      if (lane == 0) *(LAS u32x4*)(lds + 8 * HTB + (k) * 128 + wid * 16) = _h; } while (0)
#define PK_SEND(k) do { const int _op = blk[k] >> 9, _jj = (blk[k] >> 5) & 15, _ix = blk[k] & 31, _d = (lane >> 3) & 1; \
      const int _tau = _op ? (_ix & 15) : ((_ix >> 2) * 2 + (_ix & 1)); \
      const int _c0 = _op ? (_ix >> 4) : (_ix & 2), _c1 = _op ? _c0 + 2 : _c0 + 1; \
      char* const _w0 = p.ws[(_c0 * 2 + (_jj >> 3)) & 7]; char* const _w1 = p.ws[(_c1 * 2 + (_jj >> 3)) & 7]; \
      if (lane < 16 && (_d ? _c1 : _c0) != rcol) { const u32x4 _h = *(const LAS u32x4*)(lds + 8 * HTB + (k) * 128 + (lane & 7) * 16); \
        char* _dst = (_d ? _w1 : _w0) + OFF_HIN + (size_t)((((par * 8 + rank) * 2 + _op) * 8 + (_jj & 7)) * 16 + _tau) * 128 + (lane & 7) * 16; \
        asm volatile("global_store_dwordx4 %0, %1, off sc0 sc1\n\ts_nop 1" :: "v"(_dst), "v"(_h) : "memory"); } } while (0)
    PK_LOAD(0);
    call = RFL(callv) + 1; par = call & 1; tag = (call % 0xFFFFFFu) + 1;
    const rsrc_t rpi = mk(ws + OFF_PIN + (size_t)par * 24 * MiB, 24u << 20), rpc = mk(ws + OFF_PINC + (size_t)par * 16 * MiB, 16u << 20);
    PK_PACK(0);
    WAIT_V(0); WAIT_L(0); BAR; FENCE;
    if (wid == 0) PK_SEND(0);
    if (tid == 64) astore(ws + OFF_CALL + bid * 128, call);
    STAMP(1);
    if (rho >= NC1) {
      PK_LOAD(1); PK_LOAD(2); PK_LOAD(3); PK_LOAD(4); FENCE;
      PK_PACK(1); PK_PACK(2); PK_PACK(3); if (val[4]) PK_PACK(4);
      WAIT_V(0); WAIT_L(0); BAR; FENCE;
      if (wid == 1) PK_SEND(1);
      if (wid == 2) PK_SEND(2);
      if (wid == 3) PK_SEND(3);
      if (wid == 4 && val[4]) PK_SEND(4);
    }
  }
  STAMP(2);

  const unsigned ldsw = (unsigned)(size_t)lds + (unsigned)wid * 1024u;
  unsigned vA[4], vB[4];
  { const unsigned voffA = (wid * 8 + (lane >> 3)) * SA_ + 16 * ((lane & 7) ^ (lane >> 3));
    const int s = lane >> 3, pos = (lane & 7) >> 1, hf = lane & 1, cp = pos ^ (s >> 1);
    const unsigned voffB = (16 * (wid >> 2) + 8 * (s >> 2) + 4 * ((wid >> 1) & 1) + (s & 3)) * SB_ + (64 * (wid & 1) + 16 * cp + 8 * hf) * 2;
#pragma unroll
    for (int i = 0; i < 4; ++i) { vA[i] = voffA + i * 64 * SA_; vB[i] = voffB + (i & 1) * 32 * SB_ + (i >> 1) * 256; asm volatile("" : "+v"(vA[i]), "+v"(vB[i])); } }
  int aoff0 = (wr * 64 + fr) * 128 + 16 * (fq ^ (fr & 7)), aoff1 = aoff0 ^ 64, boff0, boff1;
  { const int s = (lane >> 2) & 7, pp = ((lane & 1) << 1) | ((lane >> 1) & 1), gh = lane >> 5;
    boff0 = 4 * HTB + gh * 4096 + (wc >> 1) * 1024 + 128 * s + 32 * ((2 * (wc & 1)) ^ (s >> 1)) + 8 * pp; boff1 = boff0 ^ 32; }
  asm volatile("" : "+v"(aoff0), "+v"(aoff1), "+v"(boff0), "+v"(boff1));
#define SA(b, h) (((b) * 2 + (h)) * HTB)
#define SB(b, h) ((4 + (b) * 2 + (h)) * HTB)
#define GLDS(voff, sbase, c) asm volatile("s_add_i32 m0, %2, %3\n\ts_nop 0\n\tglobal_load_lds_dwordx4 %0, %1 sc0 sc1" :: "v"(voff), "s"(sbase), "s"(ldsw), "i"(c) : "memory", "scc")
#define STAGEA(b, h, gbase) do { GLDS(vA[2 * (h)], gbase, SA(b, h)); GLDS(vA[2 * (h) + 1], gbase, SA(b, h) + 8192); } while (0)
#define STAGEB(b, h, gbase) do { GLDS(vB[2 * (h)], gbase, SB(b, h)); GLDS(vB[2 * (h) + 1], gbase, SB(b, h) + 8192); } while (0)
#define LDA(dst, b, h) do { _Pragma("unroll") for (int m = 0; m < 4; ++m) { dst[m][0] = *(const LAS bf16x8*)(lds + SA(b, h) + aoff0 + m * 2048); dst[m][1] = *(const LAS bf16x8*)(lds + SA(b, h) + aoff1 + m * 2048); } } while (0)
#define TR(off) __builtin_amdgcn_ds_read_tr16_b64_v4i16((LAS s16x4*)(lds + (off)))
#define LDB(dst, b, h) do { _Pragma("unroll") for (int k = 0; k < 2; ++k) { \
    dst[0][k] = __builtin_shufflevector(TR(SA(b, h) + boff0 + k * 8192), TR(SA(b, h) + boff0 + k * 8192 + 2048), 0, 1, 2, 3, 4, 5, 6, 7); \
    dst[1][k] = __builtin_shufflevector(TR(SA(b, h) + boff1 + k * 8192), TR(SA(b, h) + boff1 + k * 8192 + 2048), 0, 1, 2, 3, 4, 5, 6, 7); } } while (0)
#define MMA(ai, bj, At, Bt) do { __builtin_amdgcn_s_setprio(1); _Pragma("unroll") for (int m = 0; m < 4; ++m) _Pragma("unroll") for (int n = 0; n < 2; ++n) _Pragma("unroll") for (int k = 0; k < 2; ++k) \
    acc[ai][bj][m][n] = __builtin_amdgcn_mfma_f32_16x16x32_bf16(Bt[n][k], At[m][k], acc[ai][bj][m][n], 0, 0, 0); __builtin_amdgcn_s_setprio(0); } while (0)
  f32x4 acc[2][2][4][2];
#pragma unroll
  for (int a = 0; a < 2; ++a)
#pragma unroll
    for (int b = 0; b < 2; ++b)
#pragma unroll
      for (int m = 0; m < 4; ++m)
#pragma unroll
        for (int n = 0; n < 2; ++n) acc[a][b][m][n] = (f32x4){0.f, 0.f, 0.f, 0.f};
  bf16x8 At[4][2], B0[2][2], B1[2][2];
  const char* const ownA = p.xin + (size_t)((tm >> 1) * 1024 + ra * 512 + (tm & 1) * 256) * SA_;
  const char* const ownB = p.win + (size_t)(rb * 4096 + tn * 256) * 2;
  const char* const sA = ws + OFF_STGX + (size_t)tm * 256 * SA_;
  const char* const sB = ws + OFF_STGW + (size_t)tn * 512;
  const char* a1 = ownA + 128;
  const int pause = rho < NC1 ? 0 : (rho < NC1 + NC2 ? PAUSE2 : -1);
  STAGEB(0, 0, ownB); STAGEA(0, 0, ownA); STAGEB(0, 1, ownB); STAGEA(0, 1, ownA);
  if (wr == 1) BAR;
  WAIT_V(4); BAR;
  STAGEB(1, 0, ownB + 64 * SB_); STAGEA(1, 0, a1); STAGEB(1, 1, ownB + 64 * SB_);
  WAIT_V(6); BAR;
  int j = 0, s2 = 0, ready = 0;
  for (int it = 0; it < 32; ++it) {
    if (it == pause) {
      if (wr == 0) BAR;
      STAMP(3);
      int ln = tid; asm volatile("" : "+v"(ln)); ln &= 63;
      const unsigned lx = (ln >> 3) * SA_ + (ln & 7) * 16, lw = (ln >> 5) * SB_ + (ln & 31) * 16;
      for (int qi = 0; qi < (pause == 0 ? 1 : 2); ++qi) {
        char* const ctr = ws + (qi ? OFF_Q2 : OFF_Q1);
        const unsigned qb = qi ? (call - 1) * (NIN - TH1 + W2) - TH1 : (call - 1) * (TH1 + W1 + W2), qe = qi ? NIN : TH1;
        unsigned q = RFL(claim(ctr, ln)) - qb;
        while (q < qe) {
          const unsigned qn = claim(ctr, ln);
          const unsigned s = q % 6, t6 = q / 6, k16 = t6 & 15, op = (t6 >> 4) & 1, jq = t6 >> 5;
          const unsigned tau = op ? (((k16 & 1) << 3) | (7 - (k16 >> 1))) : k16;
          const unsigned src = ((((unsigned)rcol ^ (1 + (s >> 1))) << 1) | (s & 1)) & 7;
          const unsigned blk = op * 512 + (rc * 8 + jq) * 32 + (op ? rb * 16 + tau : (tau >> 1) * 4 + ra * 2 + (tau & 1));
          const char* const sw = p.ws[src];
          const u32x4 h = wait_hdr(ws + OFF_HIN + (size_t)((((par * 8 + src) * 2 + op) * 8 + jq) * 16 + tau) * 128 + (ln & 7) * 16, tag, ws, call);
          const rsrc_t rs = mk(sw + OFF_PIN + (size_t)par * 24 * MiB + (size_t)blk * 24576, 24576), rsc = mk(sw + OFF_PINC + (size_t)par * 16 * MiB + (size_t)blk * 16384, 16384);
          const rsrc_t rd = mk(ws + (op ? OFF_STGW : OFF_STGX), 24u << 20);
          const unsigned dbase = op ? ((s >> 1) * 512 + jq * 64) * SB_ + (s & 1) * 8192 + tau * 512 : ((s >> 1) * 4096 + tau * 256) * SA_ + (s & 1) * 1024 + jq * 128;
          const unsigned cs = op ? 2 * SB_ : 8 * SA_, lo = op ? lw : lx;
          constexpr int RING = 5;
          u32x4 d[RING][3];
#pragma unroll
          for (int u = 0; u < RING; ++u)
#pragma unroll
            for (int i = 0; i < 3; ++i) d[u][i] = ld16(rs, ln * 16 + i * 1024, u * 3072);
          FENCE;
#pragma unroll
          for (int u = 0; u < 8; ++u) {
            unsigned H[8];
            unpack_hi(H, d[u % RING][2], RL(h[0], u), RL(h[1], u), RL(h[3], u), ln, rsc, u * 2048);
#pragma unroll
            for (int c = 0; c < 4; ++c) {
              const unsigned a0 = d[u % RING][c >> 1][(c & 1) * 2], a1_ = d[u % RING][c >> 1][(c & 1) * 2 + 1];
              st16((u32x4){perm(H[2 * c], a0, 0x05010400u), perm(H[2 * c], a0, 0x07030602u), perm(H[2 * c + 1], a1_, 0x05010400u), perm(H[2 * c + 1], a1_, 0x07030602u)},
                   rd, lo, dbase + (4 * u + c) * cs);
            }
            FENCE;
            if (u + RING < 8) {
#pragma unroll
              for (int i = 0; i < 3; ++i) d[u % RING][i] = ld16(rs, ln * 16 + i * 1024, (u + RING) * 3072);
            }
            FENCE;
          }
          WAIT_V(0);
          if (ln == 0) astore(ws + OFF_RDY + ((op * 16 + tau) * 64 + jq * 6 + s) * 4, call);
          q = RFL(qn) - qb;
        }
      }
      STAMP(4);
      if (wr == 1) BAR;
    }
    const char *a2, *b2, *a3, *b3;
    if (it < 7 || it == 31) { const int t = (it == 31) ? 0 : 2 * it + 2; a2 = ownA + t * 128; b2 = ownB + (size_t)t * 64 * SB_; a3 = a2 + 128; b3 = b2 + 64 * SB_; }
    else {
      if (ready < 2 * it - 12) {
        int ln = tid; asm volatile("" : "+v"(ln)); ln &= 63;
        const u64 t0 = __builtin_amdgcn_s_memrealtime();
        for (;;) {
          const unsigned fx = aload(ws + OFF_RDY + (tm * 64 + ln) * 4), fw = aload(ws + OFF_RDY + ((16 + tn) * 64 + ln) * 4);
          ready = __builtin_ctzll(~__builtin_amdgcn_ballot_w64(fx == call && fw == call));
          if (ready >= 2 * it - 12) break;
          if (give_up(ws, call, t0)) { ready = 64; break; }
        }
        FENCE;
      }
      a2 = sA + (size_t)s2 * (4096 * SA_) + j * 128; b2 = sB + (size_t)(s2 * 512 + j * 64) * SB_; a3 = a2 + 1024; b3 = b2 + 8192; if (++s2 == 3) { s2 = 0; ++j; }
    }
    LDB(B0, 0, 0); SCHED; LDA(At, 0, 0); STAGEA(1, 1, a1);
    WAIT_L(8); BAR; WAIT_L(0); MMA(0, 0, At, B0); BAR; SCHED;
    LDB(B1, 0, 1); STAGEB(0, 0, b2);
    BAR; WAIT_L(0); MMA(0, 1, At, B1); BAR;
    LDA(At, 0, 1); STAGEA(0, 0, a2);
    BAR; WAIT_L(0); MMA(1, 0, At, B0); BAR; SCHED;
    STAGEB(0, 1, b2);
    WAIT_V(6); BAR; MMA(1, 1, At, B1); BAR;
    LDB(B0, 1, 0); SCHED; LDA(At, 1, 0); STAGEA(0, 1, a2);
    WAIT_L(8); BAR; WAIT_L(0); MMA(0, 0, At, B0); BAR; SCHED;
    LDB(B1, 1, 1); STAGEB(1, 0, b3);
    BAR; WAIT_L(0); MMA(0, 1, At, B1); BAR;
    LDA(At, 1, 1); STAGEA(1, 0, a3);
    BAR; WAIT_L(0); MMA(1, 0, At, B0); BAR; SCHED;
    STAGEB(1, 1, b3);
    WAIT_V(6); BAR; MMA(1, 1, At, B1); BAR;
    a1 = a3;
  }
  if (wr == 0) BAR;
  STAMP(5);

  int ln = tid; asm volatile("" : "+v"(ln)); ln &= 63;
  {
    const bool bad = RFL(aload(ws + OFF_TMO)) == call;
    const unsigned slot = (tm * 16 + tn) * 8 + wid;
    const rsrc_t rpo = mk(ws + OFF_POUT + (size_t)par * 24 * MiB + (size_t)slot * 12288, 12288), rpc = mk(ws + OFF_POUTC + (size_t)par * 16 * MiB + (size_t)slot * 8192, 8192);
    u32x4 hs = {0u, 0u, 0u, 0u};
#pragma unroll
    for (int u = 0; u < 4; ++u) {
      u32x4 r[4];
#pragma unroll
      for (int c = 0; c < 4; ++c) {
        const f32x4 x = acc[u >> 1][c & 1][(u & 1) * 2 + (c >> 1)][0], y = acc[u >> 1][c & 1][(u & 1) * 2 + (c >> 1)][1];
        const auto s0 = __builtin_amdgcn_permlane32_swap(cvt2(x[0], x[1]), cvt2(y[0], y[1]), false, false);
        const auto s1 = __builtin_amdgcn_permlane32_swap(cvt2(x[2], x[3]), cvt2(y[2], y[3]), false, false);
        r[c] = (u32x4){s0[0], s1[0], s0[1], s1[1]};
        if (bad) r[c] = (u32x4){0x7FC07FC0u, 0x7FC07FC0u, 0x7FC07FC0u, 0x7FC07FC0u};
      }
      const u32x4 h = pack_unit(r, tag, ln, rpo, u * 3072, rpc, u * 2048);
      if (ln == u) hs = h;
    }
    WAIT_V(0);
    if (ln < 4) {
      char* dst = p.ws[(tm >> 1) & 7] + OFF_HOUT + (size_t)(((par * 8 + rank) * 32 + (tm & 1) * 16 + tn) * 8 + wid) * 64 + ln * 16;
      asm volatile("global_store_dwordx4 %0, %1, off sc0 sc1\n\ts_nop 1" :: "v"(dst), "v"(hs) : "memory");
    }
  }
  STAMP(6);

  if (rho >= 32 - NRED) {
    char* const ctr = ws + OFF_QO;
    const unsigned qb = (call - 1) * (NOUT + WR);
    const rsrc_t ro = mk(p.out, 16u << 20);
    const unsigned lo = (ln & 15) * SB_ + (ln >> 4) * 16;
    unsigned q = RFL(claim(ctr, ln)) - qb;
    while (q < NOUT) {
      const unsigned qn = claim(ctr, ln);
      const unsigned col = (q + rcol + 1) & 3, w = (q >> 2) & 7, lam = q >> 5;
      const unsigned otn = ((lam & 1) << 3) | (7 - (lam >> 2)), oi = 1 - ((lam >> 1) & 1);
      const unsigned slot = ((rank * 2 + oi) * 16 + otn) * 8 + w;
      const u32x4 h = wait_hdr(ws + OFF_HOUT + (size_t)(((par * 8 + col * 2 + ((ln >> 2) & 1)) * 32 + oi * 16 + otn) * 8 + w) * 64 + (ln & 3) * 16, tag, ws, call);
      const bool bad = RFL(h[0]) == 0u;
      const char* const w0 = p.ws[col * 2]; const char* const w1 = p.ws[col * 2 + 1];
      const size_t po = OFF_POUT + (size_t)par * 24 * MiB + (size_t)slot * 12288, pc = OFF_POUTC + (size_t)par * 16 * MiB + (size_t)slot * 8192;
      const rsrc_t r0 = mk(w0 + po, 12288), r1 = mk(w1 + po, 12288), c0 = mk(w0 + pc, 8192), c1 = mk(w1 + pc, 8192);
      const unsigned obase = ((col >> 1) * 512 + oi * 256 + (w >> 2) * 64) * SB_ + ((col & 1) * 4096 + otn * 256 + (w & 3) * 32) * 2;
      u32x4 d[4][2][3];
#pragma unroll
      for (int u = 0; u < 4; ++u)
#pragma unroll
        for (int i = 0; i < 3; ++i) { d[u][0][i] = ld16(r0, ln * 16 + i * 1024, u * 3072); d[u][1][i] = ld16(r1, ln * 16 + i * 1024, u * 3072); }
      FENCE;
#pragma unroll
      for (int u = 0; u < 4; ++u) {
        unsigned H0[8], H1[8];
        unpack_hi(H0, d[u][0][2], RL(h[0], u), RL(h[1], u), RL(h[3], u), ln, c0, u * 2048);
        unpack_hi(H1, d[u][1][2], RL(h[0], 4 + u), RL(h[1], 4 + u), RL(h[3], 4 + u), ln, c1, u * 2048);
#pragma unroll
        for (int c = 0; c < 4; ++c) {
          u32x4 o;
#pragma unroll
          for (int hh = 0; hh < 2; ++hh) {
            const unsigned a0 = d[u][0][c >> 1][(c & 1) * 2 + hh], a1_ = d[u][1][c >> 1][(c & 1) * 2 + hh], h0 = H0[2 * c + hh], h1 = H1[2 * c + hh];
            float v[4];
#pragma unroll
            for (int e = 0; e < 4; ++e) { const unsigned sel = 0x04000C0Cu + e * 0x01010000u;
              v[e] = __builtin_bit_cast(float, perm(h0, a0, sel)) + __builtin_bit_cast(float, perm(h1, a1_, sel)); }
            o[2 * hh] = __builtin_bit_cast(unsigned, __builtin_elementwise_max(__builtin_bit_cast(s16x2, cvt2(v[0], v[1])), (s16x2){0, 0}));
            o[2 * hh + 1] = __builtin_bit_cast(unsigned, __builtin_elementwise_max(__builtin_bit_cast(s16x2, cvt2(v[2], v[3])), (s16x2){0, 0}));
          }
          if (bad) o = (u32x4){0x7FC07FC0u, 0x7FC07FC0u, 0x7FC07FC0u, 0x7FC07FC0u};
          __builtin_amdgcn_raw_buffer_store_b128(o, ro, lo + obase + ((u >> 1) * 128 + ((u & 1) * 2 + (c >> 1)) * 16) * SB_ + (c & 1) * 256, 0, 16);
        }
        FENCE;
      }
      q = RFL(qn) - qb;
    }
  }
  STAMP(7);
}

extern "C" void kernel_launch(int rank, int nranks, ncclComm_t comm,
                              hipStream_t stream,
                              void* const* d_in, const int* in_sizes, int n_in,
                              void* d_out, int out_size,
                              const int* in_dtypes, int out_dtype,
                              void* const* d_out_all,
                              void* const* d_ws_all, size_t ws_bytes) {
  (void)comm; (void)in_dtypes; (void)out_dtype; (void)d_out_all;
  if (nranks != 8 || (unsigned)rank >= 8u || n_in != 2 || ws_bytes < WS_NEED || in_sizes[0] != 8192 * 1024 || in_sizes[1] != 8192 * 1024 || out_size != 1024 * 8192) {
    fprintf(stderr, "gemm_rs: unexpected problem shape\n"); return; }
  HIP_CHECK(hipFuncSetAttribute((const void*)gemm_rs, hipFuncAttributeMaxDynamicSharedMemorySize, LDS_BYTES));
  Params p;
  p.xin = (const char*)d_in[0]; p.win = (const char*)d_in[1]; p.out = (char*)d_out; p.rank = rank; p.pad = 0;
  for (int i = 0; i < 8; ++i) p.ws[i] = (char*)d_ws_all[i];
  gemm_rs<<<NWG, 512, LDS_BYTES, stream>>>(p);
  HIP_CHECK(hipGetLastError());
}
```

```cpp
#include <hip/hip_runtime.h>
#include <rccl/rccl.h>
#include <cstdio>

#define HIP_CHECK(expr) do { hipError_t _e = (expr); if (_e != hipSuccess) { \
    fprintf(stderr, "HIP error %s at %s:%d\n", hipGetErrorString(_e), __FILE__, __LINE__); \
    return; } } while (0)

#define LAS __attribute__((address_space(3)))
#define GAS __attribute__((address_space(1)))
typedef short bf16x8 __attribute__((ext_vector_type(8)));
typedef short s16x4 __attribute__((ext_vector_type(4)));
typedef short s16x2 __attribute__((ext_vector_type(2)));
typedef unsigned short u16x2 __attribute__((ext_vector_type(2)));
typedef __bf16 bf16x2 __attribute__((ext_vector_type(2)));
typedef float f32x2 __attribute__((ext_vector_type(2)));
typedef float f32x4 __attribute__((ext_vector_type(4)));
typedef unsigned u32x4 __attribute__((ext_vector_type(4)));
typedef unsigned long long u64;
typedef __amdgpu_buffer_rsrc_t rsrc_t;

#define NC 4
#define NRED 4
#define RW 4
#define SPLIT 152
#define PAUSE2 19
#define GATE 0
#define DELAYS 0, 10, 8, 6, 4, 3, 1, 0
struct alignas(16) Order { unsigned char e[256]; };
constexpr Order make_order() {
  Order o{}; const int dl[8] = {DELAYS}; int n = 0;
  for (int key = 0; key < 64; ++key)
    for (int op = 0; op < 2; ++op)
      for (int k = 0; k < 16; ++k)
        for (int j = 0; j < 8; ++j) {
          const int tau = op ? (((k & 1) << 3) | (7 - (k >> 1))) : k;
          int kk = 2 * j + (op ? dl[tau & 7] : 0);
          if (op && (tau & 7) == 0 && 3 * j >= PAUSE2 - 7) kk = 40 + j;
          if (kk == key) o.e[n++] = (unsigned char)(op << 7 | tau << 3 | j);
        }
  return o;
}
constexpr Order ORDER_H = make_order();
constexpr bool order_ok() {
  for (int i = SPLIT; i < 256; ++i) { const int e = ORDER_H.e[i], op = e >> 7, tau = (e >> 3) & 15, j = e & 7;
    if ((!op || (tau & 7) == 0) && 3 * j < PAUSE2 - 7) return false; }
  return true;
}
static_assert(order_ok(), "shift 2 would wait for blocks it has to pull itself");
static_assert(NC >= 3 && NC <= 4 && NRED >= 2 && NRED <= 8, "roles");
__constant__ Order ORDER = ORDER_H;

constexpr int NWG = 256, HTB = 16384;
constexpr unsigned SA_ = 2048, SB_ = 16384;
constexpr size_t MiB = 1ull << 20;
constexpr size_t OFF_CALL = 0;
constexpr size_t OFF_TMO = 32768, OFF_Q = 33024, OFF_HAND = 36864;
constexpr size_t OFF_RDY = 40960;
constexpr size_t OFF_HIN = 65536;
constexpr size_t OFF_HOUT = OFF_HIN + 524288;
constexpr size_t OFF_STGX = 1 * MiB, OFF_STGW = 25 * MiB;
constexpr size_t OFF_PIN = 49 * MiB;
constexpr size_t OFF_PINC = 97 * MiB;
constexpr size_t OFF_POUT = 129 * MiB;
constexpr size_t OFF_POUTC = 177 * MiB;
constexpr size_t OFF_DBG = 209 * MiB;
constexpr size_t WS_NEED = 210 * MiB;
constexpr int LDS_BYTES = 8 * HTB + 1024;
constexpr u64 DEADLINE = 100000000ull;

struct Params { const char* xin; const char* win; char* out; char* own; char* ws[8]; int rank; int pad; };

#define RFL(x) __builtin_amdgcn_readfirstlane(x)
#define WAIT_V(n) asm volatile("s_waitcnt vmcnt(" #n ")" ::: "memory")
#define WAIT_L(n) asm volatile("s_waitcnt lgkmcnt(" #n ")" ::: "memory")
#define BAR __builtin_amdgcn_s_barrier()
#define SCHED __builtin_amdgcn_sched_barrier(0)
#define FENCE asm volatile("" ::: "memory")
#define DI __device__ __forceinline__
constexpr int SYS = 17;

DI rsrc_t mk(const void* p, unsigned bytes) { return __builtin_amdgcn_make_buffer_rsrc((void*)p, (short)0, (int)bytes, 0x00020000); }
DI u32x4 ld16(rsrc_t r, unsigned v, unsigned s) { return __builtin_amdgcn_raw_buffer_load_b128(r, v, s, SYS); }
DI u32x4 ld16p(rsrc_t r, unsigned v, unsigned s) { return __builtin_amdgcn_raw_buffer_load_b128(r, v, s, 0); }
DI void st16(u32x4 x, rsrc_t r, unsigned v, unsigned s) { __builtin_amdgcn_raw_buffer_store_b128(x, r, v + s, 0, SYS); }
DI unsigned aload(const char* p) { return __hip_atomic_load((const GAS unsigned*)p, __ATOMIC_RELAXED, __HIP_MEMORY_SCOPE_SYSTEM); }
DI void astore(char* p, unsigned v) { __hip_atomic_store((GAS unsigned*)p, v, __ATOMIC_RELAXED, __HIP_MEMORY_SCOPE_SYSTEM); }
DI unsigned perm(unsigned hi, unsigned lo, unsigned sel) { return __builtin_amdgcn_perm(hi, lo, sel); }
DI unsigned cvt2(float lo, float hi) { return __builtin_bit_cast(unsigned, __builtin_convertvector((f32x2){lo, hi}, bf16x2)); }

#define STAMP(i) do { } while (0)

DI bool give_up(char* ws, unsigned call, u64 t0) {
  if (RFL(aload(ws + OFF_TMO)) == call) return true;
  if (__builtin_amdgcn_s_memrealtime() - t0 > DEADLINE) { astore(ws + OFF_TMO, call); return true; }
  __builtin_amdgcn_s_sleep(2);
  return false;
}

DI u32x4 pack_unit(const u32x4 (&r)[4], unsigned tag, int lane, rsrc_t dst, unsigned doff, rsrc_t dstc, unsigned coff) {
  u16x2 mx = {0, 0};
#pragma unroll
  for (int c = 0; c < 4; ++c)
#pragma unroll
    for (int i = 0; i < 4; ++i) mx = __builtin_elementwise_max(mx, __builtin_bit_cast(u16x2, r[c][i] & 0x7FFF7FFFu));
  const unsigned f = (mx.x > mx.y ? mx.x : mx.y) >> 8;
  unsigned F = 0;
#pragma unroll
  for (int bit = 64; bit; bit >>= 1) F = __builtin_amdgcn_ballot_w64(f >= (F | bit)) ? (F | bit) : F;
  const unsigned base = F > 7 ? F - 7 : 0, B4 = base * 0x01010101u;
  unsigned A[8], H[8], T[8], P[4], e = 0;
#pragma unroll
  for (int c = 0; c < 4; ++c) {
    A[2 * c] = perm(r[c][1], r[c][0], 0x06040200u); A[2 * c + 1] = perm(r[c][3], r[c][2], 0x06040200u);
    H[2 * c] = perm(r[c][1], r[c][0], 0x07050301u); H[2 * c + 1] = perm(r[c][3], r[c][2], 0x07050301u);
    T[2 * c] = (H[2 * c] | 0x80808080u) - B4; T[2 * c + 1] = (H[2 * c + 1] | 0x80808080u) - B4;
    e |= T[2 * c] | T[2 * c + 1];
    P[c] = (H[2 * c] & 0x80808080u) | (T[2 * c] & 0x07070707u) | ((T[2 * c + 1] & 0x07070707u) << 3) | ((H[2 * c + 1] & 0x80808080u) >> 1);
  }
  st16((u32x4){A[0], A[1], A[2], A[3]}, dst, lane * 16, doff);
  st16((u32x4){A[4], A[5], A[6], A[7]}, dst, lane * 16 + 1024, doff);
  st16((u32x4){P[0], P[1], P[2], P[3]}, dst, lane * 16 + 2048, doff);
  unsigned w0 = (tag << 8) | base; u64 pt = 0;
  if (__builtin_amdgcn_ballot_w64((e & 0x78787878u) != 0)) {
    unsigned X[8]; int cnt = 0; unsigned code = 0;
#pragma unroll
    for (int k = 7; k >= 0; --k) {
      X[k] = (H[k] & 0x7F7F7F7Fu) ^ ((T[k] & 0x07070707u) + B4);
      const unsigned nz = ((X[k] >> 3) + 0x0F0F0F0Fu) & 0x10101010u;
      cnt += __builtin_popcount(nz);
      if (X[k]) { const unsigned by = (unsigned)__builtin_ctz(X[k]) >> 3; code = ((unsigned)lane << 10) | (k << 7) | (by << 5) | ((X[k] >> (8 * by + 3)) & 15u); }
    }
    u64 m = __builtin_amdgcn_ballot_w64(cnt != 0);
    if (__builtin_amdgcn_ballot_w64(cnt > 1) || __builtin_popcountll(m) > 4) {
      w0 |= 0x80u;
      st16((u32x4){X[0], X[1], X[2], X[3]}, dstc, lane * 16, coff);
      st16((u32x4){X[4], X[5], X[6], X[7]}, dstc, lane * 16 + 1024, coff);
    } else {
      while (m) { const int l = __builtin_ctzll(m); m &= m - 1; pt = (pt << 16) | (unsigned)__builtin_amdgcn_readlane((int)code, l); }
    }
  }
  return (u32x4){w0, (unsigned)pt, w0, (unsigned)(pt >> 32)};
}
DI void unpack_hi(unsigned (&H)[8], u32x4 P, unsigned hw, unsigned p0, unsigned p1, int lane, rsrc_t srcc, unsigned coff) {
  const unsigned B4 = (hw & 0x7Fu) * 0x01010101u;
#pragma unroll
  for (int c = 0; c < 4; ++c) {
    H[2 * c] = (P[c] & 0x87878787u) + B4;
    H[2 * c + 1] = (((P[c] >> 3) & 0x07070707u) + B4) | ((P[c] << 1) & 0x80808080u);
  }
  if (p0 | (hw & 0x80u)) {
    if (hw & 0x80u) {
      const u32x4 x0 = ld16(srcc, lane * 16, coff), x1 = ld16(srcc, lane * 16 + 1024, coff);
#pragma unroll
      for (int k = 0; k < 4; ++k) { H[k] ^= x0[k]; H[4 + k] ^= x1[k]; }
    } else {
      u64 pt = ((u64)p1 << 32) | p0;
      while (pt) {
        const unsigned code = (unsigned)pt & 0xFFFFu; pt >>= 16;
        const unsigned dv = (unsigned)lane == (code >> 10) ? (code & 15u) << (8 * ((code >> 5) & 3u) + 3) : 0u, kk = (code >> 7) & 7u;
#pragma unroll
        for (int k = 0; k < 8; ++k) H[k] ^= dv & (kk == (unsigned)k ? ~0u : 0u);
      }
    }
  }
}
DI bool wait_hdr(u32x4& h, rsrc_t rh, unsigned voff, unsigned soff, unsigned tag, char* ws, unsigned call) {
  if (!__builtin_amdgcn_ballot_w64((h[0] >> 8) != tag || (h[2] >> 8) != tag)) return true;
  const u64 t0 = __builtin_amdgcn_s_memrealtime();
  for (;;) {
    h = ld16(rh, voff, soff); FENCE;
    if (!__builtin_amdgcn_ballot_w64((h[0] >> 8) != tag || (h[2] >> 8) != tag)) return true;
    if (give_up(ws, call, t0)) { h = (u32x4){0u, 0u, 0u, 0u}; return false; }
  }
}
#define CLAIM(cv, base, on) do { cv = 1u; asm volatile("s_mov_b32 exec_lo, %3\n\ts_mov_b32 exec_hi, 0\n\ts_nop 2\n\tglobal_atomic_add %0, %1, %0, %2 sc0\n\ts_mov_b64 exec, -1" : "+v"(cv) : "v"(0u), "s"(base), "s"(RFL((on) ? 1u : 0u)) : "memory"); } while (0)
#define SIGNAL(base, off) asm volatile("s_mov_b64 exec, 1\n\ts_nop 3\n\tglobal_atomic_add %0, %1, %2\n\ts_mov_b64 exec, -1" :: "v"(off), "v"(1u), "s"(base) : "memory")
#define RL(x, l) ((unsigned)__builtin_amdgcn_readlane((int)(x), (l)))

__global__ __launch_bounds__(512, 2) void gemm_rs(Params p) {
  extern __shared__ __attribute__((aligned(16))) unsigned char smem[];
  LAS unsigned char* lds = (LAS unsigned char*)smem;
  const int tid = threadIdx.x, wid = RFL(tid >> 6), lane = tid & 63, wr = wid >> 2, wc = wid & 3, fr = lane & 15, fq = lane >> 4;
  const int rank = p.rank & 7, ra = rank >> 2, rb = (rank >> 1) & 1, rc = rank & 1, rcol = rank >> 1;
  char* const ws = p.own;
  const int bid = blockIdx.x, xcd = bid & 7, idx = bid >> 3;
  const int tm = (xcd >> 1) * 4 + (idx & 3), tn = (xcd & 1) * 8 + (idx >> 2);
  const int shift = (idx & 3) >= NC ? 0 : (idx >> 2) == 1 ? 1 : (idx >> 2) == 0 ? 2 : 0;
  STAMP(0);
  const unsigned callv = aload(ws + OFF_CALL + bid * 128);
  unsigned tab = ((const unsigned*)ORDER.e)[lane];

  const unsigned ldsw = (unsigned)(size_t)lds + (unsigned)wid * 1024u;
  unsigned vA[4], vB[4];
  { const unsigned voffA = (wid * 8 + (lane >> 3)) * SA_ + 16 * ((lane & 7) ^ (lane >> 3));
    const int s = lane >> 3, pos = (lane & 7) >> 1, hf = lane & 1, cp = pos ^ (s >> 1);
    const unsigned voffB = (16 * (wid >> 2) + 8 * (s >> 2) + 4 * ((wid >> 1) & 1) + (s & 3)) * SB_ + (64 * (wid & 1) + 16 * cp + 8 * hf) * 2;
#pragma unroll
    for (int i = 0; i < 4; ++i) { vA[i] = voffA + i * 64 * SA_; vB[i] = voffB + (i & 1) * 32 * SB_ + (i >> 1) * 256; asm volatile("" : "+v"(vA[i]), "+v"(vB[i])); } }
#define SA(b, h) (((b) * 2 + (h)) * HTB)
#define SB(b, h) ((4 + (b) * 2 + (h)) * HTB)
#define GLDS(voff, sbase, c) asm volatile("s_add_i32 m0, %2, %3\n\ts_nop 0\n\tglobal_load_lds_dwordx4 %0, %1 sc0 sc1" :: "v"(voff), "s"(sbase), "s"(ldsw), "i"(c) : "memory", "scc")
#define STAGEA(b, h, gbase) do { GLDS(vA[2 * (h)], gbase, SA(b, h)); GLDS(vA[2 * (h) + 1], gbase, SA(b, h) + 8192); } while (0)
#define STAGEB(b, h, gbase) do { GLDS(vB[2 * (h)], gbase, SB(b, h)); GLDS(vB[2 * (h) + 1], gbase, SB(b, h) + 8192); } while (0)
  const char* const ownA = p.xin + (size_t)((tm >> 1) * 1024 + ra * 512 + (tm & 1) * 256) * SA_;
  const char* const ownB = p.win + (size_t)(rb * 4096 + tn * 256) * 2;
  const char* a1 = ownA + 128;

  unsigned call, par, tag;
  {
    const unsigned loX = (lane >> 3) * SA_ + (lane & 7) * 16, loW = (lane >> 5) * SB_ + (lane & 31) * 16;
    const rsrc_t rx = mk(p.xin, 16u << 20), rw = mk(p.win, 16u << 20);
    const int np = NWG - 8 * NC, pid = (idx < 4 ? idx : idx - NC) * 8 + xcd;
    int blk[5]; bool val[5];
#pragma unroll
    for (int k = 0; k < 5; ++k) { const int g = k == 0 ? bid : 256 + pid + (k - 1) * np; val[k] = k == 0 || (shift != 1 && g < 1024);
      const int gg = val[k] ? g : bid, e = (((const unsigned*)ORDER.e)[gg >> 4] >> ((gg >> 2) & 3) * 8) & 255, op = e >> 7, tau = (e >> 3) & 15, hv = gg & 1;
      blk[k] = op * 512 + (((gg >> 1) & 1) * 8 + (e & 7)) * 32 + (op ? hv * 16 + tau : (tau >> 1) * 4 + hv * 2 + (tau & 1)); }
    u32x4 r[5][4];
#define PK_LOAD(k) do { const int _op = blk[k] >> 9, _jj = (blk[k] >> 5) & 15, _ix = blk[k] & 31; \
      const unsigned _so = _op ? (_jj * 64 + wid * 8) * SB_ + _ix * 512 : (_ix * 256 + wid * 32) * SA_ + _jj * 128, _cs = _op ? 2 * SB_ : 8 * SA_; \
      _Pragma("unroll") for (int c = 0; c < 4; ++c) r[k][c] = ld16p(_op ? rw : rx, _op ? loW : loX, _so + c * _cs); } while (0)
#define PK_PACK(k) do { const u32x4 _h = pack_unit(r[k], tag, lane, rpi, blk[k] * 24576 + wid * 3072, rpc, blk[k] * 16384 + wid * 2048); \
      if (lane == 0) *(LAS u32x4*)(lds + 8 * HTB + (k) * 128 + wid * 16) = _h; } while (0)
#define PK_SEND(k) do { const int _op = blk[k] >> 9, _jj = (blk[k] >> 5) & 15, _ix = blk[k] & 31, _d = (lane >> 3) & 1; \
      const int _tau = _op ? (_ix & 15) : ((_ix >> 2) * 2 + (_ix & 1)); \
      const int _c0 = _op ? (_ix >> 4) : (_ix & 2), _c1 = _op ? _c0 + 2 : _c0 + 1; \
      char* const _w0 = p.ws[(_c0 * 2 + (_jj >> 3)) & 7]; char* const _w1 = p.ws[(_c1 * 2 + (_jj >> 3)) & 7]; \
      if (lane < 16 && (_d ? _c1 : _c0) != rcol) { const u32x4 _h = *(const LAS u32x4*)(lds + 8 * HTB + (k) * 128 + (lane & 7) * 16); \
        char* _dst = (_d ? _w1 : _w0) + OFF_HIN + (size_t)((par * 8 + rank) * 256 + _op * 128 + _tau * 8 + (_jj & 7)) * 128 + (lane & 7) * 16; \
        asm volatile("global_store_dwordx4 %0, %1, off sc0 sc1\n\ts_nop 1" :: "v"(_dst), "v"(_h) : "memory"); } } while (0)
    PK_LOAD(0);
    call = RFL(callv) + 1; par = call & 1; tag = (call % 0xFFFFFFu) + 1;
    const rsrc_t rpi = mk(ws + OFF_PIN + (size_t)par * 24 * MiB, 24u << 20), rpc = mk(ws + OFF_PINC + (size_t)par * 16 * MiB, 16u << 20);
    PK_PACK(0);
    asm volatile("" : "+v"(tab));
    WAIT_V(0); WAIT_L(0); BAR; FENCE;
    if (wid == 0) PK_SEND(0);
    if (tid == 64) astore(ws + OFF_CALL + bid * 128, call);
    STAMP(1);
#define PROLOGUE() do { STAGEB(0, 0, ownB); STAGEA(0, 0, ownA); STAGEB(0, 1, ownB); STAGEA(0, 1, ownA); \
      STAGEB(1, 0, ownB + 64 * SB_); STAGEA(1, 0, a1); STAGEB(1, 1, ownB + 64 * SB_); } while (0)
    if (shift != 1) {
      PK_LOAD(1); PK_LOAD(2); PK_LOAD(3); PK_LOAD(4); FENCE;
      PK_PACK(1); PK_PACK(2); PK_PACK(3); if (val[4]) PK_PACK(4);
      PROLOGUE();
      WAIT_V(0); WAIT_L(0); BAR; FENCE;
      if (wid == 1) PK_SEND(1);
      if (wid == 2) PK_SEND(2);
      if (wid == 3) PK_SEND(3);
      if (wid == 4 && val[4]) PK_SEND(4);
      if (wr == 1) BAR;
    }
  }
  STAMP(2);

  int aoff0 = (wr * 64 + fr) * 128 + 16 * (fq ^ (fr & 7)), aoff1 = aoff0 ^ 64, boff0, boff1;
  { const int s = (lane >> 2) & 7, pp = ((lane & 1) << 1) | ((lane >> 1) & 1), gh = lane >> 5;
    boff0 = 4 * HTB + gh * 4096 + (wc >> 1) * 1024 + 128 * s + 32 * ((2 * (wc & 1)) ^ (s >> 1)) + 8 * pp; boff1 = boff0 ^ 32; }
  asm volatile("" : "+v"(aoff0), "+v"(aoff1), "+v"(boff0), "+v"(boff1));
#define LDA(dst, b, h) do { _Pragma("unroll") for (int m = 0; m < 4; ++m) { dst[m][0] = *(const LAS bf16x8*)(lds + SA(b, h) + aoff0 + m * 2048); dst[m][1] = *(const LAS bf16x8*)(lds + SA(b, h) + aoff1 + m * 2048); } } while (0)
#define TR(off) __builtin_amdgcn_ds_read_tr16_b64_v4i16((LAS s16x4*)(lds + (off)))
#define LDB(dst, b, h) do { _Pragma("unroll") for (int k = 0; k < 2; ++k) { \
    dst[0][k] = __builtin_shufflevector(TR(SA(b, h) + boff0 + k * 8192), TR(SA(b, h) + boff0 + k * 8192 + 2048), 0, 1, 2, 3, 4, 5, 6, 7); \
    dst[1][k] = __builtin_shufflevector(TR(SA(b, h) + boff1 + k * 8192), TR(SA(b, h) + boff1 + k * 8192 + 2048), 0, 1, 2, 3, 4, 5, 6, 7); } } while (0)
#define MMA(ai, bj, At, Bt) do { __builtin_amdgcn_s_setprio(1); _Pragma("unroll") for (int m = 0; m < 4; ++m) _Pragma("unroll") for (int n = 0; n < 2; ++n) _Pragma("unroll") for (int k = 0; k < 2; ++k) \
    acc[ai][bj][m][n] = __builtin_amdgcn_mfma_f32_16x16x32_bf16(Bt[n][k], At[m][k], acc[ai][bj][m][n], 0, 0, 0); __builtin_amdgcn_s_setprio(0); } while (0)
  f32x4 acc[2][2][4][2];
#pragma unroll
  for (int a = 0; a < 2; ++a)
#pragma unroll
    for (int b = 0; b < 2; ++b)
#pragma unroll
      for (int m = 0; m < 4; ++m)
#pragma unroll
        for (int n = 0; n < 2; ++n) acc[a][b][m][n] = (f32x4){0.f, 0.f, 0.f, 0.f};
  bf16x8 At[4][2], B0[2][2], B1[2][2];
  const char* const sA = ws + OFF_STGX + (size_t)tm * 256 * SA_;
  const char* const sB = ws + OFF_STGW + (size_t)tn * 512;
  const int pause = shift == 1 ? 0 : (shift == 2 ? PAUSE2 : -1);
  int j = 0, s2 = 0, ready = 0;
  for (int it = 0; it < 32; ++it) {
    if (it == pause) {
      if (wr == 0 && shift == 2) BAR;
      STAMP(3);
      int ln = tid; asm volatile("" : "+v"(ln)); ln &= 63;
      const unsigned gw = ((idx & 3) * 8 + xcd) * 8 + wid, s = gw % 6, widx = gw / 6, cnt = (NC * 64 + 5 - s) / 6;
      const unsigned lb = shift == 1 ? 0 : SPLIT * 8, le = shift == 1 ? SPLIT * 8 : 2048;
      const unsigned qb = (call - 1) * (le - lb - cnt) - lb - 2 * cnt;
      char* const ctr = ws + OFF_Q + (s * 2 + shift - 1) * 128; char* const rdy = ws + OFF_RDY;
      const unsigned src = ((((unsigned)rcol ^ (1 + (s >> 1))) << 1) | (s & 1)) & 7;
      const char* const sw = p.ws[src];
      const unsigned lx = (ln >> 3) * SA_ + (ln & 7) * 16, lw = (ln >> 5) * SB_ + (ln & 31) * 16;
      const rsrc_t rs = mk(sw + OFF_PIN + (size_t)par * 24 * MiB, 24u << 20), rsc = mk(sw + OFF_PINC + (size_t)par * 16 * MiB, 16u << 20), rd = mk(ws + OFF_STGX, 48u << 20);
      const rsrc_t rh = mk(ws + OFF_HIN + (size_t)(par * 8 + src) * 32768, 32768);
#define ENT(L) ((((RL(tab, ((L) >> 5) & 63) >> (((L) >> 3) & 3) * 8) & 255u) << 3) | ((L) & 7))
#define IN_SRC(E) (((E) >> 10) ? (512 + (rc * 8 + (((E) >> 3) & 7)) * 32 + rb * 16 + (((E) >> 6) & 15)) * 8 + ((E) & 7) \
                               : ((rc * 8 + (((E) >> 3) & 7)) * 32 + ((E) >> 7) * 4 + ra * 2 + (((E) >> 6) & 1)) * 8 + ((E) & 7))
#define IN_RDY(E) ((((E) >> 6) * 64 + (((E) >> 3) & 7) * 6 + s) * 4)
      u32x4 hv, da[3], db[3]; unsigned cv, Ep = 0, Ec = ENT(lb + widx), En, Ln = lb + cnt + widx, hw, p0, p1; bool first = true;
      hv = ld16(rh, 0, Ec * 16); wait_hdr(hv, rh, 0, Ec * 16, tag, ws, call); hw = RFL(hv[0]); p0 = RFL(hv[1]); p1 = RFL(hv[3]);
      if (shift == 2) { const u64 t0 = __builtin_amdgcn_s_memrealtime(); while (RFL(aload(ws + OFF_HAND + gw * 4)) != call && !give_up(ws, call, t0)) { } }
      STAMP(8);
#pragma unroll
      for (int i = 0; i < 3; ++i) da[i] = ld16(rs, ln * 16 + i * 1024, IN_SRC(Ec) * 3072);
      En = ENT(Ln); hv = ld16(rh, 0, En * 16); CLAIM(cv, ctr, 1);
#define IN_STEP(d, dn) { asm volatile("s_waitcnt vmcnt(0)" : "+v"(d[0]), "+v"(d[1]), "+v"(d[2]), "+v"(hv), "+v"(cv) :: "memory"); \
        unsigned nw = 0, n0 = 0, n1 = 0, Eq = 0, Lq = ~0u; \
        if (Ln < le) { wait_hdr(hv, rh, 0, En * 16, tag, ws, call); nw = RFL(hv[0]); n0 = RFL(hv[1]); n1 = RFL(hv[3]); \
          _Pragma("unroll") for (int i = 0; i < 3; ++i) dn[i] = ld16(rs, ln * 16 + i * 1024, IN_SRC(En) * 3072); \
          Lq = RFL(cv) - qb; Eq = ENT(Lq); hv = ld16(rh, 0, Eq * 16); CLAIM(cv, ctr, Lq < le); \
          if (Lq >= le && shift == 1 && ln == 0) astore(ws + OFF_HAND + gw * 4, call); } \
        if (!first) SIGNAL(rdy, IN_RDY(Ep)); \
        FENCE; \
        unsigned H[8]; unpack_hi(H, d[2], hw, p0, p1, ln, rsc, IN_SRC(Ec) * 2048); \
        const unsigned _op = Ec >> 10, _tau = (Ec >> 6) & 15, _j = (Ec >> 3) & 7, _u = Ec & 7, _cs = _op ? 2 * SB_ : 8 * SA_, _lo = _op ? lw : lx; \
        const unsigned _db = _op ? 24 * (unsigned)MiB + ((s >> 1) * 512 + _j * 64 + _u * 8) * SB_ + (s & 1) * 8192 + _tau * 512 : ((s >> 1) * 4096 + _tau * 256 + _u * 32) * SA_ + (s & 1) * 1024 + _j * 128; \
        _Pragma("unroll") for (int c = 0; c < 4; ++c) { const unsigned a0 = d[c >> 1][(c & 1) * 2], a1_ = d[c >> 1][(c & 1) * 2 + 1]; \
          st16((u32x4){perm(H[2 * c], a0, 0x05010400u), perm(H[2 * c], a0, 0x07030602u), perm(H[2 * c + 1], a1_, 0x05010400u), perm(H[2 * c + 1], a1_, 0x07030602u)}, rd, _lo, _db + c * _cs); } \
        FENCE; first = false; Ep = Ec; \
        if (Ln >= le) break; \
        Ec = En; hw = nw; p0 = n0; p1 = n1; En = Eq; Ln = Lq; }
      for (;;) { IN_STEP(da, db) IN_STEP(db, da) }
      if (shift == 1) PROLOGUE();
      WAIT_V(0);
      SIGNAL(rdy, IN_RDY(Ep));
      STAMP(4);
      if (shift == 1) BAR;
      if (wr == 1) BAR;
    }
    const char *a2, *b2, *a3, *b3;
    if (it < 7 || it == 31) { const int t = (it == 31) ? 0 : 2 * it + 2; a2 = ownA + t * 128; b2 = ownB + (size_t)t * 64 * SB_; a3 = a2 + 128; b3 = b2 + 64 * SB_; }
    else {
      if (ready < 2 * it - 12) {
        int ln = tid; asm volatile("" : "+v"(ln)); ln &= 63;
        const u64 t0 = __builtin_amdgcn_s_memrealtime();
        for (;;) {
          const unsigned fx = aload(ws + OFF_RDY + (tm * 64 + ln) * 4), fw = aload(ws + OFF_RDY + ((16 + tn) * 64 + ln) * 4);
          ready = __builtin_ctzll(~__builtin_amdgcn_ballot_w64(fx == 8 * call && fw == 8 * call && ln < 48));
          if (ready >= 2 * it - 12) break;
          if (give_up(ws, call, t0)) { ready = 64; break; }
        }
        FENCE;
      }
      a2 = sA + (size_t)s2 * (4096 * SA_) + j * 128; b2 = sB + (size_t)(s2 * 512 + j * 64) * SB_; a3 = a2 + 1024; b3 = b2 + 8192; if (++s2 == 3) { s2 = 0; ++j; }
    }
    LDB(B0, 0, 0); SCHED; LDA(At, 0, 0); STAGEA(1, 1, a1);
    WAIT_L(8); BAR; WAIT_L(0); MMA(0, 0, At, B0); BAR; SCHED;
    LDB(B1, 0, 1); STAGEB(0, 0, b2);
    BAR; WAIT_L(0); MMA(0, 1, At, B1); BAR;
    LDA(At, 0, 1); STAGEA(0, 0, a2);
    BAR; WAIT_L(0); MMA(1, 0, At, B0); BAR; SCHED;
    STAGEB(0, 1, b2);
    WAIT_V(6); BAR; MMA(1, 1, At, B1); BAR;
    LDB(B0, 1, 0); SCHED; LDA(At, 1, 0); STAGEA(0, 1, a2);
    WAIT_L(8); BAR; WAIT_L(0); MMA(0, 0, At, B0); BAR; SCHED;
    LDB(B1, 1, 1); STAGEB(1, 0, b3);
    BAR; WAIT_L(0); MMA(0, 1, At, B1); BAR;
    LDA(At, 1, 1); STAGEA(1, 0, a3);
    BAR; WAIT_L(0); MMA(1, 0, At, B0); BAR; SCHED;
    STAGEB(1, 1, b3);
    WAIT_V(6); BAR; MMA(1, 1, At, B1); BAR;
    a1 = a3;
  }
  if (wr == 0) BAR;
  STAMP(5);

  int ln = tid; asm volatile("" : "+v"(ln)); ln &= 63;
  {
    const bool bad = RFL(aload(ws + OFF_TMO)) == call;
    const unsigned slot = (tm * 16 + tn) * 8 + wid;
    const rsrc_t rpo = mk(ws + OFF_POUT + (size_t)par * 24 * MiB + (size_t)slot * 12288, 12288), rpc = mk(ws + OFF_POUTC + (size_t)par * 16 * MiB + (size_t)slot * 8192, 8192);
    u32x4 hs = {0u, 0u, 0u, 0u};
#pragma unroll
    for (int u = 0; u < 4; ++u) {
      u32x4 r[4];
#pragma unroll
      for (int c = 0; c < 4; ++c) {
        const f32x4 x = acc[u >> 1][c & 1][(u & 1) * 2 + (c >> 1)][0], y = acc[u >> 1][c & 1][(u & 1) * 2 + (c >> 1)][1];
        const auto s0 = __builtin_amdgcn_permlane32_swap(cvt2(x[0], x[1]), cvt2(y[0], y[1]), false, false);
        const auto s1 = __builtin_amdgcn_permlane32_swap(cvt2(x[2], x[3]), cvt2(y[2], y[3]), false, false);
        r[c] = (u32x4){s0[0], s1[0], s0[1], s1[1]};
        if (bad) r[c] = (u32x4){0x7FC07FC0u, 0x7FC07FC0u, 0x7FC07FC0u, 0x7FC07FC0u};
      }
      const u32x4 h = pack_unit(r, tag, ln, rpo, u * 3072, rpc, u * 2048);
      if (ln == u) hs = h;
    }
    WAIT_V(0);
    if (ln < 4) {
      char* dst = p.ws[(tm >> 1) & 7] + OFF_HOUT + (size_t)(((par * 8 + rank) * 32 + (tm & 1) * 16 + tn) * 8 + wid) * 64 + ln * 16;
      asm volatile("global_store_dwordx4 %0, %1, off sc0 sc1\n\ts_nop 1" :: "v"(dst), "v"(hs) : "memory");
    }
  }
  STAMP(6);

  if (idx >= 32 - NRED && wid < RW) {
    const unsigned gw = ((31 - idx) * 8 + xcd) * RW + wid, col = gw & 3, widx = gw >> 2, cnt = NRED * 2 * RW;
    const unsigned qb = (call - 1) * (1024 - cnt) - 2 * cnt;
    char* const ctr = ws + OFF_Q + (12 + col) * 128;
    const char* const w0 = p.ws[col * 2]; const char* const w1 = p.ws[col * 2 + 1];
    const size_t mine = (size_t)rank * 1024;
    const rsrc_t r0 = mk(w0 + OFF_POUT + (size_t)par * 24 * MiB + mine * 3072, 3u << 20), r1 = mk(w1 + OFF_POUT + (size_t)par * 24 * MiB + mine * 3072, 3u << 20);
    const rsrc_t c0 = mk(w0 + OFF_POUTC + (size_t)par * 16 * MiB + mine * 2048, 2u << 20), c1 = mk(w1 + OFF_POUTC + (size_t)par * 16 * MiB + mine * 2048, 2u << 20);
    const rsrc_t rh = mk(ws + OFF_HOUT + (size_t)(par * 8 + col * 2) * 16384, 32768), ro = mk(p.out, 16u << 20);
    const unsigned lo = (ln & 15) * SB_ + (ln >> 4) * 16, ho = (ln >> 5) * 16384;
#define OENT(L) (((((L) >> 6) & 1) << 9) | ((((L) >> 5) & 1) << 8) | ((7 - (((L) >> 7) & 7)) << 5) | ((L) & 31))
    u32x4 hv, da[2][3], db[2][3]; unsigned cv, Ec = OENT(widx), En, Ln = cnt + widx, hc[6];
    if (col != (unsigned)rcol) {
      const unsigned sg = ((col ^ rcol) - 1) * 2 + (ln & 1), cg = (NC * 64 + 5 - sg) / 6, need = call * (2048 - SPLIT * 8 - cg) - cg - GATE;
      const u64 t0 = __builtin_amdgcn_s_memrealtime();
      while (__builtin_amdgcn_ballot_w64((int)(aload(ws + OFF_Q + (sg * 2 + 1) * 128) - need) < 0) && !give_up(ws, call, t0)) { }
    }
    STAMP(11);
    hv = ld16(rh, ho, Ec * 16); bool good = wait_hdr(hv, rh, ho, Ec * 16, tag, ws, call);
    hc[0] = RL(hv[0], 0); hc[1] = RL(hv[1], 0); hc[2] = RL(hv[3], 0); hc[3] = RL(hv[0], 32); hc[4] = RL(hv[1], 32); hc[5] = RL(hv[3], 32);
#pragma unroll
    for (int i = 0; i < 3; ++i) { da[0][i] = ld16(r0, ln * 16 + i * 1024, Ec * 3072); da[1][i] = ld16(r1, ln * 16 + i * 1024, Ec * 3072); }
    En = OENT(Ln); hv = ld16(rh, ho, En * 16); CLAIM(cv, ctr, 1);
    STAMP(10);
#define OUT_STEP(d, dn) { asm volatile("s_waitcnt vmcnt(0)" : "+v"(d[0][0]), "+v"(d[0][1]), "+v"(d[0][2]), "+v"(d[1][0]), "+v"(d[1][1]), "+v"(d[1][2]), "+v"(hv), "+v"(cv) :: "memory"); \
      unsigned hn[6] = {0, 0, 0, 0, 0, 0}, Eq = 0, Lq = ~0u; \
      if (Ln < 1024) { good = wait_hdr(hv, rh, ho, En * 16, tag, ws, call) && good; \
        hn[0] = RL(hv[0], 0); hn[1] = RL(hv[1], 0); hn[2] = RL(hv[3], 0); hn[3] = RL(hv[0], 32); hn[4] = RL(hv[1], 32); hn[5] = RL(hv[3], 32); \
        _Pragma("unroll") for (int i = 0; i < 3; ++i) { dn[0][i] = ld16(r0, ln * 16 + i * 1024, En * 3072); dn[1][i] = ld16(r1, ln * 16 + i * 1024, En * 3072); } \
        Lq = RFL(cv) - qb; Eq = OENT(Lq); hv = ld16(rh, ho, Eq * 16); CLAIM(cv, ctr, Lq < 1024); } \
      FENCE; \
      unsigned H0[8], H1[8]; \
      unpack_hi(H0, d[0][2], hc[0], hc[1], hc[2], ln, c0, Ec * 2048); unpack_hi(H1, d[1][2], hc[3], hc[4], hc[5], ln, c1, Ec * 2048); \
      const unsigned _ob = ((col >> 1) * 512 + (Ec >> 9) * 256 + ((Ec >> 1) & 1) * 128 + ((Ec >> 4) & 1) * 64 + (Ec & 1) * 32) * SB_ + ((col & 1) * 4096 + ((Ec >> 5) & 15) * 256 + ((Ec >> 2) & 3) * 32) * 2; \
      _Pragma("unroll") for (int c = 0; c < 4; ++c) { u32x4 o; \
        _Pragma("unroll") for (int hh = 0; hh < 2; ++hh) { \
          const unsigned a0 = d[0][c >> 1][(c & 1) * 2 + hh], a1_ = d[1][c >> 1][(c & 1) * 2 + hh], h0 = H0[2 * c + hh], h1 = H1[2 * c + hh]; float v[4]; \
          _Pragma("unroll") for (int e = 0; e < 4; ++e) { const unsigned sel = 0x04000C0Cu + e * 0x01010000u; \
            v[e] = __builtin_bit_cast(float, perm(h0, a0, sel)) + __builtin_bit_cast(float, perm(h1, a1_, sel)); } \
          o[2 * hh] = __builtin_bit_cast(unsigned, __builtin_elementwise_max(__builtin_bit_cast(s16x2, cvt2(v[0], v[1])), (s16x2){0, 0})); \
          o[2 * hh + 1] = __builtin_bit_cast(unsigned, __builtin_elementwise_max(__builtin_bit_cast(s16x2, cvt2(v[2], v[3])), (s16x2){0, 0})); } \
        if (!good) o = (u32x4){0x7FC07FC0u, 0x7FC07FC0u, 0x7FC07FC0u, 0x7FC07FC0u}; \
        __builtin_amdgcn_raw_buffer_store_b128(o, ro, lo + _ob + (c >> 1) * 16 * SB_ + (c & 1) * 256, 0, 16); } \
      FENCE; \
      if (Ln >= 1024) break; \
      Ec = En; En = Eq; Ln = Lq; _Pragma("unroll") for (int i = 0; i < 6; ++i) hc[i] = hn[i]; }
    for (;;) { OUT_STEP(da, db) OUT_STEP(db, da) }
  }
  STAMP(7);
}

extern "C" void kernel_launch(int rank, int nranks, ncclComm_t comm,
                              hipStream_t stream,
                              void* const* d_in, const int* in_sizes, int n_in,
                              void* d_out, int out_size,
                              const int* in_dtypes, int out_dtype,
                              void* const* d_out_all,
                              void* const* d_ws_all, size_t ws_bytes) {
  (void)comm; (void)in_dtypes; (void)out_dtype; (void)d_out_all;
  if (nranks != 8 || (unsigned)rank >= 8u || n_in != 2 || ws_bytes < WS_NEED || in_sizes[0] != 8192 * 1024 || in_sizes[1] != 8192 * 1024 || out_size != 1024 * 8192) {
    fprintf(stderr, "gemm_rs: unexpected problem shape\n"); return; }
  HIP_CHECK(hipFuncSetAttribute((const void*)gemm_rs, hipFuncAttributeMaxDynamicSharedMemorySize, LDS_BYTES));
  Params p;
  p.xin = (const char*)d_in[0]; p.win = (const char*)d_in[1]; p.out = (char*)d_out; p.own = (char*)d_ws_all[rank]; p.rank = rank; p.pad = 0;
  for (int i = 0; i < 8; ++i) p.ws[i] = (char*)d_ws_all[i];
  gemm_rs<<<NWG, 512, LDS_BYTES, stream>>>(p);
  HIP_CHECK(hipGetLastError());
}
```

```cpp
#include <hip/hip_runtime.h>
#include <rccl/rccl.h>
#include <cstdio>

#define HIP_CHECK(expr) do { hipError_t _e = (expr); if (_e != hipSuccess) { \
    fprintf(stderr, "HIP error %s at %s:%d\n", hipGetErrorString(_e), __FILE__, __LINE__); \
    return; } } while (0)

#define LAS __attribute__((address_space(3)))
#define GAS __attribute__((address_space(1)))
typedef short bf16x8 __attribute__((ext_vector_type(8)));
typedef short s16x4 __attribute__((ext_vector_type(4)));
typedef short s16x2 __attribute__((ext_vector_type(2)));
typedef unsigned short u16x2 __attribute__((ext_vector_type(2)));
typedef __bf16 bf16x2 __attribute__((ext_vector_type(2)));
typedef float f32x2 __attribute__((ext_vector_type(2)));
typedef float f32x4 __attribute__((ext_vector_type(4)));
typedef unsigned u32x4 __attribute__((ext_vector_type(4)));
typedef unsigned long long u64;
typedef __amdgpu_buffer_rsrc_t rsrc_t;

#define NC 4
#define PW (NC * 64 / 6)
#define NRED 4
#define RW 4
#define SPLIT 136
#define PAUSE2 18
#define GATE 0
#define DELAYS 0, 10, 8, 6, 4, 3, 1, 0
struct alignas(16) Order { unsigned char e[256]; };
constexpr Order make_order() {
  Order o{}; const int dl[8] = {DELAYS}; int n = 0;
  for (int key = 0; key < 64; ++key)
    for (int op = 0; op < 2; ++op)
      for (int k = 0; k < 16; ++k)
        for (int j = 0; j < 8; ++j) {
          const int tau = op ? (((k & 1) << 3) | (7 - (k >> 1))) : k;
          int kk = 2 * j + (op ? dl[tau & 7] : 0);
          if (op && (tau & 7) == 0 && 3 * j >= PAUSE2 - 7) kk = 40 + j;
          if (kk == key) o.e[n++] = (unsigned char)(op << 7 | tau << 3 | j);
        }
  return o;
}
constexpr Order ORDER_H = make_order();
constexpr bool order_ok() {
  for (int i = SPLIT; i < 256; ++i) { const int e = ORDER_H.e[i], op = e >> 7, tau = (e >> 3) & 15, j = e & 7;
    if ((!op || (tau & 7) == 0) && 3 * j < PAUSE2 - 7) return false; }
  return true;
}
static_assert(order_ok(), "shift 2 would wait for blocks it has to pull itself");
static_assert(NC >= 3 && NC <= 4 && NRED >= 2 && NRED <= 8, "roles");
__constant__ Order ORDER = ORDER_H;

constexpr int NWG = 256, HTB = 16384;
constexpr unsigned SA_ = 2048, SB_ = 16384;
constexpr size_t MiB = 1ull << 20;
constexpr size_t OFF_CALL = 0;
constexpr size_t OFF_TMO = 32768, OFF_Q = 33024, OFF_HAND = 36864;
constexpr size_t OFF_RDY = 40960;
constexpr size_t OFF_HIN = 65536;
constexpr size_t OFF_HOUT = OFF_HIN + 524288;
constexpr size_t OFF_STGX = 1 * MiB, OFF_STGW = 25 * MiB;
constexpr size_t OFF_PIN = 49 * MiB;
constexpr size_t OFF_PINC = 97 * MiB;
constexpr size_t OFF_POUT = 129 * MiB;
constexpr size_t OFF_POUTC = 177 * MiB;
constexpr size_t OFF_DBG = 209 * MiB;
constexpr size_t WS_NEED = 210 * MiB;
constexpr int LDS_BYTES = 8 * HTB + 1024;
constexpr u64 DEADLINE = 100000000ull;

struct Params { const char* xin; const char* win; char* out; char* own; char* ws[8]; int rank; int pad; };

#define RFL(x) __builtin_amdgcn_readfirstlane(x)
#define WAIT_V(n) asm volatile("s_waitcnt vmcnt(" #n ")" ::: "memory")
#define WAIT_L(n) asm volatile("s_waitcnt lgkmcnt(" #n ")" ::: "memory")
#define BAR __builtin_amdgcn_s_barrier()
#define SCHED __builtin_amdgcn_sched_barrier(0)
#define FENCE asm volatile("" ::: "memory")
#define DI __device__ __forceinline__
constexpr int SYS = 17;

DI rsrc_t mk(const void* p, unsigned bytes) { return __builtin_amdgcn_make_buffer_rsrc((void*)p, (short)0, (int)bytes, 0x00020000); }
DI u32x4 ld16(rsrc_t r, unsigned v, unsigned s) { return __builtin_amdgcn_raw_buffer_load_b128(r, v, s, SYS); }
DI u32x4 ld16p(rsrc_t r, unsigned v, unsigned s) { return __builtin_amdgcn_raw_buffer_load_b128(r, v, s, 0); }
DI void st16(u32x4 x, rsrc_t r, unsigned v, unsigned s) { __builtin_amdgcn_raw_buffer_store_b128(x, r, v + s, 0, SYS); }
DI unsigned aload(const char* p) { return __hip_atomic_load((const GAS unsigned*)p, __ATOMIC_RELAXED, __HIP_MEMORY_SCOPE_SYSTEM); }
DI void astore(char* p, unsigned v) { __hip_atomic_store((GAS unsigned*)p, v, __ATOMIC_RELAXED, __HIP_MEMORY_SCOPE_SYSTEM); }
DI unsigned perm(unsigned hi, unsigned lo, unsigned sel) { return __builtin_amdgcn_perm(hi, lo, sel); }
DI unsigned cvt2(float lo, float hi) { return __builtin_bit_cast(unsigned, __builtin_convertvector((f32x2){lo, hi}, bf16x2)); }

#define STAMP(i) do { } while (0)

DI bool give_up(char* ws, unsigned call, u64 t0) {
  if (RFL(aload(ws + OFF_TMO)) == call) return true;
  if (__builtin_amdgcn_s_memrealtime() - t0 > DEADLINE) { astore(ws + OFF_TMO, call); return true; }
  __builtin_amdgcn_s_sleep(2);
  return false;
}

DI u32x4 pack_unit(const u32x4 (&r)[4], unsigned tag, int lane, rsrc_t dst, unsigned doff, rsrc_t dstc, unsigned coff) {
  u16x2 mx = {0, 0};
#pragma unroll
  for (int c = 0; c < 4; ++c)
#pragma unroll
    for (int i = 0; i < 4; ++i) mx = __builtin_elementwise_max(mx, __builtin_bit_cast(u16x2, r[c][i] & 0x7FFF7FFFu));
  const unsigned f = (mx.x > mx.y ? mx.x : mx.y) >> 8;
  unsigned F = 0;
#pragma unroll
  for (int bit = 64; bit; bit >>= 1) F = __builtin_amdgcn_ballot_w64(f >= (F | bit)) ? (F | bit) : F;
  const unsigned base = F > 7 ? F - 7 : 0, B4 = base * 0x01010101u;
  unsigned A[8], H[8], T[8], P[4], e = 0;
#pragma unroll
  for (int c = 0; c < 4; ++c) {
    A[2 * c] = perm(r[c][1], r[c][0], 0x06040200u); A[2 * c + 1] = perm(r[c][3], r[c][2], 0x06040200u);
    H[2 * c] = perm(r[c][1], r[c][0], 0x07050301u); H[2 * c + 1] = perm(r[c][3], r[c][2], 0x07050301u);
    T[2 * c] = (H[2 * c] | 0x80808080u) - B4; T[2 * c + 1] = (H[2 * c + 1] | 0x80808080u) - B4;
    e |= T[2 * c] | T[2 * c + 1];
    P[c] = (H[2 * c] & 0x80808080u) | (T[2 * c] & 0x07070707u) | ((T[2 * c + 1] & 0x07070707u) << 3) | ((H[2 * c + 1] & 0x80808080u) >> 1);
  }
  st16((u32x4){A[0], A[1], A[2], A[3]}, dst, lane * 16, doff);
  st16((u32x4){A[4], A[5], A[6], A[7]}, dst, lane * 16 + 1024, doff);
  st16((u32x4){P[0], P[1], P[2], P[3]}, dst, lane * 16 + 2048, doff);
  unsigned w0 = (tag << 8) | base; u64 pt = 0;
  if (__builtin_amdgcn_ballot_w64((e & 0x78787878u) != 0)) {
    unsigned X[8]; int cnt = 0; unsigned code = 0;
#pragma unroll
    for (int k = 7; k >= 0; --k) {
      X[k] = (H[k] & 0x7F7F7F7Fu) ^ ((T[k] & 0x07070707u) + B4);
      const unsigned nz = ((X[k] >> 3) + 0x0F0F0F0Fu) & 0x10101010u;
      cnt += __builtin_popcount(nz);
      if (X[k]) { const unsigned by = (unsigned)__builtin_ctz(X[k]) >> 3; code = ((unsigned)lane << 10) | (k << 7) | (by << 5) | ((X[k] >> (8 * by + 3)) & 15u); }
    }
    u64 m = __builtin_amdgcn_ballot_w64(cnt != 0);
    if (__builtin_amdgcn_ballot_w64(cnt > 1) || __builtin_popcountll(m) > 4) {
      w0 |= 0x80u;
      st16((u32x4){X[0], X[1], X[2], X[3]}, dstc, lane * 16, coff);
      st16((u32x4){X[4], X[5], X[6], X[7]}, dstc, lane * 16 + 1024, coff);
    } else {
      while (m) { const int l = __builtin_ctzll(m); m &= m - 1; pt = (pt << 16) | (unsigned)__builtin_amdgcn_readlane((int)code, l); }
    }
  }
  return (u32x4){w0, (unsigned)pt, w0, (unsigned)(pt >> 32)};
}
DI void unpack_hi(unsigned (&H)[8], u32x4 P, unsigned hw, unsigned p0, unsigned p1, int lane, rsrc_t srcc, unsigned coff) {
  const unsigned B4 = (hw & 0x7Fu) * 0x01010101u;
#pragma unroll
  for (int c = 0; c < 4; ++c) {
    H[2 * c] = (P[c] & 0x87878787u) + B4;
    H[2 * c + 1] = (((P[c] >> 3) & 0x07070707u) + B4) | ((P[c] << 1) & 0x80808080u);
  }
  if (p0 | (hw & 0x80u)) {
    if (hw & 0x80u) {
      const u32x4 x0 = ld16(srcc, lane * 16, coff), x1 = ld16(srcc, lane * 16 + 1024, coff);
#pragma unroll
      for (int k = 0; k < 4; ++k) { H[k] ^= x0[k]; H[4 + k] ^= x1[k]; }
    } else {
      u64 pt = ((u64)p1 << 32) | p0;
      while (pt) {
        const unsigned code = (unsigned)pt & 0xFFFFu; pt >>= 16;
        const unsigned dv = (unsigned)lane == (code >> 10) ? (code & 15u) << (8 * ((code >> 5) & 3u) + 3) : 0u, kk = (code >> 7) & 7u;
#pragma unroll
        for (int k = 0; k < 8; ++k) H[k] ^= dv & (kk == (unsigned)k ? ~0u : 0u);
      }
    }
  }
}
DI bool wait_hdr(u32x4& h, rsrc_t rh, unsigned voff, unsigned soff, unsigned tag, char* ws, unsigned call) {
  if (!__builtin_amdgcn_ballot_w64((h[0] >> 8) != tag || (h[2] >> 8) != tag)) return true;
  const u64 t0 = __builtin_amdgcn_s_memrealtime();
  for (;;) {
    h = ld16(rh, voff, soff); FENCE;
    if (!__builtin_amdgcn_ballot_w64((h[0] >> 8) != tag || (h[2] >> 8) != tag)) return true;
    if (give_up(ws, call, t0)) { h = (u32x4){0u, 0u, 0u, 0u}; return false; }
  }
}
#define CLAIM(cv, base, on) do { cv = 1u; asm volatile("s_mov_b32 exec_lo, %3\n\ts_mov_b32 exec_hi, 0\n\ts_nop 2\n\tglobal_atomic_add %0, %1, %0, %2 sc0\n\ts_mov_b64 exec, -1" : "+v"(cv) : "v"(0u), "s"(base), "s"(RFL((on) ? 1u : 0u)) : "memory"); } while (0)
#define SIGNAL(base, off) asm volatile("s_mov_b64 exec, 1\n\ts_nop 3\n\tglobal_atomic_add %0, %1, %2\n\ts_mov_b64 exec, -1" :: "v"(off), "v"(1u), "s"(base) : "memory")
#define RL(x, l) ((unsigned)__builtin_amdgcn_readlane((int)(x), (l)))

__global__ __launch_bounds__(512, 2) void gemm_rs(Params p) {
  extern __shared__ __attribute__((aligned(16))) unsigned char smem[];
  LAS unsigned char* lds = (LAS unsigned char*)smem;
  const int tid = threadIdx.x, wid = RFL(tid >> 6), lane = tid & 63, wr = wid >> 2, wc = wid & 3, fr = lane & 15, fq = lane >> 4;
  const int rank = p.rank & 7, ra = rank >> 2, rb = (rank >> 1) & 1, rc = rank & 1, rcol = rank >> 1;
  char* const ws = p.own;
  const int bid = blockIdx.x, xcd = bid & 7, idx = bid >> 3;
  const int tm = (xcd >> 1) * 4 + (idx & 3), tn = (xcd & 1) * 8 + (idx >> 2);
  const int shift = (idx & 3) >= NC ? 0 : (idx >> 2) == 1 ? 1 : (idx >> 2) == 0 ? 2 : 0;
  STAMP(0);
  const unsigned callv = aload(ws + OFF_CALL + bid * 128);
  unsigned tab = ((const unsigned*)ORDER.e)[lane];

  const unsigned ldsw = (unsigned)(size_t)lds + (unsigned)wid * 1024u;
  unsigned vA[4], vB[4];
  { const unsigned voffA = (wid * 8 + (lane >> 3)) * SA_ + 16 * ((lane & 7) ^ (lane >> 3));
    const int s = lane >> 3, pos = (lane & 7) >> 1, hf = lane & 1, cp = pos ^ (s >> 1);
    const unsigned voffB = (16 * (wid >> 2) + 8 * (s >> 2) + 4 * ((wid >> 1) & 1) + (s & 3)) * SB_ + (64 * (wid & 1) + 16 * cp + 8 * hf) * 2;
#pragma unroll
    for (int i = 0; i < 4; ++i) { vA[i] = voffA + i * 64 * SA_; vB[i] = voffB + (i & 1) * 32 * SB_ + (i >> 1) * 256; asm volatile("" : "+v"(vA[i]), "+v"(vB[i])); } }
#define SA(b, h) (((b) * 2 + (h)) * HTB)
#define SB(b, h) ((4 + (b) * 2 + (h)) * HTB)
#define GLDS(voff, sbase, c) asm volatile("s_add_i32 m0, %2, %3\n\ts_nop 0\n\tglobal_load_lds_dwordx4 %0, %1 sc0 sc1" :: "v"(voff), "s"(sbase), "s"(ldsw), "i"(c) : "memory", "scc")
#define STAGEA(b, h, gbase) do { GLDS(vA[2 * (h)], gbase, SA(b, h)); GLDS(vA[2 * (h) + 1], gbase, SA(b, h) + 8192); } while (0)
#define STAGEB(b, h, gbase) do { GLDS(vB[2 * (h)], gbase, SB(b, h)); GLDS(vB[2 * (h) + 1], gbase, SB(b, h) + 8192); } while (0)
  const char* const ownA = p.xin + (size_t)((tm >> 1) * 1024 + ra * 512 + (tm & 1) * 256) * SA_;
  const char* const ownB = p.win + (size_t)(rb * 4096 + tn * 256) * 2;
  const char* a1 = ownA + 128;

  unsigned call, par, tag;
  {
    const unsigned loX = (lane >> 3) * SA_ + (lane & 7) * 16, loW = (lane >> 5) * SB_ + (lane & 31) * 16;
    const rsrc_t rx = mk(p.xin, 16u << 20), rw = mk(p.win, 16u << 20);
    const int np = NWG - 8 * NC, pid = (idx < 4 ? idx : idx - NC) * 8 + xcd;
    int blk[5]; bool val[5];
#pragma unroll
    for (int k = 0; k < 5; ++k) { const int g = k == 0 ? bid : 256 + pid + (k - 1) * np; val[k] = k == 0 || (shift != 1 && g < 1024);
      const int gg = val[k] ? g : bid, e = (((const unsigned*)ORDER.e)[gg >> 4] >> ((gg >> 2) & 3) * 8) & 255, op = e >> 7, tau = (e >> 3) & 15, hv = gg & 1;
      blk[k] = op * 512 + (((gg >> 1) & 1) * 8 + (e & 7)) * 32 + (op ? hv * 16 + tau : (tau >> 1) * 4 + hv * 2 + (tau & 1)); }
    u32x4 r[5][4];
#define PK_LOAD(k) do { const int _op = blk[k] >> 9, _jj = (blk[k] >> 5) & 15, _ix = blk[k] & 31; \
      const unsigned _so = _op ? (_jj * 64 + wid * 8) * SB_ + _ix * 512 : (_ix * 256 + wid * 32) * SA_ + _jj * 128, _cs = _op ? 2 * SB_ : 8 * SA_; \
      _Pragma("unroll") for (int c = 0; c < 4; ++c) r[k][c] = ld16p(_op ? rw : rx, _op ? loW : loX, _so + c * _cs); } while (0)
#define PK_PACK(k) do { const u32x4 _h = pack_unit(r[k], tag, lane, rpi, blk[k] * 24576 + wid * 3072, rpc, blk[k] * 16384 + wid * 2048); \
      if (lane == 0) *(LAS u32x4*)(lds + 8 * HTB + (k) * 128 + wid * 16) = _h; } while (0)
#define PK_SEND(k) do { const int _op = blk[k] >> 9, _jj = (blk[k] >> 5) & 15, _ix = blk[k] & 31, _d = (lane >> 3) & 1; \
      const int _tau = _op ? (_ix & 15) : ((_ix >> 2) * 2 + (_ix & 1)); \
      const int _c0 = _op ? (_ix >> 4) : (_ix & 2), _c1 = _op ? _c0 + 2 : _c0 + 1; \
      char* const _w0 = p.ws[(_c0 * 2 + (_jj >> 3)) & 7]; char* const _w1 = p.ws[(_c1 * 2 + (_jj >> 3)) & 7]; \
      if (lane < 16 && (_d ? _c1 : _c0) != rcol) { const u32x4 _h = *(const LAS u32x4*)(lds + 8 * HTB + (k) * 128 + (lane & 7) * 16); \
        char* _dst = (_d ? _w1 : _w0) + OFF_HIN + (size_t)((par * 8 + rank) * 256 + _op * 128 + _tau * 8 + (_jj & 7)) * 128 + (lane & 7) * 16; \
        asm volatile("global_store_dwordx4 %0, %1, off sc0 sc1\n\ts_nop 1" :: "v"(_dst), "v"(_h) : "memory"); } } while (0)
    PK_LOAD(0);
    call = RFL(callv) + 1; par = call & 1; tag = (call % 0xFFFFFFu) + 1;
    const rsrc_t rpi = mk(ws + OFF_PIN + (size_t)par * 24 * MiB, 24u << 20), rpc = mk(ws + OFF_PINC + (size_t)par * 16 * MiB, 16u << 20);
    PK_PACK(0);
    asm volatile("" : "+v"(tab));
    WAIT_V(0); WAIT_L(0); BAR; FENCE;
    if (wid == 0) PK_SEND(0);
    if (tid == 64) astore(ws + OFF_CALL + bid * 128, call);
    STAMP(1);
#define PROLOGUE() do { STAGEB(0, 0, ownB); STAGEA(0, 0, ownA); STAGEB(0, 1, ownB); STAGEA(0, 1, ownA); \
      STAGEB(1, 0, ownB + 64 * SB_); STAGEA(1, 0, a1); STAGEB(1, 1, ownB + 64 * SB_); } while (0)
    if (shift != 1) {
      PK_LOAD(1); PK_LOAD(2); PK_LOAD(3); PK_LOAD(4); FENCE;
      PK_PACK(1); PK_PACK(2); PK_PACK(3); if (val[4]) PK_PACK(4);
      PROLOGUE();
      WAIT_V(0); WAIT_L(0); BAR; FENCE;
      if (wid == 1) PK_SEND(1);
      if (wid == 2) PK_SEND(2);
      if (wid == 3) PK_SEND(3);
      if (wid == 4 && val[4]) PK_SEND(4);
      if (wr == 1) BAR;
    }
  }
  STAMP(2);

  int aoff0 = (wr * 64 + fr) * 128 + 16 * (fq ^ (fr & 7)), aoff1 = aoff0 ^ 64, boff0, boff1;
  { const int s = (lane >> 2) & 7, pp = ((lane & 1) << 1) | ((lane >> 1) & 1), gh = lane >> 5;
    boff0 = 4 * HTB + gh * 4096 + (wc >> 1) * 1024 + 128 * s + 32 * ((2 * (wc & 1)) ^ (s >> 1)) + 8 * pp; boff1 = boff0 ^ 32; }
  asm volatile("" : "+v"(aoff0), "+v"(aoff1), "+v"(boff0), "+v"(boff1));
#define LDA(dst, b, h) do { _Pragma("unroll") for (int m = 0; m < 4; ++m) { dst[m][0] = *(const LAS bf16x8*)(lds + SA(b, h) + aoff0 + m * 2048); dst[m][1] = *(const LAS bf16x8*)(lds + SA(b, h) + aoff1 + m * 2048); } } while (0)
#define TR(off) __builtin_amdgcn_ds_read_tr16_b64_v4i16((LAS s16x4*)(lds + (off)))
#define LDB(dst, b, h) do { _Pragma("unroll") for (int k = 0; k < 2; ++k) { \
    dst[0][k] = __builtin_shufflevector(TR(SA(b, h) + boff0 + k * 8192), TR(SA(b, h) + boff0 + k * 8192 + 2048), 0, 1, 2, 3, 4, 5, 6, 7); \
    dst[1][k] = __builtin_shufflevector(TR(SA(b, h) + boff1 + k * 8192), TR(SA(b, h) + boff1 + k * 8192 + 2048), 0, 1, 2, 3, 4, 5, 6, 7); } } while (0)
#define MMA(ai, bj, At, Bt) do { __builtin_amdgcn_s_setprio(1); _Pragma("unroll") for (int m = 0; m < 4; ++m) _Pragma("unroll") for (int n = 0; n < 2; ++n) _Pragma("unroll") for (int k = 0; k < 2; ++k) \
    acc[ai][bj][m][n] = __builtin_amdgcn_mfma_f32_16x16x32_bf16(Bt[n][k], At[m][k], acc[ai][bj][m][n], 0, 0, 0); __builtin_amdgcn_s_setprio(0); } while (0)
  f32x4 acc[2][2][4][2];
#pragma unroll
  for (int a = 0; a < 2; ++a)
#pragma unroll
    for (int b = 0; b < 2; ++b)
#pragma unroll
      for (int m = 0; m < 4; ++m)
#pragma unroll
        for (int n = 0; n < 2; ++n) acc[a][b][m][n] = (f32x4){0.f, 0.f, 0.f, 0.f};
  bf16x8 At[4][2], B0[2][2], B1[2][2];
  const char* const sA = ws + OFF_STGX + (size_t)tm * 256 * SA_;
  const char* const sB = ws + OFF_STGW + (size_t)tn * 512;
  const int pause = shift == 1 ? 0 : (shift == 2 ? PAUSE2 : -1);
  int j = 0, s2 = 0, ready = 0;
  for (int it = 0; it < 32; ++it) {
    if (it == pause) {
      if (wr == 0 && shift == 2) BAR;
      STAMP(3);
      int ln = tid; asm volatile("" : "+v"(ln)); ln &= 63;
      const unsigned gw = ((idx & 3) * 8 + xcd) * 8 + wid, s = gw % 6, widx = gw / 6, cnt = PW;
      if (widx < cnt) {
      const unsigned lb = shift == 1 ? 0 : SPLIT * 8, le = shift == 1 ? SPLIT * 8 : 2048;
      const unsigned qb = (call - 1) * (le - lb - cnt) - lb - 2 * cnt;
      char* const ctr = ws + OFF_Q + (s * 2 + shift - 1) * 128; char* const rdy = ws + OFF_RDY;
      const unsigned src = ((((unsigned)rcol ^ (1 + (s >> 1))) << 1) | (s & 1)) & 7;
      const char* const sw = p.ws[src];
      const unsigned lx = (ln >> 3) * SA_ + (ln & 7) * 16, lw = (ln >> 5) * SB_ + (ln & 31) * 16;
      const rsrc_t rs = mk(sw + OFF_PIN + (size_t)par * 24 * MiB, 24u << 20), rsc = mk(sw + OFF_PINC + (size_t)par * 16 * MiB, 16u << 20), rd = mk(ws + OFF_STGX, 48u << 20);
      const rsrc_t rh = mk(ws + OFF_HIN + (size_t)(par * 8 + src) * 32768, 32768);
#define ENT(L) ((((RL(tab, ((L) >> 5) & 63) >> (((L) >> 3) & 3) * 8) & 255u) << 3) | ((L) & 7))
#define IN_SRC(E) (((E) >> 10) ? (512 + (rc * 8 + (((E) >> 3) & 7)) * 32 + rb * 16 + (((E) >> 6) & 15)) * 8 + ((E) & 7) \
                               : ((rc * 8 + (((E) >> 3) & 7)) * 32 + ((E) >> 7) * 4 + ra * 2 + (((E) >> 6) & 1)) * 8 + ((E) & 7))
#define IN_RDY(E) ((((E) >> 6) * 64 + (((E) >> 3) & 7) * 6 + s) * 4)
      u32x4 hv, da[3], db[3]; unsigned cv, Ep = 0, Ec = ENT(lb + widx), En, Ln = lb + cnt + widx, hw, p0, p1; bool first = true;
      hv = ld16(rh, 0, Ec * 16); wait_hdr(hv, rh, 0, Ec * 16, tag, ws, call); hw = RFL(hv[0]); p0 = RFL(hv[1]); p1 = RFL(hv[3]);
      if (shift == 2) { const u64 t0 = __builtin_amdgcn_s_memrealtime(); while (RFL(aload(ws + OFF_HAND + gw * 4)) != call && !give_up(ws, call, t0)) { } }
      STAMP(8);
#pragma unroll
      for (int i = 0; i < 3; ++i) da[i] = ld16(rs, ln * 16 + i * 1024, IN_SRC(Ec) * 3072);
      En = ENT(Ln); hv = ld16(rh, 0, En * 16); CLAIM(cv, ctr, 1);
#define IN_STEP(d, dn) { asm volatile("s_waitcnt vmcnt(0)" : "+v"(d[0]), "+v"(d[1]), "+v"(d[2]), "+v"(hv), "+v"(cv) :: "memory"); \
        unsigned nw = 0, n0 = 0, n1 = 0, Eq = 0, Lq = ~0u; \
        if (Ln < le) { wait_hdr(hv, rh, 0, En * 16, tag, ws, call); nw = RFL(hv[0]); n0 = RFL(hv[1]); n1 = RFL(hv[3]); \
          _Pragma("unroll") for (int i = 0; i < 3; ++i) dn[i] = ld16(rs, ln * 16 + i * 1024, IN_SRC(En) * 3072); \
          Lq = RFL(cv) - qb; Eq = ENT(Lq); hv = ld16(rh, 0, Eq * 16); CLAIM(cv, ctr, Lq < le); \
          if (Lq >= le && shift == 1 && ln == 0) astore(ws + OFF_HAND + gw * 4, call); } \
        if (!first) SIGNAL(rdy, IN_RDY(Ep)); \
        FENCE; \
        unsigned H[8]; unpack_hi(H, d[2], hw, p0, p1, ln, rsc, IN_SRC(Ec) * 2048); \
        const unsigned _op = Ec >> 10, _tau = (Ec >> 6) & 15, _j = (Ec >> 3) & 7, _u = Ec & 7, _cs = _op ? 2 * SB_ : 8 * SA_, _lo = _op ? lw : lx; \
        const unsigned _db = _op ? 24 * (unsigned)MiB + ((s >> 1) * 512 + _j * 64 + _u * 8) * SB_ + (s & 1) * 8192 + _tau * 512 : ((s >> 1) * 4096 + _tau * 256 + _u * 32) * SA_ + (s & 1) * 1024 + _j * 128; \
        _Pragma("unroll") for (int c = 0; c < 4; ++c) { const unsigned a0 = d[c >> 1][(c & 1) * 2], a1_ = d[c >> 1][(c & 1) * 2 + 1]; \
          st16((u32x4){perm(H[2 * c], a0, 0x05010400u), perm(H[2 * c], a0, 0x07030602u), perm(H[2 * c + 1], a1_, 0x05010400u), perm(H[2 * c + 1], a1_, 0x07030602u)}, rd, _lo, _db + c * _cs); } \
        FENCE; first = false; Ep = Ec; \
        if (Ln >= le) break; \
        Ec = En; hw = nw; p0 = n0; p1 = n1; En = Eq; Ln = Lq; }
      for (;;) { IN_STEP(da, db) IN_STEP(db, da) }
      WAIT_V(0);
      SIGNAL(rdy, IN_RDY(Ep));
      }
      if (shift == 1) { PROLOGUE(); WAIT_V(0); }
      STAMP(4);
      if (shift == 1) BAR;
      if (wr == 1) BAR;
    }
    const char *a2, *b2, *a3, *b3;
    if (it < 7 || it == 31) { const int t = (it == 31) ? 0 : 2 * it + 2; a2 = ownA + t * 128; b2 = ownB + (size_t)t * 64 * SB_; a3 = a2 + 128; b3 = b2 + 64 * SB_; }
    else {
      if (ready < 2 * it - 12) {
        int ln = tid; asm volatile("" : "+v"(ln)); ln &= 63;
        const u64 t0 = __builtin_amdgcn_s_memrealtime();
        for (;;) {
          const unsigned fx = aload(ws + OFF_RDY + (tm * 64 + ln) * 4), fw = aload(ws + OFF_RDY + ((16 + tn) * 64 + ln) * 4);
          ready = __builtin_ctzll(~__builtin_amdgcn_ballot_w64(fx == 8 * call && fw == 8 * call && ln < 48));
          if (ready >= 2 * it - 12) break;
          if (give_up(ws, call, t0)) { ready = 64; break; }
        }
        FENCE;
      }
      a2 = sA + (size_t)s2 * (4096 * SA_) + j * 128; b2 = sB + (size_t)(s2 * 512 + j * 64) * SB_; a3 = a2 + 1024; b3 = b2 + 8192; if (++s2 == 3) { s2 = 0; ++j; }
    }
    LDB(B0, 0, 0); SCHED; LDA(At, 0, 0); STAGEA(1, 1, a1);
    WAIT_L(8); BAR; WAIT_L(0); MMA(0, 0, At, B0); BAR; SCHED;
    LDB(B1, 0, 1); STAGEB(0, 0, b2);
    BAR; WAIT_L(0); MMA(0, 1, At, B1); BAR;
    LDA(At, 0, 1); STAGEA(0, 0, a2);
    BAR; WAIT_L(0); MMA(1, 0, At, B0); BAR; SCHED;
    STAGEB(0, 1, b2);
    WAIT_V(6); BAR; MMA(1, 1, At, B1); BAR;
    LDB(B0, 1, 0); SCHED; LDA(At, 1, 0); STAGEA(0, 1, a2);
    WAIT_L(8); BAR; WAIT_L(0); MMA(0, 0, At, B0); BAR; SCHED;
    LDB(B1, 1, 1); STAGEB(1, 0, b3);
    BAR; WAIT_L(0); MMA(0, 1, At, B1); BAR;
    LDA(At, 1, 1); STAGEA(1, 0, a3);
    BAR; WAIT_L(0); MMA(1, 0, At, B0); BAR; SCHED;
    STAGEB(1, 1, b3);
    WAIT_V(6); BAR; MMA(1, 1, At, B1); BAR;
    a1 = a3;
  }
  if (wr == 0) BAR;
  STAMP(5);

  int ln = tid; asm volatile("" : "+v"(ln)); ln &= 63;
  {
    const bool bad = RFL(aload(ws + OFF_TMO)) == call;
    const unsigned slot = (tm * 16 + tn) * 8 + wid;
    const rsrc_t rpo = mk(ws + OFF_POUT + (size_t)par * 24 * MiB + (size_t)slot * 12288, 12288), rpc = mk(ws + OFF_POUTC + (size_t)par * 16 * MiB + (size_t)slot * 8192, 8192);
    u32x4 hs = {0u, 0u, 0u, 0u};
#pragma unroll
    for (int u = 0; u < 4; ++u) {
      u32x4 r[4];
#pragma unroll
      for (int c = 0; c < 4; ++c) {
        const f32x4 x = acc[u >> 1][c & 1][(u & 1) * 2 + (c >> 1)][0], y = acc[u >> 1][c & 1][(u & 1) * 2 + (c >> 1)][1];
        const auto s0 = __builtin_amdgcn_permlane32_swap(cvt2(x[0], x[1]), cvt2(y[0], y[1]), false, false);
        const auto s1 = __builtin_amdgcn_permlane32_swap(cvt2(x[2], x[3]), cvt2(y[2], y[3]), false, false);
        r[c] = (u32x4){s0[0], s1[0], s0[1], s1[1]};
        if (bad) r[c] = (u32x4){0x7FC07FC0u, 0x7FC07FC0u, 0x7FC07FC0u, 0x7FC07FC0u};
      }
      const u32x4 h = pack_unit(r, tag, ln, rpo, u * 3072, rpc, u * 2048);
      if (ln == u) hs = h;
    }
    WAIT_V(0);
    if (ln < 4) {
      char* dst = p.ws[(tm >> 1) & 7] + OFF_HOUT + (size_t)(((par * 8 + rank) * 32 + (tm & 1) * 16 + tn) * 8 + wid) * 64 + ln * 16;
      asm volatile("global_store_dwordx4 %0, %1, off sc0 sc1\n\ts_nop 1" :: "v"(dst), "v"(hs) : "memory");
    }
  }
  STAMP(6);

  if (idx >= 32 - NRED && wid < RW) {
    const unsigned gw = ((31 - idx) * 8 + xcd) * RW + wid, col = gw & 3, widx = gw >> 2, cnt = NRED * 2 * RW;
    const unsigned qb = (call - 1) * (1024 - cnt) - 2 * cnt;
    char* const ctr = ws + OFF_Q + (12 + col) * 128;
    const char* const w0 = p.ws[col * 2]; const char* const w1 = p.ws[col * 2 + 1];
    const size_t mine = (size_t)rank * 1024;
    const rsrc_t r0 = mk(w0 + OFF_POUT + (size_t)par * 24 * MiB + mine * 3072, 3u << 20), r1 = mk(w1 + OFF_POUT + (size_t)par * 24 * MiB + mine * 3072, 3u << 20);
    const rsrc_t c0 = mk(w0 + OFF_POUTC + (size_t)par * 16 * MiB + mine * 2048, 2u << 20), c1 = mk(w1 + OFF_POUTC + (size_t)par * 16 * MiB + mine * 2048, 2u << 20);
    const rsrc_t rh = mk(ws + OFF_HOUT + (size_t)(par * 8 + col * 2) * 16384, 32768), ro = mk(p.out, 16u << 20);
    const unsigned lo = (ln & 15) * SB_ + (ln >> 4) * 16, ho = (ln >> 5) * 16384;
#define OENT(L) (((((L) >> 6) & 1) << 9) | ((((L) >> 5) & 1) << 8) | ((7 - (((L) >> 7) & 7)) << 5) | ((L) & 31))
    u32x4 hv, da[2][3], db[2][3]; unsigned cv, Ec = OENT(widx), En, Ln = cnt + widx, hc[6];
    if (col != (unsigned)rcol) {
      const unsigned sg = ((col ^ rcol) - 1) * 2 + (ln & 1), cg = PW, need = call * (2048 - SPLIT * 8 - cg) - cg - GATE;
      const u64 t0 = __builtin_amdgcn_s_memrealtime();
      while (__builtin_amdgcn_ballot_w64((int)(aload(ws + OFF_Q + (sg * 2 + 1) * 128) - need) < 0) && !give_up(ws, call, t0)) { }
    }
    STAMP(11);
    hv = ld16(rh, ho, Ec * 16); bool good = wait_hdr(hv, rh, ho, Ec * 16, tag, ws, call);
    hc[0] = RL(hv[0], 0); hc[1] = RL(hv[1], 0); hc[2] = RL(hv[3], 0); hc[3] = RL(hv[0], 32); hc[4] = RL(hv[1], 32); hc[5] = RL(hv[3], 32);
#pragma unroll
    for (int i = 0; i < 3; ++i) { da[0][i] = ld16(r0, ln * 16 + i * 1024, Ec * 3072); da[1][i] = ld16(r1, ln * 16 + i * 1024, Ec * 3072); }
    En = OENT(Ln); hv = ld16(rh, ho, En * 16); CLAIM(cv, ctr, 1);
    STAMP(10);
#define OUT_STEP(d, dn) { asm volatile("s_waitcnt vmcnt(0)" : "+v"(d[0][0]), "+v"(d[0][1]), "+v"(d[0][2]), "+v"(d[1][0]), "+v"(d[1][1]), "+v"(d[1][2]), "+v"(hv), "+v"(cv) :: "memory"); \
      unsigned hn[6] = {0, 0, 0, 0, 0, 0}, Eq = 0, Lq = ~0u; \
      if (Ln < 1024) { good = wait_hdr(hv, rh, ho, En * 16, tag, ws, call) && good; \
        hn[0] = RL(hv[0], 0); hn[1] = RL(hv[1], 0); hn[2] = RL(hv[3], 0); hn[3] = RL(hv[0], 32); hn[4] = RL(hv[1], 32); hn[5] = RL(hv[3], 32); \
        _Pragma("unroll") for (int i = 0; i < 3; ++i) { dn[0][i] = ld16(r0, ln * 16 + i * 1024, En * 3072); dn[1][i] = ld16(r1, ln * 16 + i * 1024, En * 3072); } \
        Lq = RFL(cv) - qb; Eq = OENT(Lq); hv = ld16(rh, ho, Eq * 16); CLAIM(cv, ctr, Lq < 1024); } \
      FENCE; \
      unsigned H0[8], H1[8]; \
      unpack_hi(H0, d[0][2], hc[0], hc[1], hc[2], ln, c0, Ec * 2048); unpack_hi(H1, d[1][2], hc[3], hc[4], hc[5], ln, c1, Ec * 2048); \
      const unsigned _ob = ((col >> 1) * 512 + (Ec >> 9) * 256 + ((Ec >> 1) & 1) * 128 + ((Ec >> 4) & 1) * 64 + (Ec & 1) * 32) * SB_ + ((col & 1) * 4096 + ((Ec >> 5) & 15) * 256 + ((Ec >> 2) & 3) * 32) * 2; \
      _Pragma("unroll") for (int c = 0; c < 4; ++c) { u32x4 o; \
        _Pragma("unroll") for (int hh = 0; hh < 2; ++hh) { \
          const unsigned a0 = d[0][c >> 1][(c & 1) * 2 + hh], a1_ = d[1][c >> 1][(c & 1) * 2 + hh], h0 = H0[2 * c + hh], h1 = H1[2 * c + hh]; float v[4]; \
          _Pragma("unroll") for (int e = 0; e < 4; ++e) { const unsigned sel = 0x04000C0Cu + e * 0x01010000u; \
            v[e] = __builtin_bit_cast(float, perm(h0, a0, sel)) + __builtin_bit_cast(float, perm(h1, a1_, sel)); } \
          o[2 * hh] = __builtin_bit_cast(unsigned, __builtin_elementwise_max(__builtin_bit_cast(s16x2, cvt2(v[0], v[1])), (s16x2){0, 0})); \
          o[2 * hh + 1] = __builtin_bit_cast(unsigned, __builtin_elementwise_max(__builtin_bit_cast(s16x2, cvt2(v[2], v[3])), (s16x2){0, 0})); } \
        if (!good) o = (u32x4){0x7FC07FC0u, 0x7FC07FC0u, 0x7FC07FC0u, 0x7FC07FC0u}; \
        __builtin_amdgcn_raw_buffer_store_b128(o, ro, lo + _ob + (c >> 1) * 16 * SB_ + (c & 1) * 256, 0, 16); } \
      FENCE; \
      if (Ln >= 1024) break; \
      Ec = En; En = Eq; Ln = Lq; _Pragma("unroll") for (int i = 0; i < 6; ++i) hc[i] = hn[i]; }
    for (;;) { OUT_STEP(da, db) OUT_STEP(db, da) }
  }
  STAMP(7);
}

extern "C" void kernel_launch(int rank, int nranks, ncclComm_t comm,
                              hipStream_t stream,
                              void* const* d_in, const int* in_sizes, int n_in,
                              void* d_out, int out_size,
                              const int* in_dtypes, int out_dtype,
                              void* const* d_out_all,
                              void* const* d_ws_all, size_t ws_bytes) {
  (void)comm; (void)in_dtypes; (void)out_dtype; (void)d_out_all;
  if (nranks != 8 || (unsigned)rank >= 8u || n_in != 2 || ws_bytes < WS_NEED || in_sizes[0] != 8192 * 1024 || in_sizes[1] != 8192 * 1024 || out_size != 1024 * 8192) {
    fprintf(stderr, "gemm_rs: unexpected problem shape\n"); return; }
  HIP_CHECK(hipFuncSetAttribute((const void*)gemm_rs, hipFuncAttributeMaxDynamicSharedMemorySize, LDS_BYTES));
  Params p;
  p.xin = (const char*)d_in[0]; p.win = (const char*)d_in[1]; p.out = (char*)d_out; p.own = (char*)d_ws_all[rank]; p.rank = rank; p.pad = 0;
  for (int i = 0; i < 8; ++i) p.ws[i] = (char*)d_ws_all[i];
  gemm_rs<<<NWG, 512, LDS_BYTES, stream>>>(p);
  HIP_CHECK(hipGetLastError());
}
```

```cpp
#include <hip/hip_runtime.h>
#include <rccl/rccl.h>
#include <cstdio>

#define HIP_CHECK(expr) do { hipError_t _e = (expr); if (_e != hipSuccess) { \
    fprintf(stderr, "HIP error %s at %s:%d\n", hipGetErrorString(_e), __FILE__, __LINE__); \
    return; } } while (0)

#define LAS __attribute__((address_space(3)))
#define GAS __attribute__((address_space(1)))
typedef short bf16x8 __attribute__((ext_vector_type(8)));
typedef short s16x4 __attribute__((ext_vector_type(4)));
typedef short s16x2 __attribute__((ext_vector_type(2)));
typedef unsigned short u16x2 __attribute__((ext_vector_type(2)));
typedef __bf16 bf16x2 __attribute__((ext_vector_type(2)));
typedef float f32x2 __attribute__((ext_vector_type(2)));
typedef float f32x4 __attribute__((ext_vector_type(4)));
typedef unsigned u32x2 __attribute__((ext_vector_type(2)));
typedef unsigned u32x3 __attribute__((ext_vector_type(3)));
typedef unsigned u32x4 __attribute__((ext_vector_type(4)));
typedef unsigned long long u64;
typedef __amdgpu_buffer_rsrc_t rsrc_t;

#define PW 30
#define NRED 4
#define RW 4
#define NPUSH 6
#define SPLIT 120
#define PAUSE2 16
#define GATE 64
#define DELAYS 0, 15, 13, 10, 8, 5, 3, 0
#define PACKS 0, 0, 7, 7, 5, 4, 3, 0
#define B96 0
#define EPI2 1
constexpr int NC = 4, NENT = 256 - NPUSH;
struct alignas(16) Order { unsigned char e[256]; short pk[8][8]; };
constexpr Order make_order() {
  Order o{}; const int dl[8] = {DELAYS}; int n = 0;
  for (int key = 0; key < 96; ++key)
    for (int op = 1; op >= 0; --op)
      for (int k = 0; k < 16; ++k)
        for (int j = 0; j < 8; ++j) {
          const int tau = op ? (((k & 1) << 3) | (7 - (k >> 1))) : k;
          int kk = 3 * j + (op ? dl[tau & 7] : k < 6 ? 0 : k < 11 ? 1 : 2);
          if (op && (tau & 7) == 0 && 3 * j >= PAUSE2 - 7) kk = 80 + j;
          if (kk == key && (op || j || k >= NPUSH)) o.e[n++] = (unsigned char)(op << 7 | tau << 3 | j);
        }
  const int pc[8] = {PACKS}; n = 224 - 6 * NPUSH;
  for (int k = 0; k < 8; ++k)
    for (int g = 7; g >= 0; --g) { o.pk[g][k] = (short)(pc[g] > k ? n : 1024); n += pc[g] > k ? 32 : 0; }
  return o;
}
constexpr Order ORDER_H = make_order();
constexpr bool order_ok() {
  for (int i = SPLIT; i < NENT; ++i) { const int e = ORDER_H.e[i], op = e >> 7, tau = (e >> 3) & 15, j = e & 7;
    if ((!op || (tau & 7) == 0) && 3 * j < PAUSE2 - 7) return false; }
  const int pc[8] = {PACKS}; int n = 224 - 6 * NPUSH;
  for (int g = 0; g < 8; ++g) n += 32 * pc[g];
  return n >= 4 * NENT && pc[1] == 0 && pc[2] && pc[3] && pc[4];
}
constexpr unsigned more_mask(int k) { unsigned m = 0; for (int g = 0; g < 8; ++g) m |= (ORDER_H.pk[g][k] < 4 * NENT ? 1u : 0u) << g; return m; }
static_assert(order_ok(), "shift 2 would wait for blocks it has to pull itself, or blocks stay unpacked");
static_assert(NRED >= 2 && NRED <= 8 && PW <= 42 && NPUSH >= 0 && NPUSH <= 12, "roles");
__constant__ Order ORDER = ORDER_H;

constexpr int NWG = 256, HTB = 16384;
constexpr unsigned SA_ = 2048, SB_ = 16384;
constexpr size_t MiB = 1ull << 20;
constexpr size_t OFF_CALL = 0;
constexpr size_t OFF_TMO = 32768, OFF_Q = 33024, OFF_HAND = 36864;
constexpr size_t OFF_RDY = 40960;
constexpr size_t OFF_HIN = 65536;
constexpr size_t OFF_HOUT = OFF_HIN + 262144;
constexpr size_t OFF_STG = 1 * MiB;
constexpr size_t OFF_PIN = 73 * MiB;
constexpr size_t OFF_PINC = 121 * MiB;
constexpr size_t OFF_POUT = 153 * MiB;
constexpr size_t OFF_POUTC = 201 * MiB;
constexpr size_t OFF_DBG = 233 * MiB;
constexpr size_t WS_NEED = 234 * MiB;
constexpr int SCR = 2448, LDS_BYTES = 8 * HTB + 1024 + 8 * SCR;
constexpr unsigned OOB = 0x7FFFFF00u;
constexpr u64 DEADLINE = 100000000ull;

struct Params { const char* xin; const char* win; char* out; char* own; char* ws[8]; int rank; int pad; };

#define RFL(x) __builtin_amdgcn_readfirstlane(x)
#define WAIT_V(n) asm volatile("s_waitcnt vmcnt(" #n ")" ::: "memory")
#define WAIT_L(n) asm volatile("s_waitcnt lgkmcnt(" #n ")" ::: "memory")
#define DRAIN do { asm volatile("" ::: "memory"); __builtin_amdgcn_s_waitcnt(0x0F70); asm volatile("" ::: "memory"); } while (0)
#define BAR __builtin_amdgcn_s_barrier()
#define SCHED __builtin_amdgcn_sched_barrier(0)
#define FENCE asm volatile("" ::: "memory")
#define DI __device__ __forceinline__
constexpr int SYS = 17;

DI rsrc_t mk(const void* p, unsigned bytes) { return __builtin_amdgcn_make_buffer_rsrc((void*)p, (short)0, (int)bytes, 0x00020000); }
DI u32x4 ld16(rsrc_t r, unsigned v, unsigned s) { return __builtin_amdgcn_raw_buffer_load_b128(r, v, s, SYS); }
DI u32x4 ld16p(rsrc_t r, unsigned v, unsigned s) { return __builtin_amdgcn_raw_buffer_load_b128(r, v, s, 0); }
DI void st16(u32x4 x, rsrc_t r, unsigned v, unsigned s) { __builtin_amdgcn_raw_buffer_store_b128(x, r, v + s, 0, SYS); }
DI unsigned aload(const char* p) { return __hip_atomic_load((const GAS unsigned*)p, __ATOMIC_RELAXED, __HIP_MEMORY_SCOPE_SYSTEM); }
DI void astore(char* p, unsigned v) { __hip_atomic_store((GAS unsigned*)p, v, __ATOMIC_RELAXED, __HIP_MEMORY_SCOPE_SYSTEM); }
DI unsigned perm(unsigned hi, unsigned lo, unsigned sel) { return __builtin_amdgcn_perm(hi, lo, sel); }
DI unsigned cvt2(float lo, float hi) { return __builtin_bit_cast(unsigned, __builtin_convertvector((f32x2){lo, hi}, bf16x2)); }

#define STAMP(i) do { } while (0)

DI bool give_up(char* ws, unsigned call, u64 t0) {
  const u64 dt = __builtin_amdgcn_s_memrealtime() - t0;
  if (dt > 10000) {
    if (RFL(aload(ws + OFF_TMO)) == call) return true;
    if (dt > DEADLINE) { astore(ws + OFF_TMO, call); return true; }
  }
  __builtin_amdgcn_s_sleep(2);
  return false;
}

struct Unit { u32x4 a0, a1; u32x3 p; unsigned pt; };
DI unsigned bfi(unsigned m, unsigned a, unsigned b) { unsigned r; asm("v_bfi_b32 %0, %1, %2, %3" : "=v"(r) : "s"(m), "v"(a), "v"(b)); return r; }
DI unsigned pka(unsigned a, unsigned b) { return __builtin_bit_cast(unsigned, __builtin_bit_cast(u16x2, a) + __builtin_bit_cast(u16x2, b)); }
DI unsigned pks(unsigned a, unsigned b) { return __builtin_bit_cast(unsigned, __builtin_bit_cast(u16x2, a) - __builtin_bit_cast(u16x2, b)); }
DI unsigned pick_base(unsigned f, int th) {
  unsigned g = RFL(f);
  while (__builtin_popcountll(__builtin_amdgcn_ballot_w64(f > g)) >= th) ++g;
  while (g > 7 && __builtin_popcountll(__builtin_amdgcn_ballot_w64(f >= g)) < th) --g;
  return g > 7 ? g - 7 : 0;
}
DI unsigned base_of(const u32x4 (&r)[4]) {
  u16x2 mx = {0, 0};
#pragma unroll
  for (int c = 0; c < 4; ++c)
#pragma unroll
    for (int i = 0; i < 4; i += 2) mx = __builtin_elementwise_max(mx, __builtin_bit_cast(u16x2, r[c][i] & 0x7FFF7FFFu));
  return pick_base((mx.x > mx.y ? mx.x : mx.y) >> 7, 9);
}
DI unsigned pack_unit(const u32x4 (&r)[4], unsigned base, int lane, LAS unsigned char* scr, rsrc_t dst, unsigned doff, rsrc_t dstc, unsigned coff) {
  const unsigned B2 = base * 0x00800080u, M = 0x83838383u;
  unsigned A[8], T[8], mask = 0, n = 0;
#pragma unroll
  for (int k = 0; k < 8; ++k) {
    const unsigned lo = pks(r[k >> 1][(k & 1) * 2], B2), hi = pks(r[k >> 1][(k & 1) * 2 + 1], B2);
    A[k] = perm(hi, lo, 0x06040200u); T[k] = perm(hi, lo, 0x07050301u);
    mask = bfi(0x80808080u, (T[k] & 0x7C7C7C7Cu) + 0x7C7C7C7Cu, mask >> 1);
  }
  *(LAS u32x4*)(scr + lane * 32) = (u32x4){T[0], T[1], T[2], T[3]}; *(LAS u32x4*)(scr + lane * 32 + 16) = (u32x4){T[4], T[5], T[6], T[7]};
  st16((u32x4){A[0], A[1], A[2], A[3]}, dst, lane * 16, doff);
  st16((u32x4){A[4], A[5], A[6], A[7]}, dst, lane * 16 + 1024, doff);
  const unsigned P0 = bfi(0x60606060u, T[6] << 5, bfi(M, T[0], __builtin_rotateleft32(T[1], 3)));
  const unsigned P1 = bfi(0x60606060u, T[7] << 5, bfi(M, T[2], __builtin_rotateleft32(T[3], 3)));
  const unsigned P2 = bfi(0x40404040u, T[6] >> 1, bfi(0x20202020u, T[7] >> 2, bfi(M, T[4], __builtin_rotateleft32(T[5], 3))));
  __builtin_amdgcn_raw_buffer_store_b64((u32x2){P0, P1}, dst, lane * 8 + 2048 + doff, 0, SYS);
  __builtin_amdgcn_raw_buffer_store_b32(P2, dst, lane * 4 + 2560 + doff, 0, SYS);
  FENCE;
  for (u64 m; n <= 128 && (m = __builtin_amdgcn_ballot_w64(mask != 0)) != 0; n += __builtin_popcountll(m)) {
    const unsigned i = __builtin_amdgcn_mbcnt_hi((unsigned)(m >> 32), __builtin_amdgcn_mbcnt_lo((unsigned)m, n));
    const unsigned b = __builtin_ctz(mask | 0x80000000u), a = lane * 32 + ((b & 7) << 2 | b >> 3);
    *(LAS unsigned short*)(scr + 2048 + (mask ? i : 192u) * 2) = (unsigned short)((scr[a] & 0x7Cu) << 9 | a);
    mask &= mask - 1;
  }
  FENCE;
  if (n > 128) {
    n = 255;
    st16((u32x4){T[0], T[1], T[2], T[3]} & 0x7C7C7C7Cu, dstc, lane * 16, coff);
    st16((u32x4){T[4], T[5], T[6], T[7]} & 0x7C7C7C7Cu, dstc, lane * 16 + 1024, coff);
  } else __builtin_amdgcn_raw_buffer_store_b32(*(const LAS unsigned*)(scr + 2048 + lane * 4), dst, 2u * lane < n ? lane * 4 + 2816 + doff : OOB, 0, SYS);
  FENCE;
  return base | n << 8;
}
constexpr int NLD = B96 ? 4 : 5;
DI void load_unit(Unit& d, rsrc_t src, unsigned off, unsigned hw, int lane) {
  d.pt = __builtin_amdgcn_raw_buffer_load_b32(src, 2u * lane < (hw >> 8) ? lane * 4 + 2816 : OOB, off, SYS);
  const u32x2 q = __builtin_amdgcn_raw_buffer_load_b64(src, lane * 8 + 2048, off, SYS);
  d.p = (u32x3){q[0], q[1], __builtin_amdgcn_raw_buffer_load_b32(src, lane * 4 + 2560, off, SYS)};
  d.a0 = ld16(src, lane * 16, off); d.a1 = ld16(src, lane * 16 + 1024, off);
}
DI void unpack_hi(unsigned (&T)[8], const Unit& d, unsigned hw, int lane, LAS unsigned char* scr, rsrc_t srcc, unsigned coff) {
  const unsigned n = hw >> 8, M = 0x83838383u;
  u32x4 x0, x1;
  if (n > 128) { x0 = ld16(srcc, lane * 16, coff); x1 = ld16(srcc, lane * 16 + 1024, coff); DRAIN; }
  else {
    const unsigned a0 = d.pt & 0x7FFu, a1 = (d.pt >> 16) & 0x7FFu;
    if (2u * lane < n) scr[a0] = (unsigned char)((d.pt >> 9) & 0x7Cu);
    if (2u * lane + 1 < n) scr[a1] = (unsigned char)((d.pt >> 25) & 0x7Cu);
    FENCE;
    x0 = *(const LAS u32x4*)(scr + lane * 32); x1 = *(const LAS u32x4*)(scr + lane * 32 + 16);
    FENCE;
    if (2u * lane < n) { scr[a0] = 0; scr[a1] = 0; }
    FENCE;
  }
  T[0] = (d.p[0] & M) | x0[0]; T[1] = (__builtin_rotateright32(d.p[0], 3) & M) | x0[1];
  T[2] = (d.p[1] & M) | x0[2]; T[3] = (__builtin_rotateright32(d.p[1], 3) & M) | x0[3];
  T[4] = (d.p[2] & M) | x1[0]; T[5] = (__builtin_rotateright32(d.p[2], 3) & M) | x1[1];
  T[6] = (bfi(0x03030303u, d.p[0] >> 5, d.p[2] << 1) & M) | x1[2];
  T[7] = (bfi(0x03030303u, d.p[1] >> 5, d.p[2] << 2) & M) | x1[3];
}
DI u32x4 unpack_row(const unsigned (&T)[8], const Unit& d, int c, unsigned hw) {
  const unsigned B2 = (hw & 255u) * 0x00800080u, a = (c & 2 ? d.a1 : d.a0)[(c & 1) * 2], b = (c & 2 ? d.a1 : d.a0)[(c & 1) * 2 + 1];
  return (u32x4){pka(perm(T[2 * c], a, 0x05010400u), B2), pka(perm(T[2 * c], a, 0x07030602u), B2), pka(perm(T[2 * c + 1], b, 0x05010400u), B2), pka(perm(T[2 * c + 1], b, 0x07030602u), B2)};
}
DI void zero_scr(LAS unsigned char* scr, int lane) { *(LAS u32x4*)(scr + lane * 32) = (u32x4){0u, 0u, 0u, 0u}; *(LAS u32x4*)(scr + lane * 32 + 16) = (u32x4){0u, 0u, 0u, 0u}; FENCE; }
DI u32x2 ld8(rsrc_t r, unsigned v, unsigned s) { return __builtin_amdgcn_raw_buffer_load_b64(r, v, s, SYS); }
DI bool wait_hdr(u32x2& h, rsrc_t rh, unsigned voff, unsigned soff, char* ws, unsigned call) {
  if (!__builtin_amdgcn_ballot_w64(h[0] != call)) return true;
  const u64 t0 = __builtin_amdgcn_s_memrealtime();
  for (;;) {
    h = ld8(rh, voff, soff); FENCE;
    if (!__builtin_amdgcn_ballot_w64(h[0] != call)) return true;
    if (give_up(ws, call, t0)) { h = (u32x2){0u, 0u}; return false; }
  }
}
#define CLAIM(cv, base, on) do { cv = 1u; asm volatile("s_mov_b32 exec_lo, %3\n\ts_mov_b32 exec_hi, 0\n\ts_nop 2\n\tglobal_atomic_add %0, %1, %0, %2 sc0\n\ts_mov_b64 exec, -1" : "+v"(cv) : "v"(0u), "s"(base), "s"(RFL((on) ? 1u : 0u)) : "memory"); } while (0)
#define SIGNAL(base, off) asm volatile("s_mov_b64 exec, 1\n\ts_nop 3\n\tglobal_atomic_add %0, %1, %2\n\ts_mov_b64 exec, -1" :: "v"(off), "v"(1u), "s"(base) : "memory")
#define RL(x, l) ((unsigned)__builtin_amdgcn_readlane((int)(x), (l)))

__global__ __launch_bounds__(512, 2) void gemm_rs(Params p) {
  extern __shared__ __attribute__((aligned(16))) unsigned char smem[];
  LAS unsigned char* lds = (LAS unsigned char*)smem;
  const int tid = threadIdx.x, wid = RFL(tid >> 6), lane = tid & 63, wr = wid >> 2, wc = wid & 3, fr = lane & 15, fq = lane >> 4;
  unsigned tab = ((const unsigned*)ORDER.e)[lane];
  asm volatile("" :: "s"(p.xin), "s"(p.win), "s"(p.own), "s"(p.rank), "s"(p.ws[(blockIdx.x & 3) * 2 + ((blockIdx.x >> 2) & 1)]) : "memory");
  const int rank = p.rank & 7, ra = rank >> 2, rb = (rank >> 1) & 1, rc = rank & 1, rcol = rank >> 1;
  char* const ws = p.own;
  const int bid = blockIdx.x, xcd = bid & 7, idx = bid >> 3;
  const int tm = (xcd >> 1) * 4 + (idx & 3), tn = (xcd & 1) * 8 + (idx >> 2);
  const int shift = (idx >> 2) == 1 ? 1 : (idx >> 2) == 0 ? 2 : 0;
  LAS unsigned char* const scr = lds + 8 * HTB + 1024 + wid * SCR;
  STAMP(0);
  unsigned callv = aload(ws + OFF_CALL + bid * 128);

  const unsigned ldsw = (unsigned)(size_t)lds + (unsigned)wid * 1024u;
  unsigned vA[4], vB[4];
  { const unsigned voffA = (wid * 8 + (lane >> 3)) * SA_ + 16 * ((lane & 7) ^ (lane >> 3));
    const int s = lane >> 3, pos = (lane & 7) >> 1, hf = lane & 1, cp = pos ^ (s >> 1);
    const unsigned voffB = (16 * (wid >> 2) + 8 * (s >> 2) + 4 * ((wid >> 1) & 1) + (s & 3)) * SB_ + (64 * (wid & 1) + 16 * cp + 8 * hf) * 2;
#pragma unroll
    for (int i = 0; i < 4; ++i) { vA[i] = voffA + i * 64 * SA_; vB[i] = voffB + (i & 1) * 32 * SB_ + (i >> 1) * 256; asm volatile("" : "+v"(vA[i]), "+v"(vB[i])); } }
#define SA(b, h) (((b) * 2 + (h)) * HTB)
#define SB(b, h) ((4 + (b) * 2 + (h)) * HTB)
#define GLDS(voff, sbase, c) asm volatile("s_add_i32 m0, %2, %3\n\ts_nop 0\n\tglobal_load_lds_dwordx4 %0, %1 sc0 sc1" :: "v"(voff), "s"(sbase), "s"(ldsw), "i"(c) : "memory", "scc")
#define STAGEA(b, h, gbase) do { GLDS(vA[2 * (h)], gbase, SA(b, h)); GLDS(vA[2 * (h) + 1], gbase, SA(b, h) + 8192); } while (0)
#define STAGEB(b, h, gbase) do { GLDS(vB[2 * (h)], gbase, SB(b, h)); GLDS(vB[2 * (h) + 1], gbase, SB(b, h) + 8192); } while (0)
  const char* const ownA = p.xin + (size_t)((tm >> 1) * 1024 + ra * 512 + (tm & 1) * 256) * SA_;
  const char* const ownB = p.win + (size_t)(rb * 4096 + tn * 256) * 2;
  const char* a1 = ownA + 128;

  unsigned call, par;
  {
    const unsigned loX = (lane >> 3) * SA_ + (lane & 7) * 16, loW = (lane >> 5) * SB_ + (lane & 31) * 16;
    const int pd = (xcd & 3) * 2 + (xcd >> 2), pq = ((xcd & 3) ^ rcol) - 1;
    const bool push = idx >= 8 && idx < 8 + NPUSH && pq >= 0;
    const bool more = (more_mask(0) >> (idx >> 2)) & 1;
    int blk[4]; bool val[4];
#define PK_BLOCK(k, g) do { val[k] = (g) < 4 * NENT; const int _g = val[k] ? (g) : (xcd >> 1); \
      const int _e = val[k] ? (RL(tab, _g >> 4) >> ((_g >> 2) & 3) * 8) & 255 : ((idx - 8) & 15) << 3, _op = _e >> 7, _tau = (_e >> 3) & 15, _hv = _g & 1; \
      blk[k] = _op * 512 + (((_g >> 1) & 1) * 8 + (_e & 7)) * 32 + (_op ? _hv * 16 + _tau : (_tau >> 1) * 4 + _hv * 2 + (_tau & 1)); } while (0)
    PK_BLOCK(0, push || shift == 1 ? 1024 : pq < 0 ? (idx < 4 ? idx : idx - 4) * 2 + (xcd >> 2) : 56 + (idx < 4 ? idx : idx - 4 - NPUSH) * 6 + pq * 2 + (xcd >> 2));
    u32x4 r[4][4]; int hl = 8 * HTB;
#define PK_LOAD(k, on) do { const int _op = blk[k] >> 9, _jj = (blk[k] >> 5) & 15, _ix = blk[k] & 31; \
      const unsigned _so = _op ? (_jj * 64 + wid * 8) * SB_ + _ix * 512 : (_ix * 256 + wid * 32) * SA_ + _jj * 128, _cs = _op ? 2 * SB_ : 8 * SA_; \
      _Pragma("unroll") for (int c = 0; c < 4; ++c) r[k][c] = ld16p(mk(_op ? p.win : p.xin, on ? 16u << 20 : 0u), _op ? loW : loX, _so + c * _cs); } while (0)
#define PK_PACK(k) do { const unsigned _h = pack_unit(r[k], base_of(r[k]), lane, scr, rpi, blk[k] * 24576 + wid * 3072, rpc, blk[k] * 16384 + wid * 2048); \
      if (lane == 0) *(LAS unsigned*)(lds + hl + (k) * 32 + wid * 4) = _h; } while (0)
#define PK_SEND(k) do { const int _op = blk[k] >> 9, _jj = (blk[k] >> 5) & 15, _ix = blk[k] & 31, _d = (lane >> 3) & 1; \
      const int _tau = _op ? (_ix & 15) : ((_ix >> 2) * 2 + (_ix & 1)); \
      const int _c0 = _op ? (_ix >> 4) : (_ix & 2), _c1 = _op ? _c0 + 2 : _c0 + 1; \
      char* const _w0 = p.ws[(_c0 * 2 + (_jj >> 3)) & 7]; char* const _w1 = p.ws[(_c1 * 2 + (_jj >> 3)) & 7]; \
      if (lane < 16 && (_d ? _c1 : _c0) != rcol) { const u32x2 _h = {call, *(const LAS unsigned*)(lds + hl + (k) * 32 + (lane & 7) * 4)}; \
        char* _dst = (_d ? _w1 : _w0) + OFF_HIN + (size_t)((par * 8 + rank) * 256 + _op * 128 + _tau * 8 + (_jj & 7)) * 64 + (lane & 7) * 8; \
        asm volatile("global_store_dwordx2 %0, %1, off sc0 sc1\n\ts_nop 1" :: "v"(_dst), "v"(_h) : "memory"); } } while (0)
#define PROLOGUE() do { STAGEB(0, 0, ownB); STAGEA(0, 0, ownA); STAGEB(0, 1, ownB); STAGEA(0, 1, ownA); \
      STAGEB(1, 0, ownB + 64 * SB_); STAGEA(1, 0, a1); STAGEB(1, 1, ownB + 64 * SB_); } while (0)
    PK_LOAD(0, shift != 1);
    asm volatile("" : "+v"(callv) :: "memory");
    call = RFL(callv) + 1; par = call & 1;
    const rsrc_t rpi = mk(ws + OFF_PIN + (size_t)par * 24 * MiB, 24u << 20), rpc = mk(ws + OFF_PINC + (size_t)par * 16 * MiB, 16u << 20);
    asm volatile("" : "+v"(tab), "+v"(r[0][0]), "+v"(r[0][1]), "+v"(r[0][2]), "+v"(r[0][3]));
    if (push) {
      const rsrc_t rq = mk(p.ws[pd] + OFF_STG + (size_t)par * 48 * MiB, 24u << 20);
#pragma unroll
      for (int c = 0; c < 4; ++c) st16(r[0][c], rq, loX, (pq * 4096 + (idx - 8) * 256 + wid * 32 + c * 8) * SA_ + rc * 1024);
      FENCE;
    } else if (shift == 1) BAR;
    else {
      PK_PACK(0);
      if (more) WAIT_V(0); else { PROLOGUE(); WAIT_V(14); }
      WAIT_L(0); BAR; FENCE;
      if (wid == 0) PK_SEND(0);
      if (!more) { if (wid == 0) WAIT_V(1); else WAIT_V(0); BAR; }
    }
    STAMP(1);
    if (more) {
      for (int q = 0; q < 8; q += 4) {
        const bool last = q == 4 || !((more_mask(4) >> (idx >> 2)) & 1);
        hl += 128;
#pragma unroll
        for (int k = 0; k < 4; ++k) { PK_BLOCK(k, ORDER.pk[idx >> 2][q + k] + (bid & 31)); PK_LOAD(k, val[k]); }
        FENCE;
        if (val[0]) { PK_PACK(0); if (val[1]) { PK_PACK(1); if (val[2]) { PK_PACK(2); if (val[3]) PK_PACK(3); } } }
        if (last) PROLOGUE();
        DRAIN; WAIT_L(0); BAR; FENCE;
#pragma unroll
        for (int k = 0; k < 4; ++k) if (wid == k + 1 && val[k]) PK_SEND(k);
        if (q == 0 && tid == 320 && push) astore(p.ws[pd] + OFF_RDY + ((idx - 8) * 64 + pq * 2 + rc) * 4, 8 * call);
        if (last) break;
      }
    }
    if (tid == 384) astore(ws + OFF_CALL + bid * 128, call);
    if (shift != 1 && wr == 1) BAR;
  }
  STAMP(2);

  int aoff0 = (wr * 64 + fr) * 128 + 16 * (fq ^ (fr & 7)), aoff1 = aoff0 ^ 64, boff0, boff1;
  { const int s = (lane >> 2) & 7, pp = ((lane & 1) << 1) | ((lane >> 1) & 1), gh = lane >> 5;
    boff0 = 4 * HTB + gh * 4096 + (wc >> 1) * 1024 + 128 * s + 32 * ((2 * (wc & 1)) ^ (s >> 1)) + 8 * pp; boff1 = boff0 ^ 32; }
  asm volatile("" : "+v"(aoff0), "+v"(aoff1), "+v"(boff0), "+v"(boff1));
#define LDA(dst, b, h) do { _Pragma("unroll") for (int m = 0; m < 4; ++m) { dst[m][0] = *(const LAS bf16x8*)(lds + SA(b, h) + aoff0 + m * 2048); dst[m][1] = *(const LAS bf16x8*)(lds + SA(b, h) + aoff1 + m * 2048); } } while (0)
#define TR(off) __builtin_amdgcn_ds_read_tr16_b64_v4i16((LAS s16x4*)(lds + (off)))
#define LDB(dst, b, h) do { _Pragma("unroll") for (int k = 0; k < 2; ++k) { \
    dst[0][k] = __builtin_shufflevector(TR(SA(b, h) + boff0 + k * 8192), TR(SA(b, h) + boff0 + k * 8192 + 2048), 0, 1, 2, 3, 4, 5, 6, 7); \
    dst[1][k] = __builtin_shufflevector(TR(SA(b, h) + boff1 + k * 8192), TR(SA(b, h) + boff1 + k * 8192 + 2048), 0, 1, 2, 3, 4, 5, 6, 7); } } while (0)
#define MMA(ai, bj, At, Bt) do { __builtin_amdgcn_s_setprio(1); _Pragma("unroll") for (int m = 0; m < 4; ++m) _Pragma("unroll") for (int n = 0; n < 2; ++n) _Pragma("unroll") for (int k = 0; k < 2; ++k) \
    acc[ai][bj][m][n] = __builtin_amdgcn_mfma_f32_16x16x32_bf16(Bt[n][k], At[m][k], acc[ai][bj][m][n], 0, 0, 0); __builtin_amdgcn_s_setprio(0); } while (0)
  f32x4 acc[2][2][4][2];
#pragma unroll
  for (int a = 0; a < 2; ++a)
#pragma unroll
    for (int b = 0; b < 2; ++b)
#pragma unroll
      for (int m = 0; m < 4; ++m)
#pragma unroll
        for (int n = 0; n < 2; ++n) acc[a][b][m][n] = (f32x4){0.f, 0.f, 0.f, 0.f};
  bf16x8 At[4][2], B0[2][2], B1[2][2];
  const char* const sA = ws + OFF_STG + (size_t)par * 48 * MiB + (size_t)tm * 256 * SA_;
  const char* const sB = ws + OFF_STG + 24 * MiB + (size_t)tn * 512;
  const int pause = shift == 1 ? 0 : (shift == 2 ? PAUSE2 : -1);
  const int s0 = tm < 6 ? 0 : tm < 11 ? 2 : 1;
  int j = 0, s2 = s0, ready = 0;
  for (int it = 0; it < 32; ++it) {
    if (it == pause) {
      if (wr == 0 && shift == 2) BAR;
      STAMP(3);
      int ln = tid; asm volatile("" : "+v"(ln)); ln &= 63;
      const unsigned gw = wid * 32 + (((idx & 3) * 8 + xcd + wid) & 31), s = gw % 6, widx = gw / 6, cnt = 2 * PW;
      if (widx < PW) {
      const unsigned lb = shift == 1 ? 0 : SPLIT * 8, le = shift == 1 ? SPLIT * 8 : NENT * 8;
      const unsigned qb = (call - 1) * (le - lb - cnt) - lb - 2 * cnt;
      char* const ctr = ws + OFF_Q + (s * 2 + shift - 1) * 128; char* const rdy = ws + OFF_RDY;
      const unsigned src = ((((unsigned)rcol ^ (1 + (s >> 1))) << 1) | (s & 1)) & 7;
      const char* const sw = p.ws[src];
      const unsigned lx = (ln >> 3) * SA_ + (ln & 7) * 16, lw = (ln >> 5) * SB_ + (ln & 31) * 16;
      const rsrc_t rsc = mk(sw + OFF_PINC + (size_t)par * 16 * MiB, 16u << 20), rh = mk(ws + OFF_HIN + (size_t)(par * 8 + src) * 16384, 16384);
      zero_scr(scr, ln);
      { const unsigned xm = ((~tab >> 7) & (((tab & 0x07070707u) + 0x07070707u) >> 3) & 0x01010101u) * 0x78u;
        tab = (tab & ~xm) | (((tab & xm) + (s >> 1) * 0x58585858u % 0x80808080u) & xm); }
#define ENT(L) ((((RL(tab, ((L) >> 5) & 63) >> (((L) >> 3) & 3) * 8) & 255u) << 3) | ((L) & 7))
#define IN_SRC(E) (((E) >> 10) ? (512 + (rc * 8 + (((E) >> 3) & 7)) * 32 + rb * 16 + (((E) >> 6) & 15)) * 8 + ((E) & 7) \
                               : ((rc * 8 + (((E) >> 3) & 7)) * 32 + ((E) >> 7) * 4 + ra * 2 + (((E) >> 6) & 1)) * 8 + ((E) & 7))
#define IN_RDY(E) ((((E) >> 6) * 64 + (((E) >> 3) & 7) * 6 + s) * 4)
      struct Chain { Unit d; u32x2 hv; unsigned cv, Ep, Ec, En, Ln, hw; bool live, pend; } A, B;
      A.Ec = ENT(lb + widx); B.Ec = ENT(lb + PW + widx); A.Ln = lb + cnt + widx; B.Ln = A.Ln + PW; A.En = ENT(A.Ln); B.En = ENT(B.Ln);
      A.live = B.live = true; A.pend = B.pend = false; A.Ep = B.Ep = 0;
      A.hv = ld8(rh, 0, A.Ec * 8); B.hv = ld8(rh, 0, B.Ec * 8);
      if (shift == 2) { const u64 t0 = __builtin_amdgcn_s_memrealtime(); while (RFL(aload(ws + OFF_HAND + gw * 4)) != call && !give_up(ws, call, t0)) { } }
#define IN_ISSUE(v, on) do { CLAIM(v.cv, ctr, on); v.hv = ld8(rh, 0, v.En * 8); \
        load_unit(v.d, mk(sw + OFF_PIN + (size_t)par * 24 * MiB, v.live ? 24u << 20 : 0u), IN_SRC(v.Ec) * 3072, v.hw, ln); FENCE; } while (0)
      wait_hdr(A.hv, rh, 0, A.Ec * 8, ws, call); A.hw = RFL(A.hv[1]); asm volatile("" : "+v"(B.hv)); STAMP(8); IN_ISSUE(A, 1);
      wait_hdr(B.hv, rh, 0, B.Ec * 8, ws, call); B.hw = RFL(B.hv[1]); IN_ISSUE(B, 1);
#define IN_STEP(v, o) do { __builtin_amdgcn_s_waitcnt(0x0F70 | (NLD + 1)); asm volatile("s_waitcnt vmcnt(%1)" : "+v"(v.cv) : "i"(NLD + 1) : "memory"); \
        if (o.pend) SIGNAL(rdy, IN_RDY(o.Ep)); \
        unsigned T[8]; unpack_hi(T, v.d, v.hw, ln, scr, rsc, IN_SRC(v.Ec) * 2048); \
        const unsigned _op = v.Ec >> 10, _tau = (v.Ec >> 6) & 15, _j = (v.Ec >> 3) & 7, _u = v.Ec & 7, _cs = _op ? 2 * SB_ : 8 * SA_, _lo = _op ? lw : lx; \
        const unsigned _db = _op ? 24 * (unsigned)MiB + ((s >> 1) * 512 + _j * 64 + _u * 8) * SB_ + (s & 1) * 8192 + _tau * 512 \
                                 : par * 48 * (unsigned)MiB + ((s >> 1) * 4096 + _tau * 256 + _u * 32) * SA_ + (s & 1) * 1024 + _j * 128; \
        const rsrc_t _rd = mk(ws + OFF_STG, v.live ? 72u << 20 : 0u); \
        _Pragma("unroll") for (int c = 0; c < 4; ++c) st16(unpack_row(T, v.d, c, v.hw), _rd, _lo, _db + c * _cs); \
        FENCE; \
        v.pend = v.live; v.Ep = v.Ec; v.live = v.Ln < le; \
        if (v.live) wait_hdr(v.hv, rh, 0, v.En * 8, ws, call); \
        v.hw = v.live ? RFL(v.hv[1]) : 0u; v.Ec = v.En; v.Ln = v.live ? RFL(v.cv) - qb : ~0u; v.En = ENT(v.Ln); \
        if (v.live && v.Ln >= le && shift == 1 && ln == 0) astore(ws + OFF_HAND + gw * 4, call); \
        IN_ISSUE(v, v.Ln < le); } while (0)
      do { IN_STEP(A, B); IN_STEP(B, A); } while (A.live || B.live);
      DRAIN;
      if (B.pend) SIGNAL(rdy, IN_RDY(B.Ep));
      }
      if (shift == 1) { PROLOGUE(); WAIT_V(0); }
      STAMP(4);
      if (shift == 1) BAR;
      if (wr == 1) BAR;
    }
    if (it == 7) STAMP(12); if (it == 19) STAMP(13); if (it == 25) STAMP(14); if (it == 28) STAMP(15);
    const char *a2, *b2, *a3, *b3;
    if (it < 7 || it == 31) { const int t = (it == 31) ? 0 : 2 * it + 2; a2 = ownA + t * 128; b2 = ownB + (size_t)t * 64 * SB_; a3 = a2 + 128; b3 = b2 + 64 * SB_; }
    else {
      if (ready < 2 * it - 12) {
        int ln = tid; asm volatile("" : "+v"(ln)); ln &= 63;
        const int lq = ln % 6 + 2 * s0, lr = ln + 2 * s0 - (lq >= 6 ? 6 : 0);
        const u64 t0 = __builtin_amdgcn_s_memrealtime();
        for (;;) {
          const unsigned fx = aload(ws + OFF_RDY + (tm * 64 + lr) * 4), fw = aload(ws + OFF_RDY + ((16 + tn) * 64 + lr) * 4);
          ready = __builtin_ctzll(~__builtin_amdgcn_ballot_w64((int)(fx - 8 * call) >= 0 && fw == 8 * call && ln < 48));
          if (ready >= 2 * it - 12) break;
          if (give_up(ws, call, t0)) { ready = 64; break; }
        }
        FENCE;
      }
      a2 = sA + (size_t)s2 * (4096 * SA_) + j * 128; b2 = sB + (size_t)(s2 * 512 + j * 64) * SB_; a3 = a2 + 1024; b3 = b2 + 8192; if (++s2 == 3) s2 = 0;
      if (s2 == s0) ++j;
    }
    LDB(B0, 0, 0); SCHED; LDA(At, 0, 0); STAGEA(1, 1, a1);
    WAIT_L(8); BAR; WAIT_L(0); MMA(0, 0, At, B0); BAR; SCHED;
    LDB(B1, 0, 1); STAGEB(0, 0, b2);
    BAR; WAIT_L(0); MMA(0, 1, At, B1); BAR;
    LDA(At, 0, 1); STAGEA(0, 0, a2);
    BAR; WAIT_L(0); MMA(1, 0, At, B0); BAR; SCHED;
    STAGEB(0, 1, b2);
    WAIT_V(6); BAR; MMA(1, 1, At, B1); BAR;
    LDB(B0, 1, 0); SCHED; LDA(At, 1, 0); STAGEA(0, 1, a2);
    WAIT_L(8); BAR; WAIT_L(0); MMA(0, 0, At, B0); BAR; SCHED;
    LDB(B1, 1, 1); STAGEB(1, 0, b3);
    BAR; WAIT_L(0); MMA(0, 1, At, B1); BAR;
    LDA(At, 1, 1); STAGEA(1, 0, a3);
    BAR; WAIT_L(0); MMA(1, 0, At, B0); BAR; SCHED;
    STAGEB(1, 1, b3);
    WAIT_V(6); BAR; MMA(1, 1, At, B1); BAR;
    a1 = a3;
  }
  if (wr == 0) BAR;
  STAMP(5);

  int ln = tid; asm volatile("" : "+v"(ln)); ln &= 63;
  {
    const unsigned tmo = aload(ws + OFF_TMO);
    const unsigned slot = (tm * 16 + tn) * 8 + wid;
    const rsrc_t rpo = mk(ws + OFF_POUT + (size_t)par * 24 * MiB + (size_t)slot * 12288, 12288), rpc = mk(ws + OFF_POUTC + (size_t)par * 16 * MiB + (size_t)slot * 8192, 8192);
    char* const dst = p.ws[(tm >> 1) & 7] + OFF_HOUT + (size_t)(((par * 8 + rank) * 32 + (tm & 1) * 16 + tn) * 8 + wid) * 32 + (ln & 3) * 8;
    u32x2 hs = {call, 0u}; unsigned base = 0;
#pragma unroll
    for (int u = 0; u < 4; ++u) {
      u32x4 r[4];
#pragma unroll
      for (int c = 0; c < 4; ++c) {
        const f32x4 x = acc[u >> 1][c & 1][(u & 1) * 2 + (c >> 1)][0], y = acc[u >> 1][c & 1][(u & 1) * 2 + (c >> 1)][1];
        const auto s0 = __builtin_amdgcn_permlane32_swap(cvt2(x[0], x[1]), cvt2(y[0], y[1]), false, false);
        const auto s1 = __builtin_amdgcn_permlane32_swap(cvt2(x[2], x[3]), cvt2(y[2], y[3]), false, false);
        r[c] = (u32x4){s0[0], s1[0], s0[1], s1[1]};
      }
      if (u == 0) base = base_of(r);
      const unsigned h = pack_unit(r, base, ln, scr, rpo, u * 3072, rpc, u * 2048);
      if (ln == u) hs[1] |= h;
      if (u == 2 && tmo == call) hs[1] |= 0x10000u;
      if (EPI2 && u == 2) {
        asm volatile("s_waitcnt vmcnt(%0)" :: "i"(NLD) : "memory");
        if (ln < 2) asm volatile("global_store_dwordx2 %0, %1, off sc0 sc1\n\ts_nop 1" :: "v"(dst), "v"(hs) : "memory"); }
    }
    WAIT_V(0);
    if (ln < 4 && ln >= 2 * EPI2) asm volatile("global_store_dwordx2 %0, %1, off sc0 sc1\n\ts_nop 1" :: "v"(dst), "v"(hs) : "memory");
  }
  STAMP(6);

  if (idx >= 32 - NRED && wid < RW) {
    const unsigned gw = ((31 - idx) * 8 + xcd) * RW + wid, col = gw & 3, widx = gw >> 2, cnt = NRED * 2 * RW;
    const unsigned qb = (call - 1) * (1024 - cnt) - 2 * cnt;
    char* const ctr = ws + OFF_Q + (12 + col) * 128;
    const char* const w0 = p.ws[col * 2]; const char* const w1 = p.ws[col * 2 + 1];
    const size_t mine = (size_t)rank * 1024;
    const rsrc_t r0 = mk(w0 + OFF_POUT + (size_t)par * 24 * MiB + mine * 3072, 3u << 20), r1 = mk(w1 + OFF_POUT + (size_t)par * 24 * MiB + mine * 3072, 3u << 20);
    const rsrc_t c0 = mk(w0 + OFF_POUTC + (size_t)par * 16 * MiB + mine * 2048, 2u << 20), c1 = mk(w1 + OFF_POUTC + (size_t)par * 16 * MiB + mine * 2048, 2u << 20);
    const rsrc_t rh = mk(ws + OFF_HOUT + (size_t)(par * 8 + col * 2) * 8192, 16384), ro = mk(p.out, 16u << 20);
    const unsigned lo = (ln & 15) * SB_ + (ln >> 4) * 16, ho = (ln >> 5) * 8192;
#define OENT(L) (((((L) >> 4) & 3) << 8) | ((7 - (((L) >> 7) & 7)) << 5) | ((((L) >> 1) & 7) << 2) | ((((L) >> 6) & 1) << 1) | ((L) & 1))
    u32x2 hv; Unit da[2], db[2]; unsigned cv, Ec = OENT(widx), En, Ln = cnt + widx, h0, h1;
    zero_scr(scr, ln);
    if (col != (unsigned)rcol) {
      const unsigned sg = ((col ^ rcol) - 1) * 2 + (ln & 1), cg = 2 * PW, need = call * (NENT * 8 - SPLIT * 8 - cg) - cg - GATE;
      const u64 t0 = __builtin_amdgcn_s_memrealtime();
      while (__builtin_amdgcn_ballot_w64((int)(aload(ws + OFF_Q + (sg * 2 + 1) * 128) - need) < 0) && !give_up(ws, call, t0)) { }
    }
    STAMP(11);
    hv = ld8(rh, ho, Ec * 8); bool good = wait_hdr(hv, rh, ho, Ec * 8, ws, call);
    h0 = RL(hv[1], 0); h1 = RL(hv[1], 32); good = good && !((h0 | h1) >> 16); h0 &= 0xFFFFu; h1 &= 0xFFFFu;
    load_unit(da[0], r0, Ec * 3072, h0, ln); load_unit(da[1], r1, Ec * 3072, h1, ln);
    En = OENT(Ln); hv = ld8(rh, ho, En * 8); CLAIM(cv, ctr, 1);
    STAMP(10);
#define OUT_STEP(d, dn) { asm volatile("s_waitcnt vmcnt(0)" : "+v"(d[0].a0), "+v"(d[0].a1), "+v"(d[0].p), "+v"(d[0].pt), "+v"(d[1].a0), "+v"(d[1].a1), "+v"(d[1].p), "+v"(d[1].pt), "+v"(hv), "+v"(cv) :: "memory"); \
      unsigned n0 = 0, n1 = 0, Eq = 0, Lq = ~0u; \
      if (Ln < 1024) { good = wait_hdr(hv, rh, ho, En * 8, ws, call) && good; n0 = RL(hv[1], 0); n1 = RL(hv[1], 32); \
        good = good && !((n0 | n1) >> 16); n0 &= 0xFFFFu; n1 &= 0xFFFFu; \
        load_unit(dn[0], r0, En * 3072, n0, ln); load_unit(dn[1], r1, En * 3072, n1, ln); \
        Lq = RFL(cv) - qb; Eq = OENT(Lq); hv = ld8(rh, ho, Eq * 8); CLAIM(cv, ctr, Lq < 1024); } \
      FENCE; \
      unsigned T0[8], T1[8]; \
      unpack_hi(T0, d[0], h0, ln, scr, c0, Ec * 2048); unpack_hi(T1, d[1], h1, ln, scr, c1, Ec * 2048); \
      const unsigned _ob = ((col >> 1) * 512 + (Ec >> 9) * 256 + ((Ec >> 1) & 1) * 128 + ((Ec >> 4) & 1) * 64 + (Ec & 1) * 32) * SB_ + ((col & 1) * 4096 + ((Ec >> 5) & 15) * 256 + ((Ec >> 2) & 3) * 32) * 2; \
      _Pragma("unroll") for (int c = 0; c < 4; ++c) { const u32x4 x = unpack_row(T0, d[0], c, h0), y = unpack_row(T1, d[1], c, h1); u32x4 o; \
        _Pragma("unroll") for (int e = 0; e < 4; ++e) { \
          const float v0 = __builtin_bit_cast(float, x[e] << 16) + __builtin_bit_cast(float, y[e] << 16), v1 = __builtin_bit_cast(float, x[e] & 0xFFFF0000u) + __builtin_bit_cast(float, y[e] & 0xFFFF0000u); \
          o[e] = __builtin_bit_cast(unsigned, __builtin_elementwise_max(__builtin_bit_cast(s16x2, cvt2(v0, v1)), (s16x2){0, 0})); } \
        if (!good) o = (u32x4){0x7FC07FC0u, 0x7FC07FC0u, 0x7FC07FC0u, 0x7FC07FC0u}; \
        __builtin_amdgcn_raw_buffer_store_b128(o, ro, lo + _ob + (c >> 1) * 16 * SB_ + (c & 1) * 256, 0, 16); } \
      FENCE; \
      if (Ln >= 1024) break; \
      Ec = En; En = Eq; Ln = Lq; h0 = n0; h1 = n1; }
    for (;;) { OUT_STEP(da, db) OUT_STEP(db, da) }
  }
  STAMP(7);
}

extern "C" void kernel_launch(int rank, int nranks, ncclComm_t comm,
                              hipStream_t stream,
                              void* const* d_in, const int* in_sizes, int n_in,
                              void* d_out, int out_size,
                              const int* in_dtypes, int out_dtype,
                              void* const* d_out_all,
                              void* const* d_ws_all, size_t ws_bytes) {
  (void)comm; (void)in_dtypes; (void)out_dtype; (void)d_out_all;
  if (nranks != 8 || (unsigned)rank >= 8u || n_in != 2 || ws_bytes < WS_NEED || in_sizes[0] != 8192 * 1024 || in_sizes[1] != 8192 * 1024 || out_size != 1024 * 8192) {
    fprintf(stderr, "gemm_rs: unexpected problem shape\n"); return; }
  HIP_CHECK(hipFuncSetAttribute((const void*)gemm_rs, hipFuncAttributeMaxDynamicSharedMemorySize, LDS_BYTES));
  Params p;
  p.xin = (const char*)d_in[0]; p.win = (const char*)d_in[1]; p.out = (char*)d_out; p.own = (char*)d_ws_all[rank]; p.rank = rank; p.pad = 0;
  for (int i = 0; i < 8; ++i) p.ws[i] = (char*)d_ws_all[i];
  gemm_rs<<<NWG, 512, LDS_BYTES, stream>>>(p);
  HIP_CHECK(hipGetLastError());
}
```

```cpp
#include <hip/hip_runtime.h>
#include <rccl/rccl.h>
#include <cstdio>

#define HIP_CHECK(expr) do { hipError_t _e = (expr); if (_e != hipSuccess) { \
    fprintf(stderr, "HIP error %s at %s:%d\n", hipGetErrorString(_e), __FILE__, __LINE__); \
    return; } } while (0)

#define LAS __attribute__((address_space(3)))
#define GAS __attribute__((address_space(1)))
typedef short bf16x8 __attribute__((ext_vector_type(8)));
typedef short s16x4 __attribute__((ext_vector_type(4)));
typedef short s16x2 __attribute__((ext_vector_type(2)));
typedef unsigned short u16x2 __attribute__((ext_vector_type(2)));
typedef __bf16 bf16x2 __attribute__((ext_vector_type(2)));
typedef float f32x2 __attribute__((ext_vector_type(2)));
typedef float f32x4 __attribute__((ext_vector_type(4)));
typedef unsigned u32x2 __attribute__((ext_vector_type(2)));
typedef unsigned u32x3 __attribute__((ext_vector_type(3)));
typedef unsigned u32x4 __attribute__((ext_vector_type(4)));
typedef unsigned long long u64;
typedef __amdgpu_buffer_rsrc_t rsrc_t;

#define PW 30
#define NRED 4
#define RW 4
#define NPUSH 6
#define SPLIT1 76
#define SPLIT2 152
#define PAUSE2 6
#define PAUSE3 17
#define GATE 32
#define DELAYS 2, 14, 12, 10, 8, 5, 3, 0
#define PACKS 5, 5, 0, 8, 7, 2, 0, 0
#define SKIP0 4
#define GBITS "sc0 sc1"
#define B96 0
#define EPIP 1
#define EPI2 1
constexpr int NENT = 256 - NPUSH, NP0 = 28 - SKIP0;
struct alignas(16) Order { unsigned char e[256]; short pk[8][8]; };
constexpr Order make_order() {
  Order o{}; const int dl[8] = {DELAYS}; int n = 0;
  for (int key = 0; key < 96; ++key)
    for (int op = 1; op >= 0; --op)
      for (int k = 0; k < 16; ++k)
        for (int j = 0; j < 8; ++j) {
          const int tau = op ? (((k & 1) << 3) | (7 - (k >> 1))) : k;
          int kk = 3 * j + (op ? dl[tau & 7] : k < 6 ? 0 : k < 11 ? 1 : 2);
          if (op && (tau & 7) == 0 && 3 * j >= PAUSE3 - 7) kk = 80 + j;
          if (op && (tau & 7) == 1 && 3 * j < PAUSE2 - 7) kk = 3 * j;
          if (kk == key && (op || j || k >= NPUSH)) o.e[n++] = (unsigned char)(op << 7 | tau << 3 | j);
        }
  const int pc[8] = {PACKS}; n = 8 * NP0 - 6 * NPUSH;
  for (int k = 0; k < 8; ++k)
    for (int g = 7; g >= 0; --g) { o.pk[g][k] = (short)(pc[g] > k ? n : 1024); n += pc[g] > k ? 32 : 0; }
  return o;
}
constexpr Order ORDER_H = make_order();
constexpr bool order_ok() {
  for (int i = SPLIT1; i < NENT; ++i) { const int e = ORDER_H.e[i], op = e >> 7, tau = (e >> 3) & 15, j = e & 7;
    if ((!op || (tau & 7) == 1) && 3 * j < PAUSE2 - 7) return false;
    if (i >= SPLIT2 && (!op || (tau & 7) == 0) && 3 * j < PAUSE3 - 7) return false; }
  const int pc[8] = {PACKS}; int n = 8 * NP0 - 6 * NPUSH;
  for (int g = 0; g < 8; ++g) n += 32 * pc[g];
  return n >= 4 * NENT && pc[2] == 0 && pc[0] && pc[1] && (SKIP0 == 0 || pc[7] == 0) && (SKIP0 <= 4 || pc[6] == 0) && SKIP0 <= 8;
}
constexpr unsigned more_mask(int k) { unsigned m = 0; for (int g = 0; g < 8; ++g) m |= (ORDER_H.pk[g][k] < 4 * NENT ? 1u : 0u) << g; return m; }
static_assert(order_ok(), "a shift would wait for blocks it has to pull itself, or blocks stay unpacked");
static_assert(NRED >= 2 && NRED <= 8 && PW <= 42 && NPUSH >= 0 && NPUSH <= 8 && SPLIT1 * 2 >= PW && (SPLIT2 - SPLIT1) * 2 >= PW && (NENT - SPLIT2) * 2 >= PW, "roles");
__constant__ Order ORDER = ORDER_H;

constexpr int NWG = 256, HTB = 16384;
constexpr unsigned SA_ = 2048, SB_ = 16384;
constexpr size_t MiB = 1ull << 20;
constexpr size_t OFF_CALL = 0;
constexpr size_t OFF_TMO = 32768, OFF_Q = 33024, OFF_HAND = 36864;
constexpr size_t OFF_RDY = 40960;
constexpr size_t OFF_HIN = 65536;
constexpr size_t OFF_HOUT = OFF_HIN + 262144;
constexpr size_t OFF_STG = 1 * MiB;
constexpr size_t OFF_PIN = 73 * MiB;
constexpr size_t OFF_PINC = 121 * MiB;
constexpr size_t OFF_POUT = 153 * MiB;
constexpr size_t OFF_POUTC = 201 * MiB;
constexpr size_t OFF_DBG = 233 * MiB;
constexpr size_t WS_NEED = 234 * MiB;
constexpr int SCR = 2448, LDS_BYTES = 8 * HTB + 1024 + 8 * SCR;
constexpr unsigned OOB = 0x7FFFFF00u;
constexpr u64 DEADLINE = 100000000ull;

struct Params { const char* xin; const char* win; char* out; char* own; char* ws[8]; int rank; int pad; };

#define RFL(x) __builtin_amdgcn_readfirstlane(x)
#define WAIT_V(n) asm volatile("s_waitcnt vmcnt(" #n ")" ::: "memory")
#define WAIT_L(n) asm volatile("s_waitcnt lgkmcnt(" #n ")" ::: "memory")
#define DRAIN do { asm volatile("" ::: "memory"); __builtin_amdgcn_s_waitcnt(0x0F70); asm volatile("" ::: "memory"); } while (0)
#define BAR __builtin_amdgcn_s_barrier()
#define SCHED __builtin_amdgcn_sched_barrier(0)
#define FENCE asm volatile("" ::: "memory")
#define DI __device__ __forceinline__
constexpr int SYS = 17;

DI rsrc_t mk(const void* p, unsigned bytes) { return __builtin_amdgcn_make_buffer_rsrc((void*)p, (short)0, (int)bytes, 0x00020000); }
DI u32x4 ld16(rsrc_t r, unsigned v, unsigned s) { return __builtin_amdgcn_raw_buffer_load_b128(r, v, s, SYS); }
DI u32x4 ld16p(rsrc_t r, unsigned v, unsigned s) { return __builtin_amdgcn_raw_buffer_load_b128(r, v, s, 0); }
DI void st16(u32x4 x, rsrc_t r, unsigned v, unsigned s) { __builtin_amdgcn_raw_buffer_store_b128(x, r, v + s, 0, SYS); }
DI unsigned aload(const char* p) { return __hip_atomic_load((const GAS unsigned*)p, __ATOMIC_RELAXED, __HIP_MEMORY_SCOPE_SYSTEM); }
DI void astore(char* p, unsigned v) { __hip_atomic_store((GAS unsigned*)p, v, __ATOMIC_RELAXED, __HIP_MEMORY_SCOPE_SYSTEM); }
DI unsigned perm(unsigned hi, unsigned lo, unsigned sel) { return __builtin_amdgcn_perm(hi, lo, sel); }
DI unsigned cvt2(float lo, float hi) { return __builtin_bit_cast(unsigned, __builtin_convertvector((f32x2){lo, hi}, bf16x2)); }

#define STAMP(i) do { } while (0)

DI bool give_up(char* ws, unsigned call, u64 t0) {
  const u64 dt = __builtin_amdgcn_s_memrealtime() - t0;
  if (dt > 10000) {
    if (RFL(aload(ws + OFF_TMO)) == call) return true;
    if (dt > DEADLINE) { astore(ws + OFF_TMO, call); return true; }
  }
  __builtin_amdgcn_s_sleep(2);
  return false;
}

struct Unit { u32x4 a0, a1; u32x3 p; unsigned pt; };
DI unsigned bfi(unsigned m, unsigned a, unsigned b) { unsigned r; asm("v_bfi_b32 %0, %1, %2, %3" : "=v"(r) : "s"(m), "v"(a), "v"(b)); return r; }
DI unsigned pka(unsigned a, unsigned b) { return __builtin_bit_cast(unsigned, __builtin_bit_cast(u16x2, a) + __builtin_bit_cast(u16x2, b)); }
DI unsigned pks(unsigned a, unsigned b) { return __builtin_bit_cast(unsigned, __builtin_bit_cast(u16x2, a) - __builtin_bit_cast(u16x2, b)); }
DI unsigned pick_base(unsigned f, int th) {
  unsigned g = RFL(f);
  while (__builtin_popcountll(__builtin_amdgcn_ballot_w64(f > g)) >= th) ++g;
  while (g > 7 && __builtin_popcountll(__builtin_amdgcn_ballot_w64(f >= g)) < th) --g;
  return g > 7 ? g - 7 : 0;
}
DI unsigned base_of(const u32x4 (&r)[4]) {
  u16x2 mx = {0, 0};
#pragma unroll
  for (int c = 0; c < 4; ++c)
#pragma unroll
    for (int i = 0; i < 4; i += 2) mx = __builtin_elementwise_max(mx, __builtin_bit_cast(u16x2, r[c][i] & 0x7FFF7FFFu));
  return pick_base((mx.x > mx.y ? mx.x : mx.y) >> 7, 9);
}
DI unsigned pack_unit(const u32x4 (&r)[4], unsigned base, int lane, LAS unsigned char* scr, rsrc_t dst, unsigned doff, rsrc_t dstc, unsigned coff) {
  const unsigned B2 = base * 0x00800080u, M = 0x83838383u;
  unsigned A[8], T[8], mask = 0, n = 0;
#pragma unroll
  for (int k = 0; k < 8; ++k) {
    const unsigned lo = pks(r[k >> 1][(k & 1) * 2], B2), hi = pks(r[k >> 1][(k & 1) * 2 + 1], B2);
    A[k] = perm(hi, lo, 0x06040200u); T[k] = perm(hi, lo, 0x07050301u);
    mask = bfi(0x80808080u, (T[k] & 0x7C7C7C7Cu) + 0x7C7C7C7Cu, mask >> 1);
  }
  *(LAS u32x4*)(scr + lane * 32) = (u32x4){T[0], T[1], T[2], T[3]}; *(LAS u32x4*)(scr + lane * 32 + 16) = (u32x4){T[4], T[5], T[6], T[7]};
  st16((u32x4){A[0], A[1], A[2], A[3]}, dst, lane * 16, doff);
  st16((u32x4){A[4], A[5], A[6], A[7]}, dst, lane * 16 + 1024, doff);
  const unsigned P0 = bfi(0x60606060u, T[6] << 5, bfi(M, T[0], __builtin_rotateleft32(T[1], 3)));
  const unsigned P1 = bfi(0x60606060u, T[7] << 5, bfi(M, T[2], __builtin_rotateleft32(T[3], 3)));
  const unsigned P2 = bfi(0x40404040u, T[6] >> 1, bfi(0x20202020u, T[7] >> 2, bfi(M, T[4], __builtin_rotateleft32(T[5], 3))));
  __builtin_amdgcn_raw_buffer_store_b64((u32x2){P0, P1}, dst, lane * 8 + 2048 + doff, 0, SYS);
  __builtin_amdgcn_raw_buffer_store_b32(P2, dst, lane * 4 + 2560 + doff, 0, SYS);
  FENCE;
  for (u64 m; n <= 128 && (m = __builtin_amdgcn_ballot_w64(mask != 0)) != 0; n += __builtin_popcountll(m)) {
    const unsigned i = __builtin_amdgcn_mbcnt_hi((unsigned)(m >> 32), __builtin_amdgcn_mbcnt_lo((unsigned)m, n));
    const unsigned b = __builtin_ctz(mask | 0x80000000u), a = lane * 32 + ((b & 7) << 2 | b >> 3);
    *(LAS unsigned short*)(scr + 2048 + (mask ? i : 192u) * 2) = (unsigned short)((scr[a] & 0x7Cu) << 9 | a);
    mask &= mask - 1;
  }
  FENCE;
  if (n > 128) {
    n = 255;
    st16((u32x4){T[0], T[1], T[2], T[3]} & 0x7C7C7C7Cu, dstc, lane * 16, coff);
    st16((u32x4){T[4], T[5], T[6], T[7]} & 0x7C7C7C7Cu, dstc, lane * 16 + 1024, coff);
  } else __builtin_amdgcn_raw_buffer_store_b32(*(const LAS unsigned*)(scr + 2048 + lane * 4), dst, 2u * lane < n ? lane * 4 + 2816 + doff : OOB, 0, SYS);
  FENCE;
  return base | n << 8;
}
constexpr int NLD = B96 ? 4 : 5;
DI void load_unit(Unit& d, rsrc_t src, unsigned off, unsigned hw, int lane) {
  d.pt = __builtin_amdgcn_raw_buffer_load_b32(src, 2u * lane < (hw >> 8) ? lane * 4 + 2816 : OOB, off, SYS);
  const u32x2 q = __builtin_amdgcn_raw_buffer_load_b64(src, lane * 8 + 2048, off, SYS);
  d.p = (u32x3){q[0], q[1], __builtin_amdgcn_raw_buffer_load_b32(src, lane * 4 + 2560, off, SYS)};
  d.a0 = ld16(src, lane * 16, off); d.a1 = ld16(src, lane * 16 + 1024, off);
}
DI void unpack_hi(unsigned (&T)[8], const Unit& d, unsigned hw, int lane, LAS unsigned char* scr, rsrc_t srcc, unsigned coff) {
  const unsigned n = hw >> 8, M = 0x83838383u;
  u32x4 x0, x1;
  if (n > 128) { x0 = ld16(srcc, lane * 16, coff); x1 = ld16(srcc, lane * 16 + 1024, coff); DRAIN; }
  else {
    const unsigned a0 = d.pt & 0x7FFu, a1 = (d.pt >> 16) & 0x7FFu;
    if (2u * lane < n) scr[a0] = (unsigned char)((d.pt >> 9) & 0x7Cu);
    if (2u * lane + 1 < n) scr[a1] = (unsigned char)((d.pt >> 25) & 0x7Cu);
    FENCE;
    x0 = *(const LAS u32x4*)(scr + lane * 32); x1 = *(const LAS u32x4*)(scr + lane * 32 + 16);
    FENCE;
    if (2u * lane < n) { scr[a0] = 0; scr[a1] = 0; }
    FENCE;
  }
  T[0] = (d.p[0] & M) | x0[0]; T[1] = (__builtin_rotateright32(d.p[0], 3) & M) | x0[1];
  T[2] = (d.p[1] & M) | x0[2]; T[3] = (__builtin_rotateright32(d.p[1], 3) & M) | x0[3];
  T[4] = (d.p[2] & M) | x1[0]; T[5] = (__builtin_rotateright32(d.p[2], 3) & M) | x1[1];
  T[6] = (bfi(0x03030303u, d.p[0] >> 5, d.p[2] << 1) & M) | x1[2];
  T[7] = (bfi(0x03030303u, d.p[1] >> 5, d.p[2] << 2) & M) | x1[3];
}
DI u32x4 unpack_row(const unsigned (&T)[8], const Unit& d, int c, unsigned hw) {
  const unsigned B2 = (hw & 255u) * 0x00800080u, a = (c & 2 ? d.a1 : d.a0)[(c & 1) * 2], b = (c & 2 ? d.a1 : d.a0)[(c & 1) * 2 + 1];
  return (u32x4){pka(perm(T[2 * c], a, 0x05010400u), B2), pka(perm(T[2 * c], a, 0x07030602u), B2), pka(perm(T[2 * c + 1], b, 0x05010400u), B2), pka(perm(T[2 * c + 1], b, 0x07030602u), B2)};
}
DI void zero_scr(LAS unsigned char* scr, int lane) { *(LAS u32x4*)(scr + lane * 32) = (u32x4){0u, 0u, 0u, 0u}; *(LAS u32x4*)(scr + lane * 32 + 16) = (u32x4){0u, 0u, 0u, 0u}; FENCE; }
DI u32x2 ld8(rsrc_t r, unsigned v, unsigned s) { return __builtin_amdgcn_raw_buffer_load_b64(r, v, s, SYS); }
DI bool wait_hdr(u32x2& h, rsrc_t rh, unsigned voff, unsigned soff, char* ws, unsigned call) {
  if (!__builtin_amdgcn_ballot_w64(h[0] != call)) return true;
  const u64 t0 = __builtin_amdgcn_s_memrealtime();
  for (;;) {
    h = ld8(rh, voff, soff); FENCE;
    if (!__builtin_amdgcn_ballot_w64(h[0] != call)) return true;
    if (give_up(ws, call, t0)) { h = (u32x2){0u, 0u}; return false; }
  }
}
#define CLAIM(cv, base, on) do { cv = 1u; asm volatile("s_mov_b32 exec_lo, %3\n\ts_mov_b32 exec_hi, 0\n\ts_nop 2\n\tglobal_atomic_add %0, %1, %0, %2 sc0\n\ts_mov_b64 exec, -1" : "+v"(cv) : "v"(0u), "s"(base), "s"(RFL((on) ? 1u : 0u)) : "memory"); } while (0)
#define SIGNAL(base, off) asm volatile("s_mov_b64 exec, 1\n\ts_nop 3\n\tglobal_atomic_add %0, %1, %2\n\ts_mov_b64 exec, -1" :: "v"(off), "v"(1u), "s"(base) : "memory")
#define RL(x, l) ((unsigned)__builtin_amdgcn_readlane((int)(x), (l)))

__global__ __launch_bounds__(512, 2) void gemm_rs(Params p) {
  extern __shared__ __attribute__((aligned(16))) unsigned char smem[];
  LAS unsigned char* lds = (LAS unsigned char*)smem;
  const int tid = threadIdx.x, wid = RFL(tid >> 6), lane = tid & 63, wr = wid >> 2, wc = wid & 3, fr = lane & 15, fq = lane >> 4;
  unsigned tab = ((const unsigned*)ORDER.e)[lane];
  asm volatile("" :: "s"(p.xin), "s"(p.win), "s"(p.own), "s"(p.rank), "s"(p.ws[(blockIdx.x & 3) * 2 + ((blockIdx.x >> 2) & 1)]) : "memory");
  const int rank = p.rank & 7, ra = rank >> 2, rb = (rank >> 1) & 1, rc = rank & 1, rcol = rank >> 1;
  char* const ws = p.own;
  const int bid = blockIdx.x, xcd = bid & 7, idx = bid >> 3;
  const int tm = (xcd >> 1) * 4 + (idx & 3), tn = (xcd & 1) * 8 + (idx >> 2);
  const int shift = idx < 12 ? 3 - (idx >> 2) : 0;
  LAS unsigned char* const scr = lds + 8 * HTB + 1024 + wid * SCR;
  STAMP(0);
  unsigned callv = aload(ws + OFF_CALL + bid * 128);
  volatile LAS int* const seen = (volatile LAS int*)(lds + 8 * HTB + 896);
  if (tid == 0) *seen = 0;

  const unsigned ldsw = (unsigned)(size_t)lds + (unsigned)wid * 1024u;
  unsigned vA[4], vB[4];
  { const unsigned voffA = (wid * 8 + (lane >> 3)) * SA_ + 16 * ((lane & 7) ^ (lane >> 3));
    const int s = lane >> 3, pos = (lane & 7) >> 1, hf = lane & 1, cp = pos ^ (s >> 1);
    const unsigned voffB = (16 * (wid >> 2) + 8 * (s >> 2) + 4 * ((wid >> 1) & 1) + (s & 3)) * SB_ + (64 * (wid & 1) + 16 * cp + 8 * hf) * 2;
#pragma unroll
    for (int i = 0; i < 4; ++i) { vA[i] = voffA + i * 64 * SA_; vB[i] = voffB + (i & 1) * 32 * SB_ + (i >> 1) * 256; asm volatile("" : "+v"(vA[i]), "+v"(vB[i])); } }
#define SA(b, h) (((b) * 2 + (h)) * HTB)
#define SB(b, h) ((4 + (b) * 2 + (h)) * HTB)
#define GLDS(voff, sbase, c) asm volatile("s_add_i32 m0, %2, %3\n\ts_nop 0\n\tglobal_load_lds_dwordx4 %0, %1 " GBITS :: "v"(voff), "s"(sbase), "s"(ldsw), "i"(c) : "memory", "scc")
#define STAGEA(b, h, gbase) do { GLDS(vA[2 * (h)], gbase, SA(b, h)); GLDS(vA[2 * (h) + 1], gbase, SA(b, h) + 8192); } while (0)
#define STAGEB(b, h, gbase) do { GLDS(vB[2 * (h)], gbase, SB(b, h)); GLDS(vB[2 * (h) + 1], gbase, SB(b, h) + 8192); } while (0)
  const char* const ownA = p.xin + (size_t)((tm >> 1) * 1024 + ra * 512 + (tm & 1) * 256) * SA_;
  const char* const ownB = p.win + (size_t)(rb * 4096 + tn * 256) * 2;
  const char* a1 = ownA + 128;

  unsigned call, par;
  {
    const unsigned loX = (lane >> 3) * SA_ + (lane & 7) * 16, loW = (lane >> 5) * SB_ + (lane & 31) * 16;
    const int pd = (xcd & 3) * 2 + (xcd >> 2), pq = ((xcd & 3) ^ rcol) - 1;
    const bool push = idx < NPUSH && pq >= 0;
    const bool more = (more_mask(0) >> (idx >> 2)) & 1;
    int blk[4]; bool val[4];
#define PK_BLOCK(k, g) do { val[k] = (g) < 4 * NENT; const int _g = val[k] ? (g) : (xcd >> 1); \
      const int _e = val[k] ? (RL(tab, _g >> 4) >> ((_g >> 2) & 3) * 8) & 255 : (idx & 15) << 3, _op = _e >> 7, _tau = (_e >> 3) & 15, _hv = _g & 1; \
      blk[k] = _op * 512 + (((_g >> 1) & 1) * 8 + (_e & 7)) * 32 + (_op ? _hv * 16 + _tau : (_tau >> 1) * 4 + _hv * 2 + (_tau & 1)); } while (0)
    const bool none = shift == 1 || idx >= 32 - SKIP0;
    PK_BLOCK(0, push || none ? 1024 : pq < 0 ? (idx < 8 ? idx : idx - 4) * 2 + (xcd >> 2) : 2 * NP0 + ((idx < 8 ? idx : idx - 4) - NPUSH) * 6 + pq * 2 + (xcd >> 2));
    u32x4 r[4][4]; int hl = 8 * HTB;
#define PK_LOAD(k, on) do { const int _op = blk[k] >> 9, _jj = (blk[k] >> 5) & 15, _ix = blk[k] & 31; \
      const unsigned _so = _op ? (_jj * 64 + wid * 8) * SB_ + _ix * 512 : (_ix * 256 + wid * 32) * SA_ + _jj * 128, _cs = _op ? 2 * SB_ : 8 * SA_; \
      _Pragma("unroll") for (int c = 0; c < 4; ++c) r[k][c] = ld16p(mk(_op ? p.win : p.xin, on ? 16u << 20 : 0u), _op ? loW : loX, _so + c * _cs); } while (0)
#define PK_PACK(k) do { const unsigned _h = pack_unit(r[k], base_of(r[k]), lane, scr, rpi, blk[k] * 24576 + wid * 3072, rpc, blk[k] * 16384 + wid * 2048); \
      if (lane == 0) *(LAS unsigned*)(lds + hl + (k) * 32 + wid * 4) = _h; } while (0)
#define PK_SEND(k) do { const int _op = blk[k] >> 9, _jj = (blk[k] >> 5) & 15, _ix = blk[k] & 31, _d = (lane >> 3) & 1; \
      const int _tau = _op ? (_ix & 15) : ((_ix >> 2) * 2 + (_ix & 1)); \
      const int _c0 = _op ? (_ix >> 4) : (_ix & 2), _c1 = _op ? _c0 + 2 : _c0 + 1; \
      char* const _w0 = p.ws[(_c0 * 2 + (_jj >> 3)) & 7]; char* const _w1 = p.ws[(_c1 * 2 + (_jj >> 3)) & 7]; \
      if (lane < 16 && (_d ? _c1 : _c0) != rcol) { const u32x2 _h = {call, *(const LAS unsigned*)(lds + hl + (k) * 32 + (lane & 7) * 4)}; \
        char* _dst = (_d ? _w1 : _w0) + OFF_HIN + (size_t)((par * 8 + rank) * 256 + _op * 128 + _tau * 8 + (_jj & 7)) * 64 + (lane & 7) * 8; \
        asm volatile("global_store_dwordx2 %0, %1, off sc0 sc1\n\ts_nop 1" :: "v"(_dst), "v"(_h) : "memory"); } } while (0)
#define PROLOGUE() do { STAGEB(0, 0, ownB); STAGEA(0, 0, ownA); STAGEB(0, 1, ownB); STAGEA(0, 1, ownA); \
      STAGEB(1, 0, ownB + 64 * SB_); STAGEA(1, 0, a1); STAGEB(1, 1, ownB + 64 * SB_); } while (0)
    if (none && shift != 1) PROLOGUE();
    PK_LOAD(0, !none);
    asm volatile("" : "+v"(callv) :: "memory");
    call = RFL(callv) + 1; par = call & 1;
    const rsrc_t rpi = mk(ws + OFF_PIN + (size_t)par * 24 * MiB, 24u << 20), rpc = mk(ws + OFF_PINC + (size_t)par * 16 * MiB, 16u << 20);
    asm volatile("" : "+v"(tab), "+v"(r[0][0]), "+v"(r[0][1]), "+v"(r[0][2]), "+v"(r[0][3]));
    if (push) {
      const rsrc_t rq = mk(p.ws[pd] + OFF_STG + (size_t)par * 48 * MiB, 24u << 20);
#pragma unroll
      for (int c = 0; c < 4; ++c) st16(r[0][c], rq, loX, (pq * 4096 + idx * 256 + wid * 32 + c * 8) * SA_ + rc * 1024);
      FENCE;
    } else if (none) { WAIT_V(0); WAIT_L(0); BAR; }
    else {
      PK_PACK(0);
      if (more) WAIT_V(0); else { PROLOGUE(); WAIT_V(14); }
      WAIT_L(0); BAR; FENCE;
      if (wid == 0) PK_SEND(0);
      if (!more) { if (wid == 0) WAIT_V(1); else WAIT_V(0); BAR; }
    }
    STAMP(1);
    if (more) {
      for (int q = 0; q < 8; q += 4) {
        const bool last = q == 4 || !((more_mask(4) >> (idx >> 2)) & 1);
        hl += 128;
#pragma unroll
        for (int k = 0; k < 4; ++k) { PK_BLOCK(k, ORDER.pk[idx >> 2][q + k] + (bid & 31)); PK_LOAD(k, val[k]); }
        FENCE;
        if (val[0]) { PK_PACK(0); if (val[1]) { PK_PACK(1); if (val[2]) { PK_PACK(2); if (val[3]) PK_PACK(3); } } }
        if (last) PROLOGUE();
        DRAIN; WAIT_L(0); BAR; FENCE;
#pragma unroll
        for (int k = 0; k < 4; ++k) if (wid == k + 1 && val[k]) PK_SEND(k);
        if (q == 0 && tid == 320 && push) astore(p.ws[pd] + OFF_RDY + (idx * 64 + pq * 2 + rc) * 4, 8 * call);
        if (last) break;
      }
    }
    if (tid == 384) astore(ws + OFF_CALL + bid * 128, call);
    if (shift != 1 && wr == 1) BAR;
  }
  STAMP(2);

  int aoff0 = (wr * 64 + fr) * 128 + 16 * (fq ^ (fr & 7)), aoff1 = aoff0 ^ 64, boff0, boff1;
  { const int s = (lane >> 2) & 7, pp = ((lane & 1) << 1) | ((lane >> 1) & 1), gh = lane >> 5;
    boff0 = 4 * HTB + gh * 4096 + (wc >> 1) * 1024 + 128 * s + 32 * ((2 * (wc & 1)) ^ (s >> 1)) + 8 * pp; boff1 = boff0 ^ 32; }
  asm volatile("" : "+v"(aoff0), "+v"(aoff1), "+v"(boff0), "+v"(boff1));
#define LDA(dst, b, h) do { _Pragma("unroll") for (int m = 0; m < 4; ++m) { dst[m][0] = *(const LAS bf16x8*)(lds + SA(b, h) + aoff0 + m * 2048); dst[m][1] = *(const LAS bf16x8*)(lds + SA(b, h) + aoff1 + m * 2048); } } while (0)
#define TR(off) __builtin_amdgcn_ds_read_tr16_b64_v4i16((LAS s16x4*)(lds + (off)))
#define LDB(dst, b, h) do { _Pragma("unroll") for (int k = 0; k < 2; ++k) { \
    dst[0][k] = __builtin_shufflevector(TR(SA(b, h) + boff0 + k * 8192), TR(SA(b, h) + boff0 + k * 8192 + 2048), 0, 1, 2, 3, 4, 5, 6, 7); \
    dst[1][k] = __builtin_shufflevector(TR(SA(b, h) + boff1 + k * 8192), TR(SA(b, h) + boff1 + k * 8192 + 2048), 0, 1, 2, 3, 4, 5, 6, 7); } } while (0)
#define MMA(ai, bj, At, Bt) do { __builtin_amdgcn_s_setprio(1); _Pragma("unroll") for (int m = 0; m < 4; ++m) _Pragma("unroll") for (int n = 0; n < 2; ++n) _Pragma("unroll") for (int k = 0; k < 2; ++k) \
    acc[ai][bj][m][n] = __builtin_amdgcn_mfma_f32_16x16x32_bf16(Bt[n][k], At[m][k], acc[ai][bj][m][n], 0, 0, 0); __builtin_amdgcn_s_setprio(0); } while (0)
  f32x4 acc[2][2][4][2];
#pragma unroll
  for (int a = 0; a < 2; ++a)
#pragma unroll
    for (int b = 0; b < 2; ++b)
#pragma unroll
      for (int m = 0; m < 4; ++m)
#pragma unroll
        for (int n = 0; n < 2; ++n) acc[a][b][m][n] = (f32x4){0.f, 0.f, 0.f, 0.f};
  bf16x8 At[4][2], B0[2][2], B1[2][2];
  const char* const sA = ws + OFF_STG + (size_t)par * 48 * MiB + (size_t)tm * 256 * SA_;
  const char* const sB = ws + OFF_STG + 24 * MiB + (size_t)tn * 512;
  const int pause = shift == 1 ? 0 : shift == 2 ? PAUSE2 : shift == 3 ? PAUSE3 : -1;
  const int s0 = tm < 6 ? 0 : tm < 11 ? 2 : 1;
  int j = 0, s2 = s0, ready = 0;
  unsigned fo = (lane + 2 * s0 - (lane % 6 + 2 * s0 >= 6 ? 6 : 0)) * 4, fx = 0, fw = 0;
  const char* const rX = ws + OFF_RDY + tm * 256; const char* const rW = ws + OFF_RDY + (16 + tn) * 256;
  asm volatile("" : "+v"(fo), "+v"(fx), "+v"(fw));
#define READY() (int)__builtin_ctzll(~(__builtin_amdgcn_ballot_w64((int)(fx - 8 * call) >= 0 && fw == 8 * call) & 0xFFFFFFFFFFFFull))
  for (int it = 0; it < 32; ++it) {
    if (it == pause) {
      if (wr == 0 && shift != 1) BAR;
      STAMP(3);
      int ln = tid; asm volatile("" : "+v"(ln)); ln &= 63;
      const unsigned gw = wid * 32 + (((idx & 3) * 8 + xcd + wid) & 31), s = gw % 6, widx = gw / 6;
      if (widx < PW) {
      const unsigned lb = shift == 1 ? 0 : shift == 2 ? SPLIT1 * 8 : SPLIT2 * 8, le = shift == 1 ? SPLIT1 * 8 : shift == 2 ? SPLIT2 * 8 : NENT * 8;
      const unsigned qb = (call - 1) * (le - lb - 2 * PW) - lb - 3 * PW;
      char* const ctr = ws + OFF_Q + (s * 3 + shift - 1) * 128; char* const rdy = ws + OFF_RDY;
      const unsigned src = ((((unsigned)rcol ^ (1 + (s >> 1))) << 1) | (s & 1)) & 7;
      const char* const sw = p.ws[src];
      const unsigned lx = (ln >> 3) * SA_ + (ln & 7) * 16, lw = (ln >> 5) * SB_ + (ln & 31) * 16;
      const rsrc_t rsc = mk(sw + OFF_PINC + (size_t)par * 16 * MiB, 16u << 20), rh = mk(ws + OFF_HIN + (size_t)(par * 8 + src) * 16384, 16384);
      zero_scr(scr, ln);
      { const unsigned xm = ((~tab >> 7) & (((tab & 0x07070707u) + 0x07070707u) >> 3) & 0x01010101u) * 0x78u;
        tab = (tab & ~xm) | (((tab & xm) + (s >> 1) * 0x58585858u % 0x80808080u) & xm); }
#define ENT(L) ((((RL(tab, ((L) >> 5) & 63) >> (((L) >> 3) & 3) * 8) & 255u) << 3) | ((L) & 7))
#define IN_SRC(E) (((E) >> 10) ? (512 + (rc * 8 + (((E) >> 3) & 7)) * 32 + rb * 16 + (((E) >> 6) & 15)) * 8 + ((E) & 7) \
                               : ((rc * 8 + (((E) >> 3) & 7)) * 32 + ((E) >> 7) * 4 + ra * 2 + (((E) >> 6) & 1)) * 8 + ((E) & 7))
#define IN_RDY(E) ((((E) >> 6) * 64 + (((E) >> 3) & 7) * 6 + s) * 4)
      struct Buf { Unit d; unsigned Ep, Ec, hw; bool live, pend; } A, B;
      unsigned cv, Ln = lb + 2 * PW + widx, En = ENT(Ln);
      A.Ec = ENT(lb + widx); B.Ec = ENT(lb + PW + widx); A.live = B.live = true; A.pend = B.pend = false; A.Ep = B.Ep = 0;
      u32x2 hv = ld8(rh, 0, A.Ec * 8), hb = ld8(rh, 0, B.Ec * 8);
      if (shift != 1) { const u64 t0 = __builtin_amdgcn_s_memrealtime(); while (RFL(aload(ws + OFF_HAND + (shift - 2) * 1024 + gw * 4)) != call && !give_up(ws, call, t0)) { } }
#define IN_LOAD(v) do { load_unit(v.d, mk(sw + OFF_PIN + (size_t)par * 24 * MiB, v.live ? 24u << 20 : 0u), IN_SRC(v.Ec) * 3072, v.hw, ln); FENCE; } while (0)
      wait_hdr(hv, rh, 0, A.Ec * 8, ws, call); A.hw = RFL(hv[1]); asm volatile("" : "+v"(hb)); STAMP(8); IN_LOAD(A);
      wait_hdr(hb, rh, 0, B.Ec * 8, ws, call); B.hw = RFL(hb[1]); CLAIM(cv, ctr, 1); hv = ld8(rh, 0, En * 8); FENCE; IN_LOAD(B);
#define IN_STEP(v, o) do { __builtin_amdgcn_s_waitcnt(0x0F70 | NLD); asm volatile("s_waitcnt vmcnt(%1)" : "+v"(cv) : "i"(NLD) : "memory"); \
        if (o.pend) SIGNAL(rdy, IN_RDY(o.Ep)); \
        unsigned T[8]; unpack_hi(T, v.d, v.hw, ln, scr, rsc, IN_SRC(v.Ec) * 2048); \
        const unsigned _op = v.Ec >> 10, _tau = (v.Ec >> 6) & 15, _j = (v.Ec >> 3) & 7, _u = v.Ec & 7, _cs = _op ? 2 * SB_ : 8 * SA_, _lo = _op ? lw : lx; \
        const unsigned _db = _op ? 24 * (unsigned)MiB + ((s >> 1) * 512 + _j * 64 + _u * 8) * SB_ + (s & 1) * 8192 + _tau * 512 \
                                 : par * 48 * (unsigned)MiB + ((s >> 1) * 4096 + _tau * 256 + _u * 32) * SA_ + (s & 1) * 1024 + _j * 128; \
        const rsrc_t _rd = mk(ws + OFF_STG, v.live ? 72u << 20 : 0u); \
        _Pragma("unroll") for (int c = 0; c < 4; ++c) st16(unpack_row(T, v.d, c, v.hw), _rd, _lo, _db + c * _cs); \
        FENCE; \
        v.pend = v.live; v.Ep = v.Ec; v.live = Ln < le; \
        if (v.live) wait_hdr(hv, rh, 0, En * 8, ws, call); \
        v.hw = v.live ? RFL(hv[1]) : 0u; v.Ec = En; Ln = v.live ? RFL(cv) - qb : ~0u; En = ENT(Ln); \
        if (v.live && Ln >= le && shift != 3 && ln == 0) astore(ws + OFF_HAND + (shift - 1) * 1024 + gw * 4, call); \
        CLAIM(cv, ctr, Ln < le); hv = ld8(rh, 0, En * 8); FENCE; IN_LOAD(v); } while (0)
      do { IN_STEP(A, B); IN_STEP(B, A); } while (A.live);
      DRAIN;
      if (B.pend) SIGNAL(rdy, IN_RDY(B.Ep));
      }
      if (shift == 1) { PROLOGUE(); WAIT_V(0); }
      STAMP(4);
      if (shift == 1) BAR;
      if (wr == 1) BAR;
    }
    if (it == 7) STAMP(12); if (it == 19) STAMP(13); if (it == 25) STAMP(14); if (it == 28) STAMP(15);
    const char *a2, *b2, *a3, *b3;
    if (it < 7 || it == 31) { const int t = (it == 31) ? 0 : 2 * it + 2; a2 = ownA + t * 128; b2 = ownB + (size_t)t * 64 * SB_; a3 = a2 + 128; b3 = b2 + 64 * SB_; }
    else {
      if (ready < 2 * it - 12) {
        ready = READY();
        if (ready < 2 * it - 12) {
          const u64 t0 = __builtin_amdgcn_s_memrealtime();
          for (;;) {
            if (wid == 0) { fx = aload(rX + fo); fw = aload(rW + fo); ready = READY(); } else ready = RFL(*seen);
            if (ready >= 2 * it - 12) break;
            if (give_up(ws, call, t0)) { ready = 64; break; }
          }
        }
        if (wid == 0) *seen = ready;
        FENCE;
      }
      a2 = sA + (size_t)s2 * (4096 * SA_) + j * 128; b2 = sB + (size_t)(s2 * 512 + j * 64) * SB_; a3 = a2 + 1024; b3 = b2 + 8192; if (++s2 == 3) s2 = 0;
      if (s2 == s0) ++j;
    }
    LDB(B0, 0, 0); SCHED; LDA(At, 0, 0); STAGEA(1, 1, a1);
    WAIT_L(8); BAR; WAIT_L(0); MMA(0, 0, At, B0); BAR; SCHED;
    LDB(B1, 0, 1); STAGEB(0, 0, b2);
    BAR; WAIT_L(0); MMA(0, 1, At, B1); BAR;
    LDA(At, 0, 1); STAGEA(0, 0, a2);
    BAR; WAIT_L(0); MMA(1, 0, At, B0); BAR; SCHED;
    STAGEB(0, 1, b2);
    WAIT_V(6);
    if (ready < 2 * it - 10 && it < 30) asm volatile("s_nop 4\n\tglobal_load_dword %0, %2, %3 sc0 sc1\n\tglobal_load_dword %1, %2, %4 sc0 sc1" : "+v"(fx), "+v"(fw) : "v"(fo), "s"(rX), "s"(rW) : "memory");
    BAR; MMA(1, 1, At, B1); BAR;
    LDB(B0, 1, 0); SCHED; LDA(At, 1, 0); STAGEA(0, 1, a2);
    WAIT_L(8); BAR; WAIT_L(0); MMA(0, 0, At, B0); BAR; SCHED;
    LDB(B1, 1, 1); STAGEB(1, 0, b3);
    BAR; WAIT_L(0); MMA(0, 1, At, B1); BAR;
    LDA(At, 1, 1); STAGEA(1, 0, a3);
    BAR; WAIT_L(0); MMA(1, 0, At, B0); BAR; SCHED;
    STAGEB(1, 1, b3);
    asm volatile("s_waitcnt vmcnt(6)" : "+v"(fx), "+v"(fw) :: "memory");
    BAR; MMA(1, 1, At, B1); BAR;
    a1 = a3;
  }
  if (wr == 0) BAR;
  STAMP(5);

  int ln = tid; asm volatile("" : "+v"(ln)); ln &= 63;
  {
    const unsigned tmo = aload(ws + OFF_TMO);
    const unsigned slot = (tm * 16 + tn) * 8 + wid;
    const rsrc_t rpo = mk(ws + OFF_POUT + (size_t)par * 24 * MiB + (size_t)slot * 12288, 12288), rpc = mk(ws + OFF_POUTC + (size_t)par * 16 * MiB + (size_t)slot * 8192, 8192);
    char* const dst = p.ws[(tm >> 1) & 7] + OFF_HOUT + (size_t)(((par * 8 + rank) * 32 + (tm & 1) * 16 + tn) * 8 + wid) * 32 + (ln & 3) * 8;
    u32x2 hs = {call, 0u}; unsigned base = 0;
    if (EPIP && wr == 0) __builtin_amdgcn_s_setprio(2);
#pragma unroll
    for (int u = 0; u < 4; ++u) {
      u32x4 r[4];
#pragma unroll
      for (int c = 0; c < 4; ++c) {
        const f32x4 x = acc[u >> 1][c & 1][(u & 1) * 2 + (c >> 1)][0], y = acc[u >> 1][c & 1][(u & 1) * 2 + (c >> 1)][1];
        const auto s0 = __builtin_amdgcn_permlane32_swap(cvt2(x[0], x[1]), cvt2(y[0], y[1]), false, false);
        const auto s1 = __builtin_amdgcn_permlane32_swap(cvt2(x[2], x[3]), cvt2(y[2], y[3]), false, false);
        r[c] = (u32x4){s0[0], s1[0], s0[1], s1[1]};
      }
      if (u == 0) base = base_of(r);
      const unsigned h = pack_unit(r, base, ln, scr, rpo, u * 3072, rpc, u * 2048);
      if (ln == u) hs[1] |= h;
      if (u == 2 && tmo == call) hs[1] |= 0x10000u;
      if (EPI2 && u == 2) {
        asm volatile("s_waitcnt vmcnt(%0)" :: "i"(NLD) : "memory");
        if (ln < 2) asm volatile("global_store_dwordx2 %0, %1, off sc0 sc1\n\ts_nop 1" :: "v"(dst), "v"(hs) : "memory"); }
    }
    WAIT_V(0);
    if (ln < 4 && ln >= 2 * EPI2) asm volatile("global_store_dwordx2 %0, %1, off sc0 sc1\n\ts_nop 1" :: "v"(dst), "v"(hs) : "memory");
  }
  STAMP(6);

  if (idx >= 32 - NRED && wid < RW) {
    const unsigned gw = ((31 - idx) * 8 + xcd) * RW + wid, col = gw & 3, widx = gw >> 2, cnt = NRED * 2 * RW;
    const unsigned qb = (call - 1) * (1024 - cnt) - 2 * cnt;
    char* const ctr = ws + OFF_Q + (18 + col) * 128;
    const char* const w0 = p.ws[col * 2]; const char* const w1 = p.ws[col * 2 + 1];
    const size_t mine = (size_t)rank * 1024;
    const rsrc_t r0 = mk(w0 + OFF_POUT + (size_t)par * 24 * MiB + mine * 3072, 3u << 20), r1 = mk(w1 + OFF_POUT + (size_t)par * 24 * MiB + mine * 3072, 3u << 20);
    const rsrc_t c0 = mk(w0 + OFF_POUTC + (size_t)par * 16 * MiB + mine * 2048, 2u << 20), c1 = mk(w1 + OFF_POUTC + (size_t)par * 16 * MiB + mine * 2048, 2u << 20);
    const rsrc_t rh = mk(ws + OFF_HOUT + (size_t)(par * 8 + col * 2) * 8192, 16384), ro = mk(p.out, 16u << 20);
    const unsigned lo = (ln & 15) * SB_ + (ln >> 4) * 16, ho = (ln >> 5) * 8192;
#define OENT(L) (((((L) >> 3) & 3) << 8) | ((7 - (((L) >> 7) & 7)) << 5) | ((((L) >> 6) & 1) << 4) | ((((L) >> 1) & 3) << 2) | ((((L) >> 5) & 1) << 1) | ((L) & 1))
    u32x2 hv; Unit da[2], db[2]; unsigned cv, Ec = OENT(widx), En, Ln = cnt + widx, h0, h1;
    zero_scr(scr, ln);
    if (col != (unsigned)rcol) {
      const unsigned sg = ((col ^ rcol) - 1) * 2 + (ln & 1), need = call * (NENT * 8 - SPLIT2 * 8 - 2 * PW) - PW - GATE;
      const u64 t0 = __builtin_amdgcn_s_memrealtime();
      while (__builtin_amdgcn_ballot_w64((int)(aload(ws + OFF_Q + (sg * 3 + 2) * 128) - need) < 0) && !give_up(ws, call, t0)) { }
    }
    STAMP(11);
    hv = ld8(rh, ho, Ec * 8); bool good = wait_hdr(hv, rh, ho, Ec * 8, ws, call);
    h0 = RL(hv[1], 0); h1 = RL(hv[1], 32); good = good && !((h0 | h1) >> 16); h0 &= 0xFFFFu; h1 &= 0xFFFFu;
    load_unit(da[0], r0, Ec * 3072, h0, ln); load_unit(da[1], r1, Ec * 3072, h1, ln);
    En = OENT(Ln); hv = ld8(rh, ho, En * 8); CLAIM(cv, ctr, 1);
    STAMP(10);
#define OUT_STEP(d, dn) { asm volatile("s_waitcnt vmcnt(0)" : "+v"(d[0].a0), "+v"(d[0].a1), "+v"(d[0].p), "+v"(d[0].pt), "+v"(d[1].a0), "+v"(d[1].a1), "+v"(d[1].p), "+v"(d[1].pt), "+v"(hv), "+v"(cv) :: "memory"); \
      unsigned n0 = 0, n1 = 0, Eq = 0, Lq = ~0u; \
      if (Ln < 1024) { good = wait_hdr(hv, rh, ho, En * 8, ws, call) && good; n0 = RL(hv[1], 0); n1 = RL(hv[1], 32); \
        good = good && !((n0 | n1) >> 16); n0 &= 0xFFFFu; n1 &= 0xFFFFu; \
        load_unit(dn[0], r0, En * 3072, n0, ln); load_unit(dn[1], r1, En * 3072, n1, ln); \
        Lq = RFL(cv) - qb; Eq = OENT(Lq); hv = ld8(rh, ho, Eq * 8); CLAIM(cv, ctr, Lq < 1024); } \
      FENCE; \
      unsigned T0[8], T1[8]; \
      unpack_hi(T0, d[0], h0, ln, scr, c0, Ec * 2048); unpack_hi(T1, d[1], h1, ln, scr, c1, Ec * 2048); \
      const unsigned _ob = ((col >> 1) * 512 + (Ec >> 9) * 256 + ((Ec >> 1) & 1) * 128 + ((Ec >> 4) & 1) * 64 + (Ec & 1) * 32) * SB_ + ((col & 1) * 4096 + ((Ec >> 5) & 15) * 256 + ((Ec >> 2) & 3) * 32) * 2; \
      _Pragma("unroll") for (int c = 0; c < 4; ++c) { const u32x4 x = unpack_row(T0, d[0], c, h0), y = unpack_row(T1, d[1], c, h1); u32x4 o; \
        _Pragma("unroll") for (int e = 0; e < 4; ++e) { \
          const float v0 = __builtin_bit_cast(float, x[e] << 16) + __builtin_bit_cast(float, y[e] << 16), v1 = __builtin_bit_cast(float, x[e] & 0xFFFF0000u) + __builtin_bit_cast(float, y[e] & 0xFFFF0000u); \
          o[e] = __builtin_bit_cast(unsigned, __builtin_elementwise_max(__builtin_bit_cast(s16x2, cvt2(v0, v1)), (s16x2){0, 0})); } \
        if (!good) o = (u32x4){0x7FC07FC0u, 0x7FC07FC0u, 0x7FC07FC0u, 0x7FC07FC0u}; \
        __builtin_amdgcn_raw_buffer_store_b128(o, ro, lo + _ob + (c >> 1) * 16 * SB_ + (c & 1) * 256, 0, 16); } \
      FENCE; \
      if (Ln >= 1024) break; \
      Ec = En; En = Eq; Ln = Lq; h0 = n0; h1 = n1; }
    for (;;) { OUT_STEP(da, db) OUT_STEP(db, da) }
  }
  STAMP(7);
}

extern "C" void kernel_launch(int rank, int nranks, ncclComm_t comm,
                              hipStream_t stream,
                              void* const* d_in, const int* in_sizes, int n_in,
                              void* d_out, int out_size,
                              const int* in_dtypes, int out_dtype,
                              void* const* d_out_all,
                              void* const* d_ws_all, size_t ws_bytes) {
  (void)comm; (void)in_dtypes; (void)out_dtype; (void)d_out_all;
  if (nranks != 8 || (unsigned)rank >= 8u || n_in != 2 || ws_bytes < WS_NEED || in_sizes[0] != 8192 * 1024 || in_sizes[1] != 8192 * 1024 || out_size != 1024 * 8192) {
    fprintf(stderr, "gemm_rs: unexpected problem shape\n"); return; }
  HIP_CHECK(hipFuncSetAttribute((const void*)gemm_rs, hipFuncAttributeMaxDynamicSharedMemorySize, LDS_BYTES));
  Params p;
  p.xin = (const char*)d_in[0]; p.win = (const char*)d_in[1]; p.out = (char*)d_out; p.own = (char*)d_ws_all[rank]; p.rank = rank; p.pad = 0;
  for (int i = 0; i < 8; ++i) p.ws[i] = (char*)d_ws_all[i];
  gemm_rs<<<NWG, 512, LDS_BYTES, stream>>>(p);
  HIP_CHECK(hipGetLastError());
}
```

```cpp
#include <hip/hip_runtime.h>
#include <rccl/rccl.h>
#include <cstdio>

#define HIP_CHECK(expr) do { hipError_t _e = (expr); if (_e != hipSuccess) { \
    fprintf(stderr, "HIP error %s at %s:%d\n", hipGetErrorString(_e), __FILE__, __LINE__); \
    return; } } while (0)

#define LAS __attribute__((address_space(3)))
#define GAS __attribute__((address_space(1)))
typedef short bf16x8 __attribute__((ext_vector_type(8)));
typedef short s16x4 __attribute__((ext_vector_type(4)));
typedef short s16x2 __attribute__((ext_vector_type(2)));
typedef unsigned short u16x2 __attribute__((ext_vector_type(2)));
typedef __bf16 bf16x2 __attribute__((ext_vector_type(2)));
typedef float f32x2 __attribute__((ext_vector_type(2)));
typedef float f32x4 __attribute__((ext_vector_type(4)));
typedef unsigned u32x2 __attribute__((ext_vector_type(2)));
typedef unsigned u32x3 __attribute__((ext_vector_type(3)));
typedef unsigned u32x4 __attribute__((ext_vector_type(4)));
typedef unsigned long long u64;
typedef __amdgpu_buffer_rsrc_t rsrc_t;

#define PW 25
#define NRED 4
#define RC 32
#define RT 32
#define TGATE 0
#define NPUSH 7
#define SPLIT1 76
#define SPLIT2 152
#define PAUSE2 6
#define PAUSE3 17
#define GATE 16
#define DELAYS 2, 14, 12, 10, 8, 5, 3, 0
#define PACKS 4, 4, 0, 8, 8, 3, 0, 0
#define SKIP0 4
#define PSL 8
#define BTH 8
#define BUP 3

#define GBITS "sc0 sc1"
#define B96 0
#define EPI2 1
constexpr int NENT = 256 - NPUSH, NP0 = 28 - SKIP0;
struct alignas(16) Order { unsigned char e[256]; short pk[8][8]; };
constexpr Order make_order() {
  Order o{}; const int dl[8] = {DELAYS}; int n = 0;
  for (int key = 0; key < 96; ++key)
    for (int op = 1; op >= 0; --op)
      for (int k = 0; k < 16; ++k)
        for (int j = 0; j < 8; ++j) {
          const int tau = op ? (((k & 1) << 3) | (7 - (k >> 1))) : k;
          int kk = 3 * j + (op ? dl[tau & 7] : k < 6 ? 0 : k < 11 ? 1 : 2);
          if (op && (tau & 7) == 0 && 3 * j >= PAUSE3 - 7) kk = 80 + j;
          if (op && (tau & 7) == 1 && 3 * j < PAUSE2 - 7) kk = 3 * j;
          if (kk == key && (op || j || k >= NPUSH)) o.e[n++] = (unsigned char)(op << 7 | tau << 3 | j);
        }
  const int pc[8] = {PACKS}; n = 8 * NP0 - 6 * NPUSH;
  for (int k = 0; k < 8; ++k)
    for (int g = 7; g >= 0; --g) { o.pk[g][k] = (short)(pc[g] > k ? n : 1024); n += pc[g] > k ? 32 : 0; }
  return o;
}
constexpr Order ORDER_H = make_order();
constexpr bool order_ok() {
  for (int i = SPLIT1; i < NENT; ++i) { const int e = ORDER_H.e[i], op = e >> 7, tau = (e >> 3) & 15, j = e & 7;
    if ((!op || (tau & 7) == 1) && 3 * j < PAUSE2 - 7) return false;
    if (i >= SPLIT2 && (!op || (tau & 7) == 0) && 3 * j < PAUSE3 - 7) return false; }
  const int pc[8] = {PACKS}; int n = 8 * NP0 - 6 * NPUSH;
  for (int g = 0; g < 8; ++g) n += 32 * pc[g];
  return n >= 4 * NENT && pc[2] == 0 && pc[0] && pc[1] && (SKIP0 == 0 || pc[7] == 0) && (SKIP0 <= 4 || pc[6] == 0) && SKIP0 <= 8;
}
constexpr unsigned more_mask(int k) { unsigned m = 0; for (int g = 0; g < 8; ++g) m |= (ORDER_H.pk[g][k] < 4 * NENT ? 1u : 0u) << g; return m; }
static_assert(order_ok(), "a shift would wait for blocks it has to pull itself, or blocks stay unpacked");
static_assert(NRED >= 2 && NRED <= 8 && RC <= 16 * NRED && (RT == 0 || (RC == 8 * NRED && RT == RC)) && PW <= 42 && NPUSH >= 0 && NPUSH <= 8 && SPLIT1 * 2 >= PW && (SPLIT2 - SPLIT1) * 2 >= PW && (NENT - SPLIT2) * 2 >= PW, "roles");
__constant__ Order ORDER = ORDER_H;

constexpr int NWG = 256, HTB = 16384;
constexpr unsigned SA_ = 2048, SB_ = 16384;
constexpr size_t MiB = 1ull << 20;
constexpr size_t OFF_CALL = 0;
constexpr size_t OFF_TMO = 32768, OFF_Q = 33024, OFF_HAND = 36864;
constexpr size_t OFF_RDY = 40960;
constexpr size_t OFF_HIN = 65536;
constexpr size_t OFF_HOUT = OFF_HIN + 262144;
constexpr size_t OFF_STG = 1 * MiB;
constexpr size_t OFF_PIN = 73 * MiB;
constexpr size_t OFF_PINC = 121 * MiB;
constexpr size_t OFF_POUT = 153 * MiB;
constexpr size_t OFF_POUTC = 201 * MiB;
constexpr size_t OFF_DBG = 233 * MiB;
constexpr size_t WS_NEED = 234 * MiB;
constexpr int SCR = 2448, LDS_BYTES = 8 * HTB + 1024 + 8 * SCR;
constexpr unsigned OOB = 0x7FFFFF00u;
constexpr u64 DEADLINE = 100000000ull;

struct Params { const char* xin; const char* win; char* out; char* own; char* ws[8]; int rank; int pad; };

#define RFL(x) __builtin_amdgcn_readfirstlane(x)
#define WAIT_V(n) asm volatile("s_waitcnt vmcnt(" #n ")" ::: "memory")
#define WAIT_L(n) asm volatile("s_waitcnt lgkmcnt(" #n ")" ::: "memory")
#define DRAIN do { asm volatile("" ::: "memory"); __builtin_amdgcn_s_waitcnt(0x0F70); asm volatile("" ::: "memory"); } while (0)
#define BAR __builtin_amdgcn_s_barrier()
#define SCHED __builtin_amdgcn_sched_barrier(0)
#define FENCE asm volatile("" ::: "memory")
#define DI __device__ __forceinline__
constexpr int SYS = 17;

DI rsrc_t mk(const void* p, unsigned bytes) { return __builtin_amdgcn_make_buffer_rsrc((void*)p, (short)0, (int)bytes, 0x00020000); }
DI u32x4 ld16(rsrc_t r, unsigned v, unsigned s) { return __builtin_amdgcn_raw_buffer_load_b128(r, v, s, SYS); }
DI u32x4 ld16p(rsrc_t r, unsigned v, unsigned s) { return __builtin_amdgcn_raw_buffer_load_b128(r, v, s, 0); }
DI void st16(u32x4 x, rsrc_t r, unsigned v, unsigned s) { __builtin_amdgcn_raw_buffer_store_b128(x, r, v + s, 0, SYS); }
DI unsigned aload(const char* p) { return __hip_atomic_load((const GAS unsigned*)p, __ATOMIC_RELAXED, __HIP_MEMORY_SCOPE_SYSTEM); }
DI void astore(char* p, unsigned v) { __hip_atomic_store((GAS unsigned*)p, v, __ATOMIC_RELAXED, __HIP_MEMORY_SCOPE_SYSTEM); }
DI unsigned perm(unsigned hi, unsigned lo, unsigned sel) { return __builtin_amdgcn_perm(hi, lo, sel); }
DI unsigned cvt2(float lo, float hi) { return __builtin_bit_cast(unsigned, __builtin_convertvector((f32x2){lo, hi}, bf16x2)); }

#define STAMP(i) do { } while (0)

DI bool give_up(char* ws, unsigned call, u64 t0) {
  const u64 dt = __builtin_amdgcn_s_memrealtime() - t0;
  if (dt > 10000) {
    if (RFL(aload(ws + OFF_TMO)) == call) return true;
    if (dt > DEADLINE) { astore(ws + OFF_TMO, call); return true; }
  }
  __builtin_amdgcn_s_sleep(2);
  return false;
}

struct Unit { u32x4 a0, a1; u32x3 p; unsigned pt; };
DI unsigned bfi(unsigned m, unsigned a, unsigned b) { unsigned r; asm("v_bfi_b32 %0, %1, %2, %3" : "=v"(r) : "s"(m), "v"(a), "v"(b)); return r; }
DI unsigned pka(unsigned a, unsigned b) { return __builtin_bit_cast(unsigned, __builtin_bit_cast(u16x2, a) + __builtin_bit_cast(u16x2, b)); }
DI unsigned pks(unsigned a, unsigned b) { return __builtin_bit_cast(unsigned, __builtin_bit_cast(u16x2, a) - __builtin_bit_cast(u16x2, b)); }
#define LANES(c) __builtin_popcountll(__builtin_amdgcn_ballot_w64(c))
DI unsigned pick_base(unsigned f) {
  unsigned g = RFL(f) & ~7u;
  while (LANES(f >= g + 8) >= BTH) g += 8;
  while (g && LANES(f >= g) < BTH) g -= 8;
#pragma unroll
  for (int bit = 4; bit; bit >>= 1) g = LANES(f >= g + bit) >= BTH ? g + bit : g;
  return g > 64 - BUP ? g - (64 - BUP) : 0;
}
DI unsigned base_of(const u32x4 (&r)[4]) {
  u16x2 mx = {0, 0};
#pragma unroll
  for (int c = 0; c < 4; ++c)
#pragma unroll
    for (int i = 0; i < 4; i += 2) mx = __builtin_elementwise_max(mx, __builtin_bit_cast(u16x2, r[c][i] & 0x7FFF7FFFu));
  return pick_base((mx.x > mx.y ? mx.x : mx.y) >> 4);
}
DI unsigned pack_unit(const u32x4 (&r)[4], unsigned base, bool odd, int lane, LAS unsigned char* scr, rsrc_t dst, unsigned doff, rsrc_t dstc, unsigned coff) {
  const unsigned B2 = base * 0x00100010u, M = 0x83838383u;
  unsigned A[8], T[8], mask = 0, n = 0;
#pragma unroll
  for (int k = 0; k < 8; ++k) {
    const unsigned lo = pks(r[k >> 1][(k & 1) * 2], B2), hi = pks(r[k >> 1][(k & 1) * 2 + 1], B2);
    A[k] = perm(hi, lo, 0x06040200u); T[k] = perm(hi, lo, 0x07050301u);
    mask = bfi(0x80808080u, (T[k] & 0x7C7C7C7Cu) + 0x7C7C7C7Cu, mask >> 1);
  }
  *(LAS u32x4*)(scr + lane * 32) = (u32x4){T[0], T[1], T[2], T[3]}; *(LAS u32x4*)(scr + lane * 32 + 16) = (u32x4){T[4], T[5], T[6], T[7]};
  st16((u32x4){A[0], A[1], A[2], A[3]}, dst, lane * 16, doff);
  st16((u32x4){A[4], A[5], A[6], A[7]}, dst, lane * 16 + 1024, doff);
  const unsigned P0 = bfi(0x60606060u, T[6] << 5, bfi(M, T[0], __builtin_rotateleft32(T[1], 3)));
  const unsigned P1 = bfi(0x60606060u, T[7] << 5, bfi(M, T[2], __builtin_rotateleft32(T[3], 3)));
  const unsigned P2 = bfi(0x40404040u, T[6] >> 1, bfi(0x20202020u, T[7] >> 2, bfi(M, T[4], __builtin_rotateleft32(T[5], 3))));
  __builtin_amdgcn_raw_buffer_store_b64((u32x2){P0, P1}, dst, lane * 8 + 2048 + doff, 0, SYS);
  __builtin_amdgcn_raw_buffer_store_b32(P2, dst, lane * 4 + 2560 + doff, 0, SYS);
  FENCE;
  for (u64 m; n <= 96 && (m = __builtin_amdgcn_ballot_w64(mask != 0)) != 0; n += __builtin_popcountll(m)) {
    const unsigned i = __builtin_amdgcn_mbcnt_hi((unsigned)(m >> 32), __builtin_amdgcn_mbcnt_lo((unsigned)m, n));
    const unsigned b = __builtin_ctz(mask | 0x80000000u), a = lane * 32 + ((b & 7) << 2 | b >> 3);
    *(LAS unsigned short*)(scr + 2048 + (mask ? i : 192u) * 2) = (unsigned short)((scr[a] & 0x7Cu) << 9 | a);
    mask &= mask - 1;
  }
  FENCE;
  if (n > 96) {
    n = 255;
    st16((u32x4){T[0], T[1], T[2], T[3]} & 0x7C7C7C7Cu, dstc, lane * 16, coff);
    st16((u32x4){T[4], T[5], T[6], T[7]} & 0x7C7C7C7Cu, dstc, lane * 16 + 1024, coff);
  } else __builtin_amdgcn_raw_buffer_store_b32(*(const LAS unsigned*)(scr + 2048 + lane * 4), dst, 2u * lane < n ? lane * 4 + doff + (odd && lane < 16 ? -256 : 2880) : OOB, 0, SYS);
  FENCE;
  return base | n << 12;
}
constexpr int NLD = B96 ? 4 : 5;
DI void load_unit(Unit& d, rsrc_t src, unsigned off, unsigned hw, bool odd, int lane) {
  const unsigned n = (hw >> 12) & 127u;
  d.pt = __builtin_amdgcn_raw_buffer_load_b32(src, odd ? (lane >= 16 && lane < 48 && 2u * lane < n ? lane * 4 + 2880 : OOB) : (2u * lane < (n > 32 ? n : 32) + 32 ? lane * 4 + 2816 : OOB), off, SYS);
  const u32x2 q = __builtin_amdgcn_raw_buffer_load_b64(src, lane * 8 + 2048, off, SYS);
  d.p = (u32x3){q[0], q[1], __builtin_amdgcn_raw_buffer_load_b32(src, lane * 4 + 2560, off, SYS)};
  d.a0 = ld16(src, lane * 16, off); d.a1 = ld16(src, lane * 16 + 1024, off);
}
DI void unpack_hi(unsigned (&T)[8], const Unit& d, unsigned hw, bool odd, int lane, LAS unsigned char* scr, rsrc_t srcc, unsigned coff) {
  const unsigned n = (hw >> 12) & 255u, M = 0x83838383u, e = 2u * lane - (odd ? 0u : 32u);
  u32x4 x0, x1;
  if (n > 96) { x0 = ld16(srcc, lane * 16, coff); x1 = ld16(srcc, lane * 16 + 1024, coff); DRAIN; }
  else {
    const unsigned a0 = d.pt & 0x7FFu, a1 = (d.pt >> 16) & 0x7FFu;
    if (e < n) scr[a0] = (unsigned char)((d.pt >> 9) & 0x7Cu);
    if (e + 1 < n) scr[a1] = (unsigned char)((d.pt >> 25) & 0x7Cu);
    FENCE;
    x0 = *(const LAS u32x4*)(scr + lane * 32); x1 = *(const LAS u32x4*)(scr + lane * 32 + 16);
    FENCE;
    if (e < n) { scr[a0] = 0; scr[a1] = 0; }
    FENCE;
  }
  T[0] = (d.p[0] & M) | x0[0]; T[1] = (__builtin_rotateright32(d.p[0], 3) & M) | x0[1];
  T[2] = (d.p[1] & M) | x0[2]; T[3] = (__builtin_rotateright32(d.p[1], 3) & M) | x0[3];
  T[4] = (d.p[2] & M) | x1[0]; T[5] = (__builtin_rotateright32(d.p[2], 3) & M) | x1[1];
  T[6] = (bfi(0x03030303u, d.p[0] >> 5, d.p[2] << 1) & M) | x1[2];
  T[7] = (bfi(0x03030303u, d.p[1] >> 5, d.p[2] << 2) & M) | x1[3];
}
DI u32x4 unpack_row(const unsigned (&T)[8], const Unit& d, int c, unsigned hw) {
  const unsigned B2 = (hw & 0xFFFu) * 0x00100010u, a = (c & 2 ? d.a1 : d.a0)[(c & 1) * 2], b = (c & 2 ? d.a1 : d.a0)[(c & 1) * 2 + 1];
  return (u32x4){pka(perm(T[2 * c], a, 0x05010400u), B2), pka(perm(T[2 * c], a, 0x07030602u), B2), pka(perm(T[2 * c + 1], b, 0x05010400u), B2), pka(perm(T[2 * c + 1], b, 0x07030602u), B2)};
}
DI void zero_scr(LAS unsigned char* scr, int lane) { *(LAS u32x4*)(scr + lane * 32) = (u32x4){0u, 0u, 0u, 0u}; *(LAS u32x4*)(scr + lane * 32 + 16) = (u32x4){0u, 0u, 0u, 0u}; FENCE; }
DI bool wait_hdr(u32x4& h, rsrc_t rh, unsigned voff, unsigned soff, char* ws, unsigned call) {
  if (!__builtin_amdgcn_ballot_w64(h[0] != call || h[2] != call)) return true;
  const u64 t0 = __builtin_amdgcn_s_memrealtime();
  for (;;) {
    h = ld16(rh, voff, soff); FENCE;
    if (!__builtin_amdgcn_ballot_w64(h[0] != call || h[2] != call)) return true;
    if (give_up(ws, call, t0)) { h = (u32x4){0u, 0u, 0u, 0u}; return false; }
  }
}
#define CLAIM(cv, base, on) do { cv = 1u; asm volatile("s_mov_b32 exec_lo, %3\n\ts_mov_b32 exec_hi, 0\n\ts_nop 2\n\tglobal_atomic_add %0, %1, %0, %2 sc0\n\ts_mov_b64 exec, -1" : "+v"(cv) : "v"(0u), "s"(base), "s"(RFL((on) ? 1u : 0u)) : "memory"); } while (0)
#define RL(x, l) ((unsigned)__builtin_amdgcn_readlane((int)(x), (l)))

__global__ __launch_bounds__(512, 2) void gemm_rs(Params p) {
  extern __shared__ __attribute__((aligned(16))) unsigned char smem[];
  LAS unsigned char* lds = (LAS unsigned char*)smem;
  const int tid = threadIdx.x, wid = RFL(tid >> 6), lane = tid & 63, wr = wid >> 2, wc = wid & 3, fr = lane & 15, fq = lane >> 4;
  unsigned tab = ((const unsigned*)ORDER.e)[lane];
  asm volatile("" :: "s"(p.xin), "s"(p.win), "s"(p.own), "s"(p.rank), "s"(p.ws[(blockIdx.x & 3) * 2 + ((blockIdx.x >> 2) & 1)]) : "memory");
  const int rank = p.rank & 7, ra = rank >> 2, rb = (rank >> 1) & 1, rc = rank & 1, rcol = rank >> 1;
  char* const ws = p.own;
  const int bid = blockIdx.x, xcd = bid & 7, idx = bid >> 3;
  const int tm = (xcd >> 1) * 4 + (idx & 3), tn = (xcd & 1) * 8 + (idx >> 2);
  const int shift = idx < 12 ? 3 - (idx >> 2) : 0;
  LAS unsigned char* const scr = lds + 8 * HTB + 1024 + wid * SCR;
  STAMP(0);
  unsigned callv = aload(ws + OFF_CALL + bid * 128);
  volatile LAS int* const seen = (volatile LAS int*)(lds + 8 * HTB + 896);
  if (tid < 8) seen[tid] = 0;

  const unsigned ldsw = (unsigned)(size_t)lds + (unsigned)wid * 1024u;
  unsigned vA[4], vB[4];
  { const unsigned voffA = (wid * 8 + (lane >> 3)) * SA_ + 16 * ((lane & 7) ^ (lane >> 3));
    const int s = lane >> 3, pos = (lane & 7) >> 1, hf = lane & 1, cp = pos ^ (s >> 1);
    const unsigned voffB = (16 * (wid >> 2) + 8 * (s >> 2) + 4 * ((wid >> 1) & 1) + (s & 3)) * SB_ + (64 * (wid & 1) + 16 * cp + 8 * hf) * 2;
#pragma unroll
    for (int i = 0; i < 4; ++i) { vA[i] = voffA + i * 64 * SA_; vB[i] = voffB + (i & 1) * 32 * SB_ + (i >> 1) * 256; asm volatile("" : "+v"(vA[i]), "+v"(vB[i])); } }
#define SA(b, h) (((b) * 2 + (h)) * HTB)
#define SB(b, h) ((4 + (b) * 2 + (h)) * HTB)
#define GLDS(voff, sbase, c) asm volatile("s_add_i32 m0, %2, %3\n\ts_nop 0\n\tglobal_load_lds_dwordx4 %0, %1 " GBITS :: "v"(voff), "s"(sbase), "s"(ldsw), "i"(c) : "memory", "scc")
#define STAGEA(b, h, gbase) do { GLDS(vA[2 * (h)], gbase, SA(b, h)); GLDS(vA[2 * (h) + 1], gbase, SA(b, h) + 8192); } while (0)
#define STAGEB(b, h, gbase) do { GLDS(vB[2 * (h)], gbase, SB(b, h)); GLDS(vB[2 * (h) + 1], gbase, SB(b, h) + 8192); } while (0)
  const char* const ownA = p.xin + (size_t)((tm >> 1) * 1024 + ra * 512 + (tm & 1) * 256) * SA_;
  const char* const ownB = p.win + (size_t)(rb * 4096 + tn * 256) * 2;
  const char* a1 = ownA + 128;

  unsigned call, par;
  {
    const unsigned loX = (lane >> 3) * SA_ + (lane & 7) * 16, loW = (lane >> 5) * SB_ + (lane & 31) * 16;
    const int pd = (xcd & 3) * 2 + (xcd >> 2), pq = ((xcd & 3) ^ rcol) - 1;
    const bool push = idx < NPUSH && pq >= 0;
    const bool more = (more_mask(0) >> (idx >> 2)) & 1;
    int blk[4]; bool val[4];
#define PK_BLOCK(k, g) do { val[k] = (g) < 4 * NENT; const int _g = val[k] ? (g) : (xcd >> 1); \
      const int _e = val[k] ? (RL(tab, _g >> 4) >> ((_g >> 2) & 3) * 8) & 255 : (idx & 15) << 3, _op = _e >> 7, _tau = (_e >> 3) & 15, _hv = _g & 1; \
      blk[k] = _op * 512 + (((_g >> 1) & 1) * 8 + (_e & 7)) * 32 + (_op ? _hv * 16 + _tau : (_tau >> 1) * 4 + _hv * 2 + (_tau & 1)); } while (0)
    const bool none = shift == 1 || idx >= 32 - SKIP0;
    PK_BLOCK(0, push || none ? 1024 : pq < 0 ? (idx < 8 ? idx : idx - 4) * 2 + (xcd >> 2) : 2 * NP0 + ((idx < 8 ? idx : idx - 4) - NPUSH) * 6 + pq * 2 + (xcd >> 2));
    u32x4 r[4][4]; int hl = 8 * HTB;
#define PK_LOAD(k, on) do { const int _op = blk[k] >> 9, _jj = (blk[k] >> 5) & 15, _ix = blk[k] & 31; \
      const unsigned _so = _op ? (_jj * 64 + wid * 8) * SB_ + _ix * 512 : (_ix * 256 + wid * 32) * SA_ + _jj * 128, _cs = _op ? 2 * SB_ : 8 * SA_; \
      _Pragma("unroll") for (int c = 0; c < 4; ++c) r[k][c] = ld16p(mk(_op ? p.win : p.xin, on ? 16u << 20 : 0u), _op ? loW : loX, _so + c * _cs); } while (0)
#define PK_PACK(k) do { const unsigned _h = pack_unit(r[k], base_of(r[k]), wid & 1, lane, scr, rpi, blk[k] * 24576 + wid * 3072, rpc, blk[k] * 16384 + wid * 2048); \
      if (lane == 0) *(LAS unsigned*)(lds + hl + (k) * 32 + wid * 4) = _h; } while (0)
#define PK_SEND(k) do { const int _op = blk[k] >> 9, _jj = (blk[k] >> 5) & 15, _ix = blk[k] & 31, _d = (lane >> 3) & 1; \
      const int _tau = _op ? (_ix & 15) : ((_ix >> 2) * 2 + (_ix & 1)); \
      const int _c0 = _op ? (_ix >> 4) : (_ix & 2), _c1 = _op ? _c0 + 2 : _c0 + 1; \
      char* const _w0 = p.ws[(_c0 * 2 + (_jj >> 3)) & 7]; char* const _w1 = p.ws[(_c1 * 2 + (_jj >> 3)) & 7]; \
      if (lane < 16 && (_d ? _c1 : _c0) != rcol) { const u32x2 _h = {call, *(const LAS unsigned*)(lds + hl + (k) * 32 + (lane & 7) * 4)}; \
        char* _dst = (_d ? _w1 : _w0) + OFF_HIN + (size_t)((par * 8 + rank) * 256 + _op * 128 + _tau * 8 + (_jj & 7)) * 64 + (lane & 7) * 8; \
        asm volatile("global_store_dwordx2 %0, %1, off sc0 sc1\n\ts_nop 1" :: "v"(_dst), "v"(_h) : "memory"); } } while (0)
#define PROLOGUE() do { STAGEB(0, 0, ownB); STAGEA(0, 0, ownA); STAGEB(0, 1, ownB); STAGEA(0, 1, ownA); \
      STAGEB(1, 0, ownB + 64 * SB_); STAGEA(1, 0, a1); STAGEB(1, 1, ownB + 64 * SB_); } while (0)
    if (none && shift != 1) PROLOGUE();
    PK_LOAD(0, !none);
    asm volatile("" : "+v"(callv) :: "memory");
    call = RFL(callv) + 1; par = call & 1;
    const rsrc_t rpi = mk(ws + OFF_PIN + (size_t)par * 24 * MiB, 24u << 20), rpc = mk(ws + OFF_PINC + (size_t)par * 16 * MiB, 16u << 20);
    asm volatile("" : "+v"(tab), "+v"(r[0][0]), "+v"(r[0][1]), "+v"(r[0][2]), "+v"(r[0][3]));
    if (push) {
      const rsrc_t rq = mk(p.ws[pd] + OFF_STG + (size_t)par * 48 * MiB, 24u << 20);
#pragma unroll
      for (int c = 0; c < 4; ++c) st16(r[0][c], rq, loX, (pq * 4096 + idx * 256 + wid * 32 + c * 8) * SA_ + rc * 1024);
      FENCE;
    } else if (none) { WAIT_V(0); WAIT_L(0); BAR; }
    else {
      PK_PACK(0);
      if (more) WAIT_V(0); else { PROLOGUE(); WAIT_V(14); }
      WAIT_L(0); BAR; FENCE;
      if (wid == 0) PK_SEND(0);
      if (!more) { if (wid == 0) WAIT_V(1); else WAIT_V(0); BAR; }
    }
    STAMP(1);
    if (more) {
      for (int q = 0; q < 8; q += 4) {
        const bool last = q == 4 || !((more_mask(4) >> (idx >> 2)) & 1);
        hl += 128;
#pragma unroll
        for (int k = 0; k < 4; ++k) { PK_BLOCK(k, ORDER.pk[idx >> 2][q + k] + (bid & 31)); PK_LOAD(k, val[k]); }
        FENCE;
        if (val[0]) { PK_PACK(0); if (val[1]) { PK_PACK(1); if (val[2]) { PK_PACK(2); if (val[3]) PK_PACK(3); } } }
        if (last) PROLOGUE();
        DRAIN; WAIT_L(0); BAR; FENCE;
#pragma unroll
        for (int k = 0; k < 4; ++k) if (wid == k + 1 && val[k]) PK_SEND(k);
        if (q == 0 && (tid >> 2) == 80 && push) astore(p.ws[pd] + OFF_RDY + (idx * 48 + pq * 2 + rc) * 16 + (tid & 3) * 4, call);
        if (last) break;
      }
    }
    if (tid == 384) astore(ws + OFF_CALL + bid * 128, call);
    if (shift != 1 && wr == 1) BAR;
  }
  STAMP(2);

  int aoff0 = (wr * 64 + fr) * 128 + 16 * (fq ^ (fr & 7)), aoff1 = aoff0 ^ 64, boff0, boff1;
  { const int s = (lane >> 2) & 7, pp = ((lane & 1) << 1) | ((lane >> 1) & 1), gh = lane >> 5;
    boff0 = 4 * HTB + gh * 4096 + (wc >> 1) * 1024 + 128 * s + 32 * ((2 * (wc & 1)) ^ (s >> 1)) + 8 * pp; boff1 = boff0 ^ 32; }
  asm volatile("" : "+v"(aoff0), "+v"(aoff1), "+v"(boff0), "+v"(boff1));
#define LDA(dst, b, h) do { _Pragma("unroll") for (int m = 0; m < 4; ++m) { dst[m][0] = *(const LAS bf16x8*)(lds + SA(b, h) + aoff0 + m * 2048); dst[m][1] = *(const LAS bf16x8*)(lds + SA(b, h) + aoff1 + m * 2048); } } while (0)
#define TR(off) __builtin_amdgcn_ds_read_tr16_b64_v4i16((LAS s16x4*)(lds + (off)))
#define LDB(dst, b, h) do { _Pragma("unroll") for (int k = 0; k < 2; ++k) { \
    dst[0][k] = __builtin_shufflevector(TR(SA(b, h) + boff0 + k * 8192), TR(SA(b, h) + boff0 + k * 8192 + 2048), 0, 1, 2, 3, 4, 5, 6, 7); \
    dst[1][k] = __builtin_shufflevector(TR(SA(b, h) + boff1 + k * 8192), TR(SA(b, h) + boff1 + k * 8192 + 2048), 0, 1, 2, 3, 4, 5, 6, 7); } } while (0)
#define MMA(ai, bj, At, Bt) do { __builtin_amdgcn_s_setprio(1); _Pragma("unroll") for (int m = 0; m < 4; ++m) _Pragma("unroll") for (int n = 0; n < 2; ++n) _Pragma("unroll") for (int k = 0; k < 2; ++k) \
    acc[ai][bj][m][n] = __builtin_amdgcn_mfma_f32_16x16x32_bf16(Bt[n][k], At[m][k], acc[ai][bj][m][n], 0, 0, 0); __builtin_amdgcn_s_setprio(0); } while (0)
  f32x4 acc[2][2][4][2];
#pragma unroll
  for (int a = 0; a < 2; ++a)
#pragma unroll
    for (int b = 0; b < 2; ++b)
#pragma unroll
      for (int m = 0; m < 4; ++m)
#pragma unroll
        for (int n = 0; n < 2; ++n) acc[a][b][m][n] = (f32x4){0.f, 0.f, 0.f, 0.f};
  bf16x8 At[4][2], B0[2][2], B1[2][2];
  const char* const sA = ws + OFF_STG + (size_t)par * 48 * MiB + (size_t)tm * 256 * SA_;
  const char* const sB = ws + OFF_STG + 24 * MiB + (size_t)tn * 512;
  const int pause = shift == 1 ? 0 : shift == 2 ? PAUSE2 : shift == 3 ? PAUSE3 : -1;
  const int s0 = tm < 6 ? 0 : tm < 11 ? 2 : 1;
  int j = 0, s2 = s0, ready = 0;
  unsigned fl = 0;
  const char* const rdy = ws + OFF_RDY;
#define FOFF() ([&] { int l = tid; asm volatile("" : "+v"(l)); const int k0 = ready + ((l >> 2) & 7), k = k0 < 47 ? k0 : 47; \
    return (unsigned)(((l & 32 ? 16 + tn : tm) * 48 + k + 2 * s0 - (k % 6 + 2 * s0 >= 6 ? 6 : 0)) * 16 + (l & 3) * 4); }())
#define READY() do { const u64 _m = __builtin_amdgcn_ballot_w64((int)(fl - call) < 0); ready += __builtin_ctzll((_m | _m >> 32 | 1ull << 32) & 0x1FFFFFFFFull) >> 2; } while (0)
  for (int it = 0; it < 32; ++it) {
    if (it == pause) {
      if (wr == 0 && shift != 1) BAR;
      STAMP(3);
      int ln = tid; asm volatile("" : "+v"(ln)); ln &= 63;
      const unsigned gw = wid * 32 + (((idx & 3) * 8 + xcd + wid) & 31), s = gw % 6, widx = gw / 6;
      if (widx < PW) {
      const unsigned lb = shift == 1 ? 0 : shift == 2 ? SPLIT1 * 4 : SPLIT2 * 4, le = shift == 1 ? SPLIT1 * 4 : shift == 2 ? SPLIT2 * 4 : NENT * 4;
      const unsigned qb = (call - 1) * (le - lb - PW) - lb - 2 * PW;
      char* const ctr = ws + OFF_Q + (s * 3 + shift - 1) * 128;
      const unsigned src = ((((unsigned)rcol ^ (1 + (s >> 1))) << 1) | (s & 1)) & 7;
      const char* const sw = p.ws[src];
      const unsigned lx = (ln >> 3) * SA_ + (ln & 7) * 16, lw = (ln >> 5) * SB_ + (ln & 31) * 16;
      const rsrc_t rsc = mk(sw + OFF_PINC + (size_t)par * 16 * MiB, 16u << 20), rh = mk(ws + OFF_HIN + (size_t)(par * 8 + src) * 16384, 16384), rd = mk(ws + OFF_STG, 72u << 20);
      zero_scr(scr, ln);
      { const unsigned xm = ((~tab >> 7) & (((tab & 0x07070707u) + 0x07070707u) >> 3) & 0x01010101u) * 0x78u;
        tab = (tab & ~xm) | (((tab & xm) + (s >> 1) * 0x58585858u % 0x80808080u) & xm); }
#define ENT(L) ((((RL(tab, ((L) >> 4) & 63) >> (((L) >> 2) & 3) * 8) & 255u) << 3) | ((L) & 3) << 1)
#define IN_SRC(E) (((E) >> 10) ? (512 + (rc * 8 + (((E) >> 3) & 7)) * 32 + rb * 16 + (((E) >> 6) & 15)) * 8 + ((E) & 7) \
                               : ((rc * 8 + (((E) >> 3) & 7)) * 32 + ((E) >> 7) * 4 + ra * 2 + (((E) >> 6) & 1)) * 8 + ((E) & 7))
#define SIGNAL(E) do { if (ln == 0) astore(ws + OFF_RDY + (((E) >> 6) * 48 + (((E) >> 3) & 7) * 6 + s) * 16 + ((E) & 6) * 2, call); } while (0)
      struct Buf { Unit d; unsigned Ec, hw; bool live; } A, B;
      unsigned cv = 0, keep, hn, Ep = ~0u, Ln = lb + PW + widx, En = ENT(Ln);
      A.Ec = ENT(lb + widx); B.Ec = A.Ec + 1; A.live = B.live = true;
      u32x4 hv = ld16(rh, 0, A.Ec * 8);
      if (shift != 1) { const u64 t0 = __builtin_amdgcn_s_memrealtime(); while (RFL(aload(ws + OFF_HAND + (shift - 2) * 1024 + gw * 4)) != call && !give_up(ws, call, t0)) { } }
#define IN_LOAD(v, odd) do { FENCE; load_unit(v.d, mk(sw + OFF_PIN + (size_t)par * 24 * MiB, v.live ? 24u << 20 : 0u), IN_SRC(v.Ec) * 3072, v.hw, odd, ln); FENCE; } while (0)
      wait_hdr(hv, rh, 0, A.Ec * 8, ws, call); A.hw = RFL(hv[1]); B.hw = RFL(hv[3]); STAMP(8);
      IN_LOAD(A, false); hv = ld16(rh, 0, En * 8); IN_LOAD(B, true);
#define IN_STEP(v, odd) do { __builtin_amdgcn_s_waitcnt(0x0F70 | NLD); asm volatile("s_waitcnt vmcnt(%1)" : "+v"(cv) : "i"(NLD) : "memory"); \
        if (!odd && Ep != ~0u) SIGNAL(Ep); \
        if (odd) v.d.pt = ln < 16 ? keep : v.d.pt; else keep = v.d.pt; \
        unsigned T[8]; unpack_hi(T, v.d, v.hw, odd, ln, scr, rsc, IN_SRC(v.Ec) * 2048); \
        const unsigned _op = v.Ec >> 10, _tau = (v.Ec >> 6) & 15, _j = (v.Ec >> 3) & 7, _u = v.Ec & 7, _cs = _op ? 2 * SB_ : 8 * SA_, _lo = _op ? lw : lx; \
        const unsigned _db = _op ? 24 * (unsigned)MiB + ((s >> 1) * 512 + _j * 64 + _u * 8) * SB_ + (s & 1) * 8192 + _tau * 512 \
                                 : par * 48 * (unsigned)MiB + ((s >> 1) * 4096 + _tau * 256 + _u * 32) * SA_ + (s & 1) * 1024 + _j * 128; \
        _Pragma("unroll") for (int c = 0; c < 4; ++c) st16(unpack_row(T, v.d, c, v.hw), rd, _lo, _db + c * _cs); \
        FENCE; } while (0)
      do {
        IN_STEP(A, false);
        A.live = Ln < le;
        if (A.live) wait_hdr(hv, rh, 0, En * 8, ws, call);
        A.hw = RFL(hv[1]); hn = RFL(hv[3]); A.Ec = En;
        CLAIM(cv, ctr, A.live); IN_LOAD(A, false);
        IN_STEP(B, true);
        Ep = B.Ec;
        B.live = A.live; B.hw = hn; B.Ec = A.Ec + 1; Ln = RFL(cv) - qb; En = ENT(Ln);
        if (A.live && Ln >= le - (shift == 3 ? GATE : 0) && ln == 0) astore(ws + OFF_HAND + (shift - 1) * 1024 + gw * 4, call);
        hv = ld16(rh, 0, En * 8); IN_LOAD(B, true);
      } while (A.live);
      DRAIN;
      SIGNAL(Ep);
      }
      if (shift == 1) { PROLOGUE(); WAIT_V(0); }
      STAMP(4);
      if (shift == 1) BAR;
      if (wr == 1) BAR;
    }
    if (it == 7) STAMP(12); if (it == 19) STAMP(13); if (it == 25) STAMP(14); if (it == 28) STAMP(15);
    const char *a2, *b2, *a3, *b3;
    if (it < 7 || it == 31) { const int t = (it == 31) ? 0 : 2 * it + 2; a2 = ownA + t * 128; b2 = ownB + (size_t)t * 64 * SB_; a3 = a2 + 128; b3 = b2 + 64 * SB_; }
    else {
      if (ready < 2 * it - 12) {
        ready = RFL(*seen);
        if (ready < 2 * it - 12) {
          const u64 t0 = __builtin_amdgcn_s_memrealtime();
          for (;;) {
            if (wid == 0) { fl = aload(rdy + FOFF()); READY(); } else ready = RFL(*seen);
            if (ready >= 2 * it - 12) break;
            if (give_up(ws, call, t0)) { ready = 64; break; }
            if (PSL && wid == 0) __builtin_amdgcn_s_sleep(PSL);
          }
        }
        if (wid == 0) *seen = ready;
        FENCE;
      }
      a2 = sA + (size_t)s2 * (4096 * SA_) + j * 128; b2 = sB + (size_t)(s2 * 512 + j * 64) * SB_; a3 = a2 + 1024; b3 = b2 + 8192; if (++s2 == 3) s2 = 0;
      if (s2 == s0) ++j;
    }
    LDB(B0, 0, 0); SCHED; LDA(At, 0, 0); STAGEA(1, 1, a1);
    WAIT_L(8); BAR; WAIT_L(0); MMA(0, 0, At, B0); BAR; SCHED;
    LDB(B1, 0, 1); STAGEB(0, 0, b2);
    BAR; WAIT_L(0); MMA(0, 1, At, B1); BAR;
    LDA(At, 0, 1); STAGEA(0, 0, a2);
    BAR; WAIT_L(0); MMA(1, 0, At, B0); BAR; SCHED;
    STAGEB(0, 1, b2);
    WAIT_V(6);
    const bool look = ready < 2 * it - 10 && it < 30 && wid == 0;
    if (look) asm volatile("s_nop 4\n\tglobal_load_dword %0, %1, %2 sc0 sc1" : "+v"(fl) : "v"(FOFF()), "s"(rdy) : "memory");
    BAR; MMA(1, 1, At, B1); BAR;
    LDB(B0, 1, 0); SCHED; LDA(At, 1, 0); STAGEA(0, 1, a2);
    WAIT_L(8); BAR; WAIT_L(0); MMA(0, 0, At, B0); BAR; SCHED;
    LDB(B1, 1, 1); STAGEB(1, 0, b3);
    BAR; WAIT_L(0); MMA(0, 1, At, B1); BAR;
    LDA(At, 1, 1); STAGEA(1, 0, a3);
    BAR; WAIT_L(0); MMA(1, 0, At, B0); BAR; SCHED;
    STAGEB(1, 1, b3);
    asm volatile("s_waitcnt vmcnt(6)" : "+v"(fl) :: "memory");
    if (look) { READY(); *seen = ready; }
    BAR; MMA(1, 1, At, B1); BAR;
    a1 = a3;
  }
  if (wr == 0) BAR;
  STAMP(5);

  int ln = tid; asm volatile("" : "+v"(ln)); ln &= 63;
  {
    const unsigned tmo = aload(ws + OFF_TMO);
    const unsigned slot = (tm * 16 + tn) * 8 + wid;
    const rsrc_t rpo = mk(ws + OFF_POUT + (size_t)par * 24 * MiB + (size_t)slot * 12288, 12288), rpc = mk(ws + OFF_POUTC + (size_t)par * 16 * MiB + (size_t)slot * 8192, 8192);
    char* const dst = p.ws[(tm >> 1) & 7] + OFF_HOUT + (size_t)(((par * 8 + rank) * 32 + (tm & 1) * 16 + tn) * 8 + wid) * 32 + (ln & 3) * 8;
    u32x2 hs = {call, 0u}; unsigned base = 0;
    if (wr == 0) __builtin_amdgcn_s_setprio(2);
#pragma unroll
    for (int u = 0; u < 4; ++u) {
      u32x4 r[4];
#pragma unroll
      for (int c = 0; c < 4; ++c) {
        const f32x4 x = acc[u >> 1][c & 1][(u & 1) * 2 + (c >> 1)][0], y = acc[u >> 1][c & 1][(u & 1) * 2 + (c >> 1)][1];
        const auto s0 = __builtin_amdgcn_permlane32_swap(cvt2(x[0], x[1]), cvt2(y[0], y[1]), false, false);
        const auto s1 = __builtin_amdgcn_permlane32_swap(cvt2(x[2], x[3]), cvt2(y[2], y[3]), false, false);
        r[c] = (u32x4){s0[0], s1[0], s0[1], s1[1]};
      }
      if (u == 0) base = base_of(r);
      const unsigned h = pack_unit(r, base, u & 1, ln, scr, rpo, u * 3072, rpc, u * 2048);
      if (ln == u) hs[1] |= h;
      if (u == 2 && tmo == call) hs[1] |= 0x1000000u;
      if (EPI2 && u == 2) {
        asm volatile("s_waitcnt vmcnt(%0)" :: "i"(NLD) : "memory");
        if (ln < 2) asm volatile("global_store_dwordx2 %0, %1, off sc0 sc1\n\ts_nop 1" :: "v"(dst), "v"(hs) : "memory"); }
    }
    WAIT_V(0);
    if (ln < 4 && ln >= 2 * EPI2) asm volatile("global_store_dwordx2 %0, %1, off sc0 sc1\n\ts_nop 1" :: "v"(dst), "v"(hs) : "memory");
  }
  STAMP(6);

  const unsigned gr = wid * (8 * NRED) + ((31 - idx) * 8 + xcd + wid) % (8 * NRED), col = gr & 3, widx = gr >> 2;
  if (idx >= 32 - NRED && widx < RC + RT) {
    constexpr unsigned NM = 512 - RT; const bool tail = widx >= RC;
    const unsigned qb = (call - 1) * NM - RC;
    char* const ctr = ws + OFF_Q + (18 + col) * 128;
    const char* const w0 = p.ws[col * 2]; const char* const w1 = p.ws[col * 2 + 1];
    const size_t mine = (size_t)rank * 1024;
    const rsrc_t r0 = mk(w0 + OFF_POUT + (size_t)par * 24 * MiB + mine * 3072, 3u << 20), r1 = mk(w1 + OFF_POUT + (size_t)par * 24 * MiB + mine * 3072, 3u << 20);
    const rsrc_t c0 = mk(w0 + OFF_POUTC + (size_t)par * 16 * MiB + mine * 2048, 2u << 20), c1 = mk(w1 + OFF_POUTC + (size_t)par * 16 * MiB + mine * 2048, 2u << 20);
    const rsrc_t rh = mk(ws + OFF_HOUT + (size_t)(par * 8 + col * 2) * 8192, 16384), ro = mk(p.out, 16u << 20);
    const unsigned lo = (ln & 15) * SB_ + (ln >> 4) * 16, ho = (ln >> 5) * 8192;
#define OENT(L) (((((L) >> 2) & 3) << 8) | ((7 - (((L) >> 6) & 7)) << 5) | ((((L) >> 5) & 1) << 4) | (((L) & 3) << 2) | ((((L) >> 4) & 1) << 1))
    u32x4 hv; Unit da[2], db[2]; unsigned cv, Ea = OENT(tail ? NM - RC + widx : widx), Eb, Ln, ha[2], hb[2], hn[2]; bool good = true;
    zero_scr(scr, ln);
    if (col != (unsigned)rcol && !tail) {
      const unsigned sg = ((col ^ rcol) - 1) * 2 + (ln & 1) + widx % PW * 6;
      const u64 t0 = __builtin_amdgcn_s_memrealtime();
      while (__builtin_amdgcn_ballot_w64(aload(ws + OFF_HAND + 2048 + sg * 4) != call) && !give_up(ws, call, t0)) { }
    }
    if (tail) {
      const u64 t0 = __builtin_amdgcn_s_memrealtime();
      while (!RFL(seen[wid - 3]) && !give_up(ws, call, t0)) { }
    }
    STAMP(11);
#define OUT_PAIR() do { good = wait_hdr(hv, rh, ho, Ea * 8, ws, call) && good && !__builtin_amdgcn_ballot_w64(((hv[1] | hv[3]) >> 24) != 0); \
      ha[0] = RL(hv[1], 0); ha[1] = RL(hv[1], 32); hn[0] = RL(hv[3], 0); hn[1] = RL(hv[3], 32); \
      load_unit(da[0], r0, Ea * 3072, ha[0], false, ln); load_unit(da[1], r1, Ea * 3072, ha[1], false, ln); CLAIM(cv, ctr, !tail); } while (0)
    hv = ld16(rh, ho, Ea * 8); OUT_PAIR();
    STAMP(10);
#define OUT_WAIT(d) asm volatile("s_waitcnt vmcnt(0)" : "+v"(d[0].a0), "+v"(d[0].a1), "+v"(d[0].p), "+v"(d[0].pt), "+v"(d[1].a0), "+v"(d[1].a1), "+v"(d[1].p), "+v"(d[1].pt), "+v"(hv), "+v"(cv) :: "memory")
#define OUT_SUM(d, Ec, h, odd) do { FENCE; unsigned T0[8], T1[8]; \
      unpack_hi(T0, d[0], h[0], odd, ln, scr, c0, Ec * 2048); unpack_hi(T1, d[1], h[1], odd, ln, scr, c1, Ec * 2048); \
      const unsigned _ob = ((col >> 1) * 512 + (Ec >> 9) * 256 + ((Ec >> 1) & 1) * 128 + ((Ec >> 4) & 1) * 64 + (Ec & 1) * 32) * SB_ + ((col & 1) * 4096 + ((Ec >> 5) & 15) * 256 + ((Ec >> 2) & 3) * 32) * 2; \
      _Pragma("unroll") for (int c = 0; c < 4; ++c) { const u32x4 x = unpack_row(T0, d[0], c, h[0]), y = unpack_row(T1, d[1], c, h[1]); u32x4 o; \
        _Pragma("unroll") for (int e = 0; e < 4; ++e) { \
          const float v0 = __builtin_bit_cast(float, x[e] << 16) + __builtin_bit_cast(float, y[e] << 16), v1 = __builtin_bit_cast(float, x[e] & 0xFFFF0000u) + __builtin_bit_cast(float, y[e] & 0xFFFF0000u); \
          o[e] = __builtin_bit_cast(unsigned, __builtin_elementwise_max(__builtin_bit_cast(s16x2, cvt2(v0, v1)), (s16x2){0, 0})); } \
        if (!good) o = (u32x4){0x7FC07FC0u, 0x7FC07FC0u, 0x7FC07FC0u, 0x7FC07FC0u}; \
        __builtin_amdgcn_raw_buffer_store_b128(o, ro, lo + _ob + (c >> 1) * 16 * SB_ + (c & 1) * 256, 0, 16); } \
      FENCE; } while (0)
    do {
      OUT_WAIT(da);
      Eb = Ea + 1; hb[0] = hn[0]; hb[1] = hn[1];
      load_unit(db[0], r0, Eb * 3072, hb[0], true, ln); load_unit(db[1], r1, Eb * 3072, hb[1], true, ln);
      Ln = tail ? NM : RFL(cv) - qb; hv = ld16(rh, ho, OENT(Ln) * 8);
      if (RT && !tail && Ln >= NM - TGATE) seen[wid + 1] = 1;
      OUT_SUM(da, Ea, ha, false);
      OUT_WAIT(db);
      db[0].pt = ln < 16 ? da[0].pt : db[0].pt; db[1].pt = ln < 16 ? da[1].pt : db[1].pt;
      Ea = OENT(Ln);
      if (Ln < NM) OUT_PAIR();
      OUT_SUM(db, Eb, hb, true);
    } while (Ln < NM);
  }
  STAMP(7);
}

extern "C" void kernel_launch(int rank, int nranks, ncclComm_t comm,
                              hipStream_t stream,
                              void* const* d_in, const int* in_sizes, int n_in,
                              void* d_out, int out_size,
                              const int* in_dtypes, int out_dtype,
                              void* const* d_out_all,
                              void* const* d_ws_all, size_t ws_bytes) {
  (void)comm; (void)in_dtypes; (void)out_dtype; (void)d_out_all;
  if (nranks != 8 || (unsigned)rank >= 8u || n_in != 2 || ws_bytes < WS_NEED || in_sizes[0] != 8192 * 1024 || in_sizes[1] != 8192 * 1024 || out_size != 1024 * 8192) {
    fprintf(stderr, "gemm_rs: unexpected problem shape\n"); return; }
  HIP_CHECK(hipFuncSetAttribute((const void*)gemm_rs, hipFuncAttributeMaxDynamicSharedMemorySize, LDS_BYTES));
  Params p;
  p.xin = (const char*)d_in[0]; p.win = (const char*)d_in[1]; p.out = (char*)d_out; p.own = (char*)d_ws_all[rank]; p.rank = rank; p.pad = 0;
  for (int i = 0; i < 8; ++i) p.ws[i] = (char*)d_ws_all[i];
  gemm_rs<<<NWG, 512, LDS_BYTES, stream>>>(p);
  HIP_CHECK(hipGetLastError());
}
```

```cpp
#include <hip/hip_runtime.h>
#include <rccl/rccl.h>
#include <cstdio>

#define HIP_CHECK(expr) do { hipError_t _e = (expr); if (_e != hipSuccess) { \
    fprintf(stderr, "HIP error %s at %s:%d\n", hipGetErrorString(_e), __FILE__, __LINE__); \
    return; } } while (0)

#define LAS __attribute__((address_space(3)))
#define GAS __attribute__((address_space(1)))
typedef short bf16x8 __attribute__((ext_vector_type(8)));
typedef short s16x4 __attribute__((ext_vector_type(4)));
typedef short s16x2 __attribute__((ext_vector_type(2)));
typedef unsigned short u16x2 __attribute__((ext_vector_type(2)));
typedef __bf16 bf16x2 __attribute__((ext_vector_type(2)));
typedef float f32x2 __attribute__((ext_vector_type(2)));
typedef float f32x4 __attribute__((ext_vector_type(4)));
typedef unsigned u32x2 __attribute__((ext_vector_type(2)));
typedef unsigned u32x3 __attribute__((ext_vector_type(3)));
typedef unsigned u32x4 __attribute__((ext_vector_type(4)));
typedef unsigned long long u64;
typedef __amdgpu_buffer_rsrc_t rsrc_t;

#define PW 24
#define NRED 4
#define RC 28
#define NPUSH 8
#define SPLIT1 76
#define SPLIT2 152
#define PAUSE2 6
#define PAUSE3 17
#define GATE 16
#define DELAYS 2, 14, 12, 10, 7, 5, 3, 0
#define PACKS 4, 4, 0, 8, 8, 3, 0, 0
#define SKIP0 4
#define PSL 8
#define GSL 2
#define HST 128
constexpr int BTH = 8, BUP = 3;

#define EPI2 1
constexpr int NENT = 256 - NPUSH, NP0 = 28 - SKIP0;
struct alignas(16) Order { unsigned char e[256]; short pk[8][8]; };
constexpr Order make_order() {
  Order o{}; const int dl[8] = {DELAYS}; int n = 0;
  for (int key = 0; key < 96; ++key)
    for (int op = 1; op >= 0; --op)
      for (int k = 0; k < 16; ++k)
        for (int j = 0; j < 8; ++j) {
          const int tau = op ? (((k & 1) << 3) | (7 - (k >> 1))) : k;
          int kk = 3 * j + (op ? dl[tau & 7] : k < 6 ? 0 : k < 11 ? 1 : 2);
          if (op && (tau & 7) == 0 && 3 * j >= PAUSE3 - 7) kk = 80 + j;
          if (op && (tau & 7) == 1 && 3 * j < PAUSE2 - 7) kk = 3 * j;
          if (kk == key && (op || j || k >= NPUSH)) o.e[n++] = (unsigned char)(op << 7 | tau << 3 | j);
        }
  const int pc[8] = {PACKS}; n = 8 * NP0 - 6 * NPUSH;
  for (int k = 0; k < 8; ++k)
    for (int g = 7; g >= 0; --g) { o.pk[g][k] = (short)(pc[g] > k ? n : 1024); n += pc[g] > k ? 32 : 0; }
  return o;
}
constexpr Order ORDER_H = make_order();
constexpr bool order_ok() {
  for (int i = SPLIT1; i < NENT; ++i) { const int e = ORDER_H.e[i], op = e >> 7, tau = (e >> 3) & 15, j = e & 7;
    if ((!op || (tau & 7) == 1) && 3 * j < PAUSE2 - 7) return false;
    if (i >= SPLIT2 && (!op || (tau & 7) == 0) && 3 * j < PAUSE3 - 7) return false; }
  const int pc[8] = {PACKS}; int n = 8 * NP0 - 6 * NPUSH;
  for (int g = 0; g < 8; ++g) n += 32 * pc[g];
  return n >= 4 * NENT && pc[2] == 0 && pc[0] && pc[1] && (SKIP0 == 0 || pc[7] == 0) && (SKIP0 <= 4 || pc[6] == 0) && SKIP0 <= 8;
}
constexpr unsigned more_mask(int k) { unsigned m = 0; for (int g = 0; g < 8; ++g) m |= (ORDER_H.pk[g][k] < 4 * NENT ? 1u : 0u) << g; return m; }
static_assert(order_ok(), "a shift would wait for blocks it has to pull itself, or blocks stay unpacked");
static_assert(NRED >= 2 && NRED <= 8 && RC <= 16 * NRED && PW <= 42 && NPUSH >= 0 && NPUSH <= 8 && SPLIT1 * 2 >= PW && (SPLIT2 - SPLIT1) * 2 >= PW && (NENT - SPLIT2) * 2 >= PW, "roles");
__constant__ Order ORDER = ORDER_H;

constexpr int NWG = 256, HTB = 16384;
constexpr unsigned SA_ = 2048, SB_ = 16384;
constexpr size_t MiB = 1ull << 20;
constexpr size_t OFF_CALL = 0;
constexpr size_t OFF_TMO = 32768, OFF_Q = 33024, OFF_HAND = 524288;
constexpr size_t OFF_RDY = 40960;
constexpr size_t OFF_HIN = 65536;
constexpr size_t OFF_HOUT = OFF_HIN + 262144;
constexpr size_t OFF_STG = 1 * MiB;
constexpr size_t OFF_PIN = 73 * MiB;
constexpr size_t OFF_PINC = 121 * MiB;
constexpr size_t OFF_POUT = 153 * MiB;
constexpr size_t OFF_POUTC = 201 * MiB;
constexpr size_t OFF_DBG = 233 * MiB;
constexpr size_t WS_NEED = 234 * MiB;
constexpr int SCR = 2448, LDS_BYTES = 8 * HTB + 1024 + 8 * SCR;
constexpr unsigned OOB = 0x7FFFFF00u;
constexpr u64 DEADLINE = 100000000ull;

struct Params { const char* xin; const char* win; char* out; char* own; char* ws[8]; int rank; int pad; };

#define RFL(x) __builtin_amdgcn_readfirstlane(x)
#define WAIT_V(n) asm volatile("s_waitcnt vmcnt(" #n ")" ::: "memory")
#define WAIT_L(n) asm volatile("s_waitcnt lgkmcnt(" #n ")" ::: "memory")
#define DRAIN do { asm volatile("" ::: "memory"); __builtin_amdgcn_s_waitcnt(0x0F70); asm volatile("" ::: "memory"); } while (0)
#define BAR __builtin_amdgcn_s_barrier()
#define SCHED __builtin_amdgcn_sched_barrier(0)
#define FENCE asm volatile("" ::: "memory")
#define DI __device__ __forceinline__
constexpr int SYS = 17;

DI rsrc_t mk(const void* p, unsigned bytes) { return __builtin_amdgcn_make_buffer_rsrc((void*)p, (short)0, (int)bytes, 0x00020000); }
DI u32x4 ld16(rsrc_t r, unsigned v, unsigned s) { return __builtin_amdgcn_raw_buffer_load_b128(r, v, s, SYS); }
DI u32x4 ld16p(rsrc_t r, unsigned v, unsigned s) { return __builtin_amdgcn_raw_buffer_load_b128(r, v, s, 0); }
DI void st16(u32x4 x, rsrc_t r, unsigned v, unsigned s) { __builtin_amdgcn_raw_buffer_store_b128(x, r, v + s, 0, SYS); }
DI unsigned aload(const char* p) { return __hip_atomic_load((const GAS unsigned*)p, __ATOMIC_RELAXED, __HIP_MEMORY_SCOPE_SYSTEM); }
DI void astore(char* p, unsigned v) { __hip_atomic_store((GAS unsigned*)p, v, __ATOMIC_RELAXED, __HIP_MEMORY_SCOPE_SYSTEM); }
DI unsigned perm(unsigned hi, unsigned lo, unsigned sel) { return __builtin_amdgcn_perm(hi, lo, sel); }
DI unsigned cvt2(float lo, float hi) { return __builtin_bit_cast(unsigned, __builtin_convertvector((f32x2){lo, hi}, bf16x2)); }

#define STAMP(i) do { } while (0)

DI bool give_up(char* ws, unsigned call, u64 t0) {
  const u64 dt = __builtin_amdgcn_s_memrealtime() - t0;
  if (dt > 10000) {
    if (RFL(aload(ws + OFF_TMO)) == call) return true;
    if (dt > DEADLINE) { astore(ws + OFF_TMO, call); return true; }
  }
  __builtin_amdgcn_s_sleep(GSL);
  return false;
}

struct Unit { u32x4 a0, a1; u32x3 p; unsigned pt; };
DI unsigned bfi(unsigned m, unsigned a, unsigned b) { unsigned r; asm("v_bfi_b32 %0, %1, %2, %3" : "=v"(r) : "s"(m), "v"(a), "v"(b)); return r; }
DI unsigned pka(unsigned a, unsigned b) { return __builtin_bit_cast(unsigned, __builtin_bit_cast(u16x2, a) + __builtin_bit_cast(u16x2, b)); }
DI unsigned pks(unsigned a, unsigned b) { return __builtin_bit_cast(unsigned, __builtin_bit_cast(u16x2, a) - __builtin_bit_cast(u16x2, b)); }
#define LANES(c) __builtin_popcountll(__builtin_amdgcn_ballot_w64(c))
DI unsigned pick_base(unsigned f) {
  unsigned g = RFL(f) & ~7u;
  while (LANES(f >= g + 8) >= BTH) g += 8;
  while (g && LANES(f >= g) < BTH) g -= 8;
#pragma unroll
  for (int bit = 4; bit; bit >>= 1) g = LANES(f >= g + bit) >= BTH ? g + bit : g;
  return g > 64 - BUP ? g - (64 - BUP) : 0;
}
DI unsigned base_of(const u32x4 (&r)[4]) {
  u16x2 mx = {0, 0};
#pragma unroll
  for (int c = 0; c < 4; ++c)
#pragma unroll
    for (int i = 0; i < 4; i += 2) mx = __builtin_elementwise_max(mx, __builtin_bit_cast(u16x2, r[c][i] & 0x7FFF7FFFu));
  return pick_base((mx.x > mx.y ? mx.x : mx.y) >> 4);
}
DI unsigned pack_unit(const u32x4 (&r)[4], unsigned base, bool odd, int lane, LAS unsigned char* scr, rsrc_t dst, unsigned doff, rsrc_t dstc, unsigned coff) {
  const unsigned B2 = base * 0x00100010u, M = 0x83838383u;
  unsigned A[8], T[8], mask = 0, n = 0;
#pragma unroll
  for (int k = 0; k < 8; ++k) {
    const unsigned lo = pks(r[k >> 1][(k & 1) * 2], B2), hi = pks(r[k >> 1][(k & 1) * 2 + 1], B2);
    A[k] = perm(hi, lo, 0x06040200u); T[k] = perm(hi, lo, 0x07050301u);
    mask = bfi(0x80808080u, (T[k] & 0x7C7C7C7Cu) + 0x7C7C7C7Cu, mask >> 1);
  }
  *(LAS u32x4*)(scr + lane * 32) = (u32x4){T[0], T[1], T[2], T[3]}; *(LAS u32x4*)(scr + lane * 32 + 16) = (u32x4){T[4], T[5], T[6], T[7]};
  st16((u32x4){A[0], A[1], A[2], A[3]}, dst, lane * 16, doff);
  st16((u32x4){A[4], A[5], A[6], A[7]}, dst, lane * 16 + 1024, doff);
  const unsigned P0 = bfi(0x60606060u, T[6] << 5, bfi(M, T[0], __builtin_rotateleft32(T[1], 3)));
  const unsigned P1 = bfi(0x60606060u, T[7] << 5, bfi(M, T[2], __builtin_rotateleft32(T[3], 3)));
  const unsigned P2 = bfi(0x40404040u, T[6] >> 1, bfi(0x20202020u, T[7] >> 2, bfi(M, T[4], __builtin_rotateleft32(T[5], 3))));
  __builtin_amdgcn_raw_buffer_store_b64((u32x2){P0, P1}, dst, lane * 8 + 2048 + doff, 0, SYS);
  __builtin_amdgcn_raw_buffer_store_b32(P2, dst, lane * 4 + 2560 + doff, 0, SYS);
  FENCE;
  for (u64 m; n <= 96 && (m = __builtin_amdgcn_ballot_w64(mask != 0)) != 0; n += __builtin_popcountll(m)) {
    const unsigned i = __builtin_amdgcn_mbcnt_hi((unsigned)(m >> 32), __builtin_amdgcn_mbcnt_lo((unsigned)m, n));
    const unsigned b = __builtin_ctz(mask | 0x80000000u), a = lane * 32 + ((b & 7) << 2 | b >> 3);
    *(LAS unsigned short*)(scr + 2048 + (mask ? i : 192u) * 2) = (unsigned short)((scr[a] & 0x7Cu) << 9 | a);
    mask &= mask - 1;
  }
  FENCE;
  if (n > 96) {
    n = 255;
    st16((u32x4){T[0], T[1], T[2], T[3]} & 0x7C7C7C7Cu, dstc, lane * 16, coff);
    st16((u32x4){T[4], T[5], T[6], T[7]} & 0x7C7C7C7Cu, dstc, lane * 16 + 1024, coff);
  } else __builtin_amdgcn_raw_buffer_store_b32(*(const LAS unsigned*)(scr + 2048 + lane * 4), dst, 2u * lane < n ? lane * 4 + doff + (odd && lane < 16 ? -256 : 2880) : OOB, 0, SYS);
  FENCE;
  return base | n << 12;
}
constexpr int NLD = 5;
DI void load_unit(Unit& d, rsrc_t src, unsigned off, unsigned hw, bool odd, int lane) {
  const unsigned n = (hw >> 12) & 127u;
  d.pt = __builtin_amdgcn_raw_buffer_load_b32(src, odd ? (lane >= 16 && lane < 48 && 2u * lane < n ? lane * 4 + 2880 : OOB) : (2u * lane < (n > 32 ? n : 32) + 32 ? lane * 4 + 2816 : OOB), off, SYS);
  const u32x2 q = __builtin_amdgcn_raw_buffer_load_b64(src, lane * 8 + 2048, off, SYS);
  d.p = (u32x3){q[0], q[1], __builtin_amdgcn_raw_buffer_load_b32(src, lane * 4 + 2560, off, SYS)};
  d.a0 = ld16(src, lane * 16, off); d.a1 = ld16(src, lane * 16 + 1024, off);
}
DI void unpack_hi(unsigned (&T)[8], const Unit& d, unsigned hw, bool odd, int lane, LAS unsigned char* scr, rsrc_t srcc, unsigned coff) {
  const unsigned n = (hw >> 12) & 255u, M = 0x83838383u, e = 2u * lane - (odd ? 0u : 32u);
  u32x4 x0, x1;
  if (n > 96) { x0 = ld16(srcc, lane * 16, coff); x1 = ld16(srcc, lane * 16 + 1024, coff); DRAIN; }
  else {
    const unsigned a0 = d.pt & 0x7FFu, a1 = (d.pt >> 16) & 0x7FFu;
    if (e < n) scr[a0] = (unsigned char)((d.pt >> 9) & 0x7Cu);
    if (e + 1 < n) scr[a1] = (unsigned char)((d.pt >> 25) & 0x7Cu);
    FENCE;
    x0 = *(const LAS u32x4*)(scr + lane * 32); x1 = *(const LAS u32x4*)(scr + lane * 32 + 16);
    FENCE;
    if (e < n) { scr[a0] = 0; scr[a1] = 0; }
    FENCE;
  }
  T[0] = (d.p[0] & M) | x0[0]; T[1] = (__builtin_rotateright32(d.p[0], 3) & M) | x0[1];
  T[2] = (d.p[1] & M) | x0[2]; T[3] = (__builtin_rotateright32(d.p[1], 3) & M) | x0[3];
  T[4] = (d.p[2] & M) | x1[0]; T[5] = (__builtin_rotateright32(d.p[2], 3) & M) | x1[1];
  T[6] = (bfi(0x03030303u, d.p[0] >> 5, d.p[2] << 1) & M) | x1[2];
  T[7] = (bfi(0x03030303u, d.p[1] >> 5, d.p[2] << 2) & M) | x1[3];
}
DI u32x4 unpack_row(const unsigned (&T)[8], const Unit& d, int c, unsigned hw) {
  const unsigned B2 = (hw & 0xFFFu) * 0x00100010u, a = (c & 2 ? d.a1 : d.a0)[(c & 1) * 2], b = (c & 2 ? d.a1 : d.a0)[(c & 1) * 2 + 1];
  return (u32x4){pka(perm(T[2 * c], a, 0x05010400u), B2), pka(perm(T[2 * c], a, 0x07030602u), B2), pka(perm(T[2 * c + 1], b, 0x05010400u), B2), pka(perm(T[2 * c + 1], b, 0x07030602u), B2)};
}
DI void zero_scr(LAS unsigned char* scr, int lane) { *(LAS u32x4*)(scr + lane * 32) = (u32x4){0u, 0u, 0u, 0u}; *(LAS u32x4*)(scr + lane * 32 + 16) = (u32x4){0u, 0u, 0u, 0u}; FENCE; }
DI bool wait_hdr(u32x4& h, rsrc_t rh, unsigned voff, unsigned soff, char* ws, unsigned call) {
  if (!__builtin_amdgcn_ballot_w64(h[0] != call || h[2] != call)) return true;
  const u64 t0 = __builtin_amdgcn_s_memrealtime();
  for (;;) {
    h = ld16(rh, voff, soff); FENCE;
    if (!__builtin_amdgcn_ballot_w64(h[0] != call || h[2] != call)) return true;
    if (give_up(ws, call, t0)) { h = (u32x4){0u, 0u, 0u, 0u}; return false; }
  }
}
#define CLAIM(cv, base, on) do { cv = 1u; asm volatile("s_mov_b32 exec_lo, %3\n\ts_mov_b32 exec_hi, 0\n\ts_nop 2\n\tglobal_atomic_add %0, %1, %0, %2 sc0\n\ts_mov_b64 exec, -1" : "+v"(cv) : "v"(0u), "s"(base), "s"(RFL((on) ? 1u : 0u)) : "memory"); } while (0)
#define RL(x, l) ((unsigned)__builtin_amdgcn_readlane((int)(x), (l)))

__global__ __launch_bounds__(512, 2) void gemm_rs(Params p) {
  extern __shared__ __attribute__((aligned(16))) unsigned char smem[];
  LAS unsigned char* lds = (LAS unsigned char*)smem;
  const int tid = threadIdx.x, wid = RFL(tid >> 6), lane = tid & 63, wr = wid >> 2, wc = wid & 3, fr = lane & 15, fq = lane >> 4;
  unsigned tab = ((const unsigned*)ORDER.e)[lane];
  asm volatile("" :: "s"(p.xin), "s"(p.win), "s"(p.own), "s"(p.rank), "s"(p.ws[(blockIdx.x & 3) * 2 + ((blockIdx.x >> 2) & 1)]) : "memory");
  const int rank = p.rank & 7, ra = rank >> 2, rb = (rank >> 1) & 1, rc = rank & 1, rcol = rank >> 1;
  char* const ws = p.own;
  const int bid = blockIdx.x, xcd = bid & 7, idx = bid >> 3;
  const int tm = (xcd >> 1) * 4 + (idx & 3), tn = (xcd & 1) * 8 + (idx >> 2);
  const int shift = idx < 12 ? 3 - (idx >> 2) : 0;
  LAS unsigned char* const scr = lds + 8 * HTB + 1024 + wid * SCR;
  STAMP(0);
  unsigned callv = aload(ws + OFF_CALL + bid * 128);
  volatile LAS int* const seen = (volatile LAS int*)(lds + 8 * HTB + 896);
  if (tid == 0) *seen = 0;

  const unsigned ldsw = (unsigned)(size_t)lds + (unsigned)wid * 1024u;
  unsigned vA[4], vB[4];
  { const unsigned voffA = (wid * 8 + (lane >> 3)) * SA_ + 16 * ((lane & 7) ^ (lane >> 3));
    const int s = lane >> 3, pos = (lane & 7) >> 1, hf = lane & 1, cp = pos ^ (s >> 1);
    const unsigned voffB = (16 * (wid >> 2) + 8 * (s >> 2) + 4 * ((wid >> 1) & 1) + (s & 3)) * SB_ + (64 * (wid & 1) + 16 * cp + 8 * hf) * 2;
#pragma unroll
    for (int i = 0; i < 4; ++i) { vA[i] = voffA + i * 64 * SA_; vB[i] = voffB + (i & 1) * 32 * SB_ + (i >> 1) * 256; asm volatile("" : "+v"(vA[i]), "+v"(vB[i])); } }
#define SA(b, h) (((b) * 2 + (h)) * HTB)
#define SB(b, h) ((4 + (b) * 2 + (h)) * HTB)
#define GLDS(voff, sbase, c) asm volatile("s_add_i32 m0, %2, %3\n\ts_nop 0\n\tglobal_load_lds_dwordx4 %0, %1 sc0 sc1" :: "v"(voff), "s"(sbase), "s"(ldsw), "i"(c) : "memory", "scc")
#define STAGEA(b, h, gbase) do { GLDS(vA[2 * (h)], gbase, SA(b, h)); GLDS(vA[2 * (h) + 1], gbase, SA(b, h) + 8192); } while (0)
#define STAGEB(b, h, gbase) do { GLDS(vB[2 * (h)], gbase, SB(b, h)); GLDS(vB[2 * (h) + 1], gbase, SB(b, h) + 8192); } while (0)
  const char* const ownA = p.xin + (size_t)((tm >> 1) * 1024 + ra * 512 + (tm & 1) * 256) * SA_;
  const char* const ownB = p.win + (size_t)(rb * 4096 + tn * 256) * 2;
  const char* a1 = ownA + 128;

  unsigned call, par;
  {
    const unsigned loX = (lane >> 3) * SA_ + (lane & 7) * 16, loW = (lane >> 5) * SB_ + (lane & 31) * 16;
    const int pd = (xcd & 3) * 2 + (xcd >> 2), pq = ((xcd & 3) ^ rcol) - 1;
    const bool push = idx < NPUSH && pq >= 0;
    const bool more = (more_mask(0) >> (idx >> 2)) & 1;
    int blk[4]; bool val[4];
#define PK_BLOCK(k, g) do { val[k] = (g) < 4 * NENT; const int _g = val[k] ? (g) : (xcd >> 1); \
      const int _e = val[k] ? (RL(tab, _g >> 4) >> ((_g >> 2) & 3) * 8) & 255 : (idx & 15) << 3, _op = _e >> 7, _tau = (_e >> 3) & 15, _hv = _g & 1; \
      blk[k] = _op * 512 + (((_g >> 1) & 1) * 8 + (_e & 7)) * 32 + (_op ? _hv * 16 + _tau : (_tau >> 1) * 4 + _hv * 2 + (_tau & 1)); } while (0)
    const bool none = shift == 1 || idx >= 32 - SKIP0;
    PK_BLOCK(0, push || none ? 1024 : pq < 0 ? (idx < 8 ? idx : idx - 4) * 2 + (xcd >> 2) : 2 * NP0 + ((idx < 8 ? idx : idx - 4) - NPUSH) * 6 + pq * 2 + (xcd >> 2));
    u32x4 r[4][4]; int hl = 8 * HTB;
#define PK_LOAD(k, on) do { const int _op = blk[k] >> 9, _jj = (blk[k] >> 5) & 15, _ix = blk[k] & 31; \
      const unsigned _so = _op ? (_jj * 64 + wid * 8) * SB_ + _ix * 512 : (_ix * 256 + wid * 32) * SA_ + _jj * 128, _cs = _op ? 2 * SB_ : 8 * SA_; \
      _Pragma("unroll") for (int c = 0; c < 4; ++c) r[k][c] = ld16p(mk(_op ? p.win : p.xin, on ? 16u << 20 : 0u), _op ? loW : loX, _so + c * _cs); } while (0)
#define PK_PACK(k) do { const unsigned _h = pack_unit(r[k], base_of(r[k]), wid & 1, lane, scr, rpi, blk[k] * 24576 + wid * 3072, rpc, blk[k] * 16384 + wid * 2048); \
      if (lane == 0) *(LAS unsigned*)(lds + hl + (k) * 32 + wid * 4) = _h; } while (0)
#define PK_SEND(k) do { const int _op = blk[k] >> 9, _jj = (blk[k] >> 5) & 15, _ix = blk[k] & 31, _d = (lane >> 3) & 1; \
      const int _tau = _op ? (_ix & 15) : ((_ix >> 2) * 2 + (_ix & 1)); \
      const int _c0 = _op ? (_ix >> 4) : (_ix & 2), _c1 = _op ? _c0 + 2 : _c0 + 1; \
      char* const _w0 = p.ws[(_c0 * 2 + (_jj >> 3)) & 7]; char* const _w1 = p.ws[(_c1 * 2 + (_jj >> 3)) & 7]; \
      if (lane < 16 && (_d ? _c1 : _c0) != rcol) { const u32x2 _h = {call, *(const LAS unsigned*)(lds + hl + (k) * 32 + (lane & 7) * 4)}; \
        char* _dst = (_d ? _w1 : _w0) + OFF_HIN + (size_t)((par * 8 + rank) * 256 + _op * 128 + _tau * 8 + (_jj & 7)) * 64 + (lane & 7) * 8; \
        asm volatile("global_store_dwordx2 %0, %1, off sc0 sc1\n\ts_nop 1" :: "v"(_dst), "v"(_h) : "memory"); } } while (0)
#define PROLOGUE() do { STAGEB(0, 0, ownB); STAGEA(0, 0, ownA); STAGEB(0, 1, ownB); STAGEA(0, 1, ownA); \
      STAGEB(1, 0, ownB + 64 * SB_); STAGEA(1, 0, a1); STAGEB(1, 1, ownB + 64 * SB_); } while (0)
    if (none && shift != 1) PROLOGUE();
    PK_LOAD(0, !none);
    asm volatile("" : "+v"(callv) :: "memory");
    call = RFL(callv) + 1; par = call & 1;
    const rsrc_t rpi = mk(ws + OFF_PIN + (size_t)par * 24 * MiB, 24u << 20), rpc = mk(ws + OFF_PINC + (size_t)par * 16 * MiB, 16u << 20);
    asm volatile("" : "+v"(tab), "+v"(r[0][0]), "+v"(r[0][1]), "+v"(r[0][2]), "+v"(r[0][3]));
    if (push) {
      const rsrc_t rq = mk(p.ws[pd] + OFF_STG + (size_t)par * 48 * MiB, 24u << 20);
#pragma unroll
      for (int c = 0; c < 4; ++c) st16(r[0][c], rq, loX, (pq * 4096 + idx * 256 + wid * 32 + c * 8) * SA_ + rc * 1024);
      FENCE;
    } else if (none) { WAIT_V(0); WAIT_L(0); BAR; }
    else {
      PK_PACK(0);
      if (more) WAIT_V(0); else { PROLOGUE(); WAIT_V(14); }
      WAIT_L(0); BAR; FENCE;
      if (wid == 0) PK_SEND(0);
      if (!more) { if (wid == 0) WAIT_V(1); else WAIT_V(0); BAR; }
    }
    STAMP(1);
    if (more) {
      for (int q = 0; q < 8; q += 4) {
        const bool last = q == 4 || !((more_mask(4) >> (idx >> 2)) & 1);
        hl += 128;
#pragma unroll
        for (int k = 0; k < 4; ++k) { PK_BLOCK(k, ORDER.pk[idx >> 2][q + k] + (bid & 31)); PK_LOAD(k, val[k]); }
        FENCE;
        if (val[0]) { PK_PACK(0); if (val[1]) { PK_PACK(1); if (val[2]) { PK_PACK(2); if (val[3]) PK_PACK(3); } } }
        if (last) PROLOGUE();
        DRAIN; WAIT_L(0); BAR; FENCE;
#pragma unroll
        for (int k = 0; k < 4; ++k) if (wid == k + 1 && val[k]) PK_SEND(k);
        if (q == 0 && (tid >> 2) == 80 && push) astore(p.ws[pd] + OFF_RDY + (idx * 48 + pq * 2 + rc) * 16 + (tid & 3) * 4, call);
        if (last) break;
      }
    }
    if (tid == 384) astore(ws + OFF_CALL + bid * 128, call);
    if (shift != 1 && wr == 1) BAR;
  }
  STAMP(2);

  int aoff0 = (wr * 64 + fr) * 128 + 16 * (fq ^ (fr & 7)), aoff1 = aoff0 ^ 64, boff0, boff1;
  { const int s = (lane >> 2) & 7, pp = ((lane & 1) << 1) | ((lane >> 1) & 1), gh = lane >> 5;
    boff0 = 4 * HTB + gh * 4096 + (wc >> 1) * 1024 + 128 * s + 32 * ((2 * (wc & 1)) ^ (s >> 1)) + 8 * pp; boff1 = boff0 ^ 32; }
  asm volatile("" : "+v"(aoff0), "+v"(aoff1), "+v"(boff0), "+v"(boff1));
#define LDA(dst, b, h) do { _Pragma("unroll") for (int m = 0; m < 4; ++m) { dst[m][0] = *(const LAS bf16x8*)(lds + SA(b, h) + aoff0 + m * 2048); dst[m][1] = *(const LAS bf16x8*)(lds + SA(b, h) + aoff1 + m * 2048); } } while (0)
#define TR(off) __builtin_amdgcn_ds_read_tr16_b64_v4i16((LAS s16x4*)(lds + (off)))
#define LDB(dst, b, h) do { _Pragma("unroll") for (int k = 0; k < 2; ++k) { \
    dst[0][k] = __builtin_shufflevector(TR(SA(b, h) + boff0 + k * 8192), TR(SA(b, h) + boff0 + k * 8192 + 2048), 0, 1, 2, 3, 4, 5, 6, 7); \
    dst[1][k] = __builtin_shufflevector(TR(SA(b, h) + boff1 + k * 8192), TR(SA(b, h) + boff1 + k * 8192 + 2048), 0, 1, 2, 3, 4, 5, 6, 7); } } while (0)
#define MMA(ai, bj, At, Bt) do { __builtin_amdgcn_s_setprio(1); _Pragma("unroll") for (int m = 0; m < 4; ++m) _Pragma("unroll") for (int n = 0; n < 2; ++n) _Pragma("unroll") for (int k = 0; k < 2; ++k) \
    acc[ai][bj][m][n] = __builtin_amdgcn_mfma_f32_16x16x32_bf16(Bt[n][k], At[m][k], acc[ai][bj][m][n], 0, 0, 0); __builtin_amdgcn_s_setprio(0); } while (0)
  f32x4 acc[2][2][4][2];
#pragma unroll
  for (int a = 0; a < 2; ++a)
#pragma unroll
    for (int b = 0; b < 2; ++b)
#pragma unroll
      for (int m = 0; m < 4; ++m)
#pragma unroll
        for (int n = 0; n < 2; ++n) acc[a][b][m][n] = (f32x4){0.f, 0.f, 0.f, 0.f};
  bf16x8 At[4][2], B0[2][2], B1[2][2];
  const char* const sA = ws + OFF_STG + (size_t)par * 48 * MiB + (size_t)tm * 256 * SA_;
  const char* const sB = ws + OFF_STG + 24 * MiB + (size_t)tn * 512;
  const int pause = shift == 1 ? 0 : shift == 2 ? PAUSE2 : shift == 3 ? PAUSE3 : -1;
  const int s0 = tm < 6 ? 0 : tm < 11 ? 2 : 1;
  int j = 0, s2 = s0, ready = 0;
  unsigned fl = 0;
  const char* const rdy = ws + OFF_RDY;
#define FOFF() ([&] { int l = tid; asm volatile("" : "+v"(l)); const int k0 = ready + ((l >> 2) & 7), k = k0 < 47 ? k0 : 47; \
    return (unsigned)(((l & 32 ? 16 + tn : tm) * 48 + k + 2 * s0 - (k % 6 + 2 * s0 >= 6 ? 6 : 0)) * 16 + (l & 3) * 4); }())
#define READY() do { const u64 _m = __builtin_amdgcn_ballot_w64((int)(fl - call) < 0); ready += __builtin_ctzll((_m | _m >> 32 | 1ull << 32) & 0x1FFFFFFFFull) >> 2; } while (0)
  for (int it = 0; it < 32; ++it) {
    if (it == pause) {
      if (wr == 0 && shift != 1) BAR;
      STAMP(3);
      int ln = tid; asm volatile("" : "+v"(ln)); ln &= 63;
      const unsigned gw = wid * 32 + (((idx & 3) * 8 + xcd + wid) & 31), s = gw % 6, widx = gw / 6;
      if (widx < PW) {
      const unsigned lb = shift == 1 ? 0 : shift == 2 ? SPLIT1 * 4 : SPLIT2 * 4, le = shift == 1 ? SPLIT1 * 4 : shift == 2 ? SPLIT2 * 4 : NENT * 4;
      const unsigned qb = (call - 1) * (le - lb - PW) - lb - 2 * PW;
      char* const ctr = ws + OFF_Q + (s * 3 + shift - 1) * 128;
      const unsigned src = ((((unsigned)rcol ^ (1 + (s >> 1))) << 1) | (s & 1)) & 7;
      const char* const sw = p.ws[src];
      const unsigned lx = (ln >> 3) * SA_ + (ln & 7) * 16, lw = (ln >> 5) * SB_ + (ln & 31) * 16;
      const rsrc_t rsc = mk(sw + OFF_PINC + (size_t)par * 16 * MiB, 16u << 20), rh = mk(ws + OFF_HIN + (size_t)(par * 8 + src) * 16384, 16384), rd = mk(ws + OFF_STG, 72u << 20);
      zero_scr(scr, ln);
      { const unsigned xm = ((~tab >> 7) & (((tab & 0x07070707u) + 0x07070707u) >> 3) & 0x01010101u) * 0x78u;
        tab = (tab & ~xm) | (((tab & xm) + (s >> 1) * 0x58585858u % 0x80808080u) & xm); }
#define ENT(L) ((((RL(tab, ((L) >> 4) & 63) >> (((L) >> 2) & 3) * 8) & 255u) << 3) | ((L) & 3) << 1)
#define IN_SRC(E) (((E) >> 10) ? (512 + (rc * 8 + (((E) >> 3) & 7)) * 32 + rb * 16 + (((E) >> 6) & 15)) * 8 + ((E) & 7) \
                               : ((rc * 8 + (((E) >> 3) & 7)) * 32 + ((E) >> 7) * 4 + ra * 2 + (((E) >> 6) & 1)) * 8 + ((E) & 7))
#define SIGNAL(E) do { if (ln == 0) astore(ws + OFF_RDY + (((E) >> 6) * 48 + (((E) >> 3) & 7) * 6 + s) * 16 + ((E) & 6) * 2, call); } while (0)
      struct Buf { Unit d; unsigned Ec, hw; bool live; } A, B;
      unsigned cv = 0, keep, hn, Ep = ~0u, Ln = lb + PW + widx, En = ENT(Ln);
      A.Ec = ENT(lb + widx); B.Ec = A.Ec + 1; A.live = B.live = true;
      u32x4 hv = ld16(rh, 0, A.Ec * 8);
      if (shift != 1) { const u64 t0 = __builtin_amdgcn_s_memrealtime(); while (RFL(aload(ws + OFF_HAND + ((shift - 2) * 256 + gw) * HST)) != call && !give_up(ws, call, t0)) { } }
#define IN_LOAD(v, odd) do { FENCE; load_unit(v.d, mk(sw + OFF_PIN + (size_t)par * 24 * MiB, v.live ? 24u << 20 : 0u), IN_SRC(v.Ec) * 3072, v.hw, odd, ln); FENCE; } while (0)
      wait_hdr(hv, rh, 0, A.Ec * 8, ws, call); A.hw = RFL(hv[1]); B.hw = RFL(hv[3]); STAMP(8);
      IN_LOAD(A, false); hv = ld16(rh, 0, En * 8); IN_LOAD(B, true);
#define IN_STEP(v, odd) do { __builtin_amdgcn_s_waitcnt(0x0F70 | NLD); asm volatile("s_waitcnt vmcnt(%1)" : "+v"(cv) : "i"(NLD) : "memory"); \
        if (!odd && Ep != ~0u) SIGNAL(Ep); \
        if (odd) v.d.pt = ln < 16 ? keep : v.d.pt; else keep = v.d.pt; \
        unsigned T[8]; unpack_hi(T, v.d, v.hw, odd, ln, scr, rsc, IN_SRC(v.Ec) * 2048); \
        const unsigned _op = v.Ec >> 10, _tau = (v.Ec >> 6) & 15, _j = (v.Ec >> 3) & 7, _u = v.Ec & 7, _cs = _op ? 2 * SB_ : 8 * SA_, _lo = _op ? lw : lx; \
        const unsigned _db = _op ? 24 * (unsigned)MiB + ((s >> 1) * 512 + _j * 64 + _u * 8) * SB_ + (s & 1) * 8192 + _tau * 512 \
                                 : par * 48 * (unsigned)MiB + ((s >> 1) * 4096 + _tau * 256 + _u * 32) * SA_ + (s & 1) * 1024 + _j * 128; \
        _Pragma("unroll") for (int c = 0; c < 4; ++c) st16(unpack_row(T, v.d, c, v.hw), rd, _lo, _db + c * _cs); \
        FENCE; } while (0)
      do {
        IN_STEP(A, false);
        A.live = Ln < le;
        if (A.live) wait_hdr(hv, rh, 0, En * 8, ws, call);
        A.hw = RFL(hv[1]); hn = RFL(hv[3]); A.Ec = En;
        CLAIM(cv, ctr, A.live); IN_LOAD(A, false);
        IN_STEP(B, true);
        Ep = B.Ec;
        B.live = A.live; B.hw = hn; B.Ec = A.Ec + 1; Ln = RFL(cv) - qb; En = ENT(Ln);
        if (A.live && Ln >= le - (shift == 3 ? GATE : 0) && ln == 0) astore(ws + OFF_HAND + ((shift - 1) * 256 + gw) * HST, call);
        hv = ld16(rh, 0, En * 8); IN_LOAD(B, true);
      } while (A.live);
      DRAIN;
      SIGNAL(Ep);
      }
      if (shift == 1) { PROLOGUE(); WAIT_V(0); }
      STAMP(4);
      if (shift == 1) BAR;
      if (wr == 1) BAR;
    }
    if (it == 7) STAMP(12); if (it == 19) STAMP(13); if (it == 25) STAMP(14); if (it == 28) STAMP(15);
    const char *a2, *b2, *a3, *b3;
    if (it < 7 || it == 31) { const int t = (it == 31) ? 0 : 2 * it + 2; a2 = ownA + t * 128; b2 = ownB + (size_t)t * 64 * SB_; a3 = a2 + 128; b3 = b2 + 64 * SB_; }
    else {
      if (ready < 2 * it - 12) {
        ready = RFL(*seen);
        if (ready < 2 * it - 12) {
          const u64 t0 = __builtin_amdgcn_s_memrealtime();
          for (;;) {
            if (wid == 0) { fl = aload(rdy + FOFF()); READY(); } else ready = RFL(*seen);
            if (ready >= 2 * it - 12) break;
            if (give_up(ws, call, t0)) { ready = 64; break; }
            if (PSL && wid == 0) __builtin_amdgcn_s_sleep(PSL);
          }
        }
        if (wid == 0) *seen = ready;
        FENCE;
      }
      a2 = sA + (size_t)s2 * (4096 * SA_) + j * 128; b2 = sB + (size_t)(s2 * 512 + j * 64) * SB_; a3 = a2 + 1024; b3 = b2 + 8192; if (++s2 == 3) s2 = 0;
      if (s2 == s0) ++j;
    }
    LDB(B0, 0, 0); SCHED; LDA(At, 0, 0); STAGEA(1, 1, a1);
    WAIT_L(8); BAR; WAIT_L(0); MMA(0, 0, At, B0); BAR; SCHED;
    LDB(B1, 0, 1); STAGEB(0, 0, b2);
    BAR; WAIT_L(0); MMA(0, 1, At, B1); BAR;
    LDA(At, 0, 1); STAGEA(0, 0, a2);
    BAR; WAIT_L(0); MMA(1, 0, At, B0); BAR; SCHED;
    STAGEB(0, 1, b2);
    WAIT_V(6);
    const bool look = ready < 2 * it - 10 && it < 30 && wid == 0;
    if (look) asm volatile("s_nop 4\n\tglobal_load_dword %0, %1, %2 sc0 sc1" : "+v"(fl) : "v"(FOFF()), "s"(rdy) : "memory");
    BAR; MMA(1, 1, At, B1); BAR;
    LDB(B0, 1, 0); SCHED; LDA(At, 1, 0); STAGEA(0, 1, a2);
    WAIT_L(8); BAR; WAIT_L(0); MMA(0, 0, At, B0); BAR; SCHED;
    LDB(B1, 1, 1); STAGEB(1, 0, b3);
    BAR; WAIT_L(0); MMA(0, 1, At, B1); BAR;
    LDA(At, 1, 1); STAGEA(1, 0, a3);
    BAR; WAIT_L(0); MMA(1, 0, At, B0); BAR; SCHED;
    STAGEB(1, 1, b3);
    asm volatile("s_waitcnt vmcnt(6)" : "+v"(fl) :: "memory");
    if (look) { READY(); *seen = ready; }
    BAR; MMA(1, 1, At, B1); BAR;
    a1 = a3;
  }
  if (wr == 0) BAR;
  STAMP(5);

  int ln = tid; asm volatile("" : "+v"(ln)); ln &= 63;
  {
    const unsigned tmo = aload(ws + OFF_TMO);
    const unsigned slot = (tm * 16 + tn) * 8 + wid;
    const rsrc_t rpo = mk(ws + OFF_POUT + (size_t)par * 24 * MiB + (size_t)slot * 12288, 12288), rpc = mk(ws + OFF_POUTC + (size_t)par * 16 * MiB + (size_t)slot * 8192, 8192);
    char* const dst = p.ws[(tm >> 1) & 7] + OFF_HOUT + (size_t)(((par * 8 + rank) * 32 + (tm & 1) * 16 + tn) * 8 + wid) * 32 + (ln & 3) * 8;
    u32x2 hs = {call, 0u}; unsigned base = 0;
    if (wr == 0) __builtin_amdgcn_s_setprio(2);
#pragma unroll
    for (int u = 0; u < 4; ++u) {
      u32x4 r[4];
#pragma unroll
      for (int c = 0; c < 4; ++c) {
        const f32x4 x = acc[u >> 1][c & 1][(u & 1) * 2 + (c >> 1)][0], y = acc[u >> 1][c & 1][(u & 1) * 2 + (c >> 1)][1];
        const auto s0 = __builtin_amdgcn_permlane32_swap(cvt2(x[0], x[1]), cvt2(y[0], y[1]), false, false);
        const auto s1 = __builtin_amdgcn_permlane32_swap(cvt2(x[2], x[3]), cvt2(y[2], y[3]), false, false);
        r[c] = (u32x4){s0[0], s1[0], s0[1], s1[1]};
      }
      if (u == 0) base = base_of(r);
      const unsigned h = pack_unit(r, base, u & 1, ln, scr, rpo, u * 3072, rpc, u * 2048);
      if (ln == u) hs[1] |= h;
      if (u == 2 && tmo == call) hs[1] |= 0x1000000u;
      if (EPI2 && u == 2) {
        asm volatile("s_waitcnt vmcnt(%0)" :: "i"(NLD) : "memory");
        if (ln < 2) asm volatile("global_store_dwordx2 %0, %1, off sc0 sc1\n\ts_nop 1" :: "v"(dst), "v"(hs) : "memory"); }
    }
    WAIT_V(0);
    if (ln < 4 && ln >= 2 * EPI2) asm volatile("global_store_dwordx2 %0, %1, off sc0 sc1\n\ts_nop 1" :: "v"(dst), "v"(hs) : "memory");
  }
  STAMP(6);

  const unsigned gr = wid * (8 * NRED) + ((31 - idx) * 8 + xcd + wid) % (8 * NRED), col = gr & 3, widx = gr >> 2;
  if (idx >= 32 - NRED && widx < RC) {
    const unsigned qb = (call - 1) * 512 - RC;
    char* const ctr = ws + OFF_Q + (18 + col) * 128;
    const char* const w0 = p.ws[col * 2]; const char* const w1 = p.ws[col * 2 + 1];
    const size_t mine = (size_t)rank * 1024;
    const rsrc_t r0 = mk(w0 + OFF_POUT + (size_t)par * 24 * MiB + mine * 3072, 3u << 20), r1 = mk(w1 + OFF_POUT + (size_t)par * 24 * MiB + mine * 3072, 3u << 20);
    const rsrc_t c0 = mk(w0 + OFF_POUTC + (size_t)par * 16 * MiB + mine * 2048, 2u << 20), c1 = mk(w1 + OFF_POUTC + (size_t)par * 16 * MiB + mine * 2048, 2u << 20);
    const rsrc_t rh = mk(ws + OFF_HOUT + (size_t)(par * 8 + col * 2) * 8192, 16384), ro = mk(p.out, 16u << 20);
    const unsigned lo = (ln & 15) * SB_ + (ln >> 4) * 16, ho = (ln >> 5) * 8192;
#define OENT(L) (((((L) >> 2) & 3) << 8) | ((7 - (((L) >> 6) & 7)) << 5) | ((((L) >> 5) & 1) << 4) | (((L) & 3) << 2) | ((((L) >> 4) & 1) << 1))
    u32x4 hv; Unit da[2], db[2]; unsigned cv, Ea = OENT(widx), Eb, Ln, ha[2], hb[2], hn[2]; bool good = true;
    zero_scr(scr, ln);
    if (col != (unsigned)rcol) {
      const unsigned sg = ((col ^ rcol) - 1) * 2 + (ln & 1) + widx % PW * 6;
      const u64 t0 = __builtin_amdgcn_s_memrealtime();
      while (__builtin_amdgcn_ballot_w64(aload(ws + OFF_HAND + (512 + sg) * HST) != call) && !give_up(ws, call, t0)) { }
    }
    STAMP(11);
#define OUT_PAIR() do { good = wait_hdr(hv, rh, ho, Ea * 8, ws, call) && good && !__builtin_amdgcn_ballot_w64(((hv[1] | hv[3]) >> 24) != 0); \
      ha[0] = RL(hv[1], 0); ha[1] = RL(hv[1], 32); hn[0] = RL(hv[3], 0); hn[1] = RL(hv[3], 32); \
      load_unit(da[0], r0, Ea * 3072, ha[0], false, ln); load_unit(da[1], r1, Ea * 3072, ha[1], false, ln); CLAIM(cv, ctr, 1); } while (0)
    hv = ld16(rh, ho, Ea * 8); OUT_PAIR();
    STAMP(10);
#define OUT_WAIT(d) asm volatile("s_waitcnt vmcnt(0)" : "+v"(d[0].a0), "+v"(d[0].a1), "+v"(d[0].p), "+v"(d[0].pt), "+v"(d[1].a0), "+v"(d[1].a1), "+v"(d[1].p), "+v"(d[1].pt), "+v"(hv), "+v"(cv) :: "memory")
#define OUT_SUM(d, Ec, h, odd) do { FENCE; unsigned T0[8], T1[8]; \
      unpack_hi(T0, d[0], h[0], odd, ln, scr, c0, Ec * 2048); unpack_hi(T1, d[1], h[1], odd, ln, scr, c1, Ec * 2048); \
      const unsigned _ob = ((col >> 1) * 512 + (Ec >> 9) * 256 + ((Ec >> 1) & 1) * 128 + ((Ec >> 4) & 1) * 64 + (Ec & 1) * 32) * SB_ + ((col & 1) * 4096 + ((Ec >> 5) & 15) * 256 + ((Ec >> 2) & 3) * 32) * 2; \
      _Pragma("unroll") for (int c = 0; c < 4; ++c) { const u32x4 x = unpack_row(T0, d[0], c, h[0]), y = unpack_row(T1, d[1], c, h[1]); u32x4 o; \
        _Pragma("unroll") for (int e = 0; e < 4; ++e) { \
          const float v0 = __builtin_bit_cast(float, x[e] << 16) + __builtin_bit_cast(float, y[e] << 16), v1 = __builtin_bit_cast(float, x[e] & 0xFFFF0000u) + __builtin_bit_cast(float, y[e] & 0xFFFF0000u); \
          o[e] = __builtin_bit_cast(unsigned, __builtin_elementwise_max(__builtin_bit_cast(s16x2, cvt2(v0, v1)), (s16x2){0, 0})); } \
        if (!good) o = (u32x4){0x7FC07FC0u, 0x7FC07FC0u, 0x7FC07FC0u, 0x7FC07FC0u}; \
        __builtin_amdgcn_raw_buffer_store_b128(o, ro, lo + _ob + (c >> 1) * 16 * SB_ + (c & 1) * 256, 0, 16); } \
      FENCE; } while (0)
    do {
      OUT_WAIT(da);
      Eb = Ea + 1; hb[0] = hn[0]; hb[1] = hn[1];
      load_unit(db[0], r0, Eb * 3072, hb[0], true, ln); load_unit(db[1], r1, Eb * 3072, hb[1], true, ln);
      Ln = RFL(cv) - qb; hv = ld16(rh, ho, OENT(Ln) * 8);
      OUT_SUM(da, Ea, ha, false);
      OUT_WAIT(db);
      db[0].pt = ln < 16 ? da[0].pt : db[0].pt; db[1].pt = ln < 16 ? da[1].pt : db[1].pt;
      Ea = OENT(Ln);
      if (Ln < 512) OUT_PAIR();
      OUT_SUM(db, Eb, hb, true);
    } while (Ln < 512);
  }
  STAMP(7);
}

extern "C" void kernel_launch(int rank, int nranks, ncclComm_t comm,
                              hipStream_t stream,
                              void* const* d_in, const int* in_sizes, int n_in,
                              void* d_out, int out_size,
                              const int* in_dtypes, int out_dtype,
                              void* const* d_out_all,
                              void* const* d_ws_all, size_t ws_bytes) {
  (void)comm; (void)in_dtypes; (void)out_dtype; (void)d_out_all;
  if (nranks != 8 || (unsigned)rank >= 8u || n_in != 2 || ws_bytes < WS_NEED || in_sizes[0] != 8192 * 1024 || in_sizes[1] != 8192 * 1024 || out_size != 1024 * 8192) {
    fprintf(stderr, "gemm_rs: unexpected problem shape\n"); return; }
  HIP_CHECK(hipFuncSetAttribute((const void*)gemm_rs, hipFuncAttributeMaxDynamicSharedMemorySize, LDS_BYTES));
  Params p;
  p.xin = (const char*)d_in[0]; p.win = (const char*)d_in[1]; p.out = (char*)d_out; p.own = (char*)d_ws_all[rank]; p.rank = rank; p.pad = 0;
  for (int i = 0; i < 8; ++i) p.ws[i] = (char*)d_ws_all[i];
  gemm_rs<<<NWG, 512, LDS_BYTES, stream>>>(p);
  HIP_CHECK(hipGetLastError());
}
```
